```python
import numpy as np

kernel_name = "gpu_gemm_ar_m8192_k8192_n8192_f32_relu"


ABSMAX_RTOL = 0.001

COMPUTE_DTYPE = "float32"
EPILOGUE = "relu"
IN_QUANT = None
OUT_QUANT = None
IN_DTYPES = {"x": "float32", "w_mat": "float32"}
WORLD_SIZE = 8

def setup_inputs():

    x = np.random.randn(8192, 8192).astype(np.float32)
    w_mat = (np.random.randn(8192, 8192) / np.sqrt(8192)).astype(np.float32)
    return {"x": x, "w_mat": w_mat}

def _epilogue(y):
    return np.maximum(y, 0.0).astype(np.float32)

def reference(x, w_mat):
    y = (x @ w_mat).astype(np.float32)
    return _epilogue(y)
```

```cpp
#include <hip/hip_runtime.h>
#include <rccl/rccl.h>
#include <cstdio>

#define HIP_CHECK(expr) do { hipError_t _e = (expr); if (_e != hipSuccess) { \
    fprintf(stderr, "HIP error %s at %s:%d\n", hipGetErrorString(_e), __FILE__, __LINE__); \
    return; } } while (0)

namespace gar {

#define LAS __attribute__((address_space(3)))
typedef _Float16 h16x8 __attribute__((ext_vector_type(8)));
typedef float f32x4 __attribute__((ext_vector_type(4)));
typedef unsigned u32x4 __attribute__((ext_vector_type(4)));
typedef unsigned u32x2 __attribute__((ext_vector_type(2)));
typedef unsigned long long u64;

constexpr int NRANK = 8;
constexpr int GM = 8192, GN = 8192, KS = 1024;
#define NL 32
#define NE 96
#define LW 6
#define LW2 2
#define EW 4
constexpr int NG = 128;
constexpr int GRID = NL + NG + NE;
constexpr int NSTEP = 16;
constexpr int NREC = 2048;
constexpr unsigned REC_BYTES = 17408;

constexpr unsigned OFF_CTR = 0, OFF_TMO = 128, OFF_EXPQ = 256, OFF_LNKQ = 384;
constexpr unsigned OFF_INFLAG = 4096;
constexpr unsigned OFF_OUTFLAG = 65536;
constexpr unsigned OFF_DBG = 2u << 20;
constexpr unsigned OFF_DATA = 4u << 20;
constexpr unsigned A_BYTES = 64u << 20, B_BYTES = 32u << 20, O_BYTES = NRANK * NREC * REC_BYTES;
constexpr unsigned PAR_BYTES = A_BYTES + B_BYTES + O_BYTES;
constexpr unsigned WS_USED = OFF_DATA + 2 * PAR_BYTES;
constexpr unsigned STAGE = 49152, LDS_MISC = 3 * STAGE, LDS_BYTES = LDS_MISC + 1024;
constexpr u64 DEADLINE = 50000000ull;
#define AUX_PULL 17
constexpr int AUX_SYS = 17;

struct Params {
    const float* x; const float* w; float* out; char* ws[NRANK]; int rank;
};

#define RELAY_DEPTH 7
#define WAIT_V(n) WAIT_V_(n)
#define WAIT_V_(n) asm volatile("s_waitcnt vmcnt(" #n ")" ::: "memory")
#define WAIT_L(n) asm volatile("s_waitcnt lgkmcnt(" #n ")" ::: "memory")
#define BAR() do { asm volatile("" ::: "memory"); __builtin_amdgcn_s_barrier(); asm volatile("" ::: "memory"); } while (0)

#define STAMPW(w, i) do {} while (0)
#define STAMP(i) do {} while (0)

__device__ __forceinline__ __amdgpu_buffer_rsrc_t mk_rsrc(const void* p, unsigned bytes) {
    return __builtin_amdgcn_make_buffer_rsrc((void*)p, 0, bytes, 0x00020000);
}
__device__ __forceinline__ u32x4 cvt8(f32x4 a, f32x4 b) {
    h16x8 h;
#pragma unroll
    for (int j = 0; j < 4; ++j) { h[j] = (_Float16)a[j]; h[4 + j] = (_Float16)b[j]; }
    return __builtin_bit_cast(u32x4, h);
}
__device__ __forceinline__ bool timed_out(unsigned* tmo, unsigned call, u64 t0) {
    if ((unsigned)__builtin_amdgcn_readfirstlane(__hip_atomic_load(tmo, __ATOMIC_RELAXED, __HIP_MEMORY_SCOPE_AGENT)) == call) return true;
    if (__builtin_amdgcn_s_memrealtime() - t0 > DEADLINE) {
        __hip_atomic_store(tmo, call, __ATOMIC_RELAXED, __HIP_MEMORY_SCOPE_AGENT);
        return true;
    }
    return false;
}

__device__ __forceinline__ void store_dense(float* out, int row0, int col0, int fr, int fq, const f32x4 (&v)[4][4]) {
    float* p = out + (size_t)(row0 + fr) * GN + col0 + 4 * fq;
#pragma unroll
    for (int m = 0; m < 4; ++m)
#pragma unroll
        for (int r = 0; r < 4; ++r) {
            f32x4 o = {v[m][0][r], v[m][1][r], v[m][2][r], v[m][3][r]};
            *(f32x4*)(p + (size_t)m * 16 * GN + r * 16) = o;
        }
}

__device__ __forceinline__ void link_role(const Params& P, LAS unsigned char* lds, unsigned call, int wid, int lane) {
    const int wg = blockIdx.x, fr = lane & 15, fq = lane >> 4;
    constexpr int PER1 = 1024 / NL, PER2 = 512 / NL, NTICK = 8 * (PER1 + PER2);
    const unsigned par = OFF_DATA + (call & 1u) * PAR_BYTES;
    LAS unsigned* cnt = (LAS unsigned*)(lds + LDS_MISC + 256);
    __amdgpu_buffer_rsrc_t rs[NRANK];
#pragma unroll
    for (int p = 0; p < NRANK; ++p) rs[p] = mk_rsrc(P.ws[p], WS_USED);
    char* myws = P.ws[lane & 7];
    const int rank = P.rank;
    const unsigned lo16 = lane * 16;
    if (wid >= LW) return;
    int pending = -1;
    auto signal = [&](int st) {
        unsigned old = 0;
        if (lane == 0) old = __hip_atomic_fetch_add(cnt + st, 1u, __ATOMIC_RELAXED, __HIP_MEMORY_SCOPE_WORKGROUP);
        old = __builtin_amdgcn_readfirstlane(old);
        if (old == (unsigned)(st < 8 ? PER1 - 1 : PER2 - 1)) {
            if (lane < NRANK)
                __hip_atomic_store((unsigned*)(myws + OFF_INFLAG + ((st * 8 + rank) * 64 + wg) * 4), call, __ATOMIC_RELAXED, __HIP_MEMORY_SCOPE_SYSTEM);
            STAMPW(0, 1 + st);
        }
    };
    for (;;) {
        unsigned tk = 0;
        if (lane == 0) tk = __hip_atomic_fetch_add(cnt + 16, 1u, __ATOMIC_RELAXED, __HIP_MEMORY_SCOPE_WORKGROUP);
        tk = __builtin_amdgcn_readfirstlane(tk);
        if (tk >= (unsigned)NTICK) break;
        int st, t;
        if (tk < (unsigned)(8 * PER1)) { st = tk / PER1; t = tk - st * PER1; } else { const int u = tk - 8 * PER1; st = u / PER2; t = u - st * PER2; st += 8; }
        const int g = st >> 3, j = st & 7;
        const bool isw = (g == 0) && (t & 1);
        const int idx = (g == 0 ? (t >> 1) : t) * NL + wg;
        if (!isw) {
            const int rgg = idx >> 2, kq = idx & 3;
            const int lrow = g * 2048 + rgg * 16;
            const float* s0 = P.x + (size_t)(lrow + fr) * KS + j * 128 + kq * 32 + fq * 8;
            const float* s1 = s0 + (size_t)4096 * KS;
            f32x4 a0 = *(const f32x4*)s0, a1 = *(const f32x4*)(s0 + 4), b0 = *(const f32x4*)s1, b1 = *(const f32x4*)(s1 + 4);
            WAIT_V(0);
            if (pending >= 0) signal(pending);
            const u32x4 h0 = cvt8(a0, a1), h1 = cvt8(b0, b1);
            const int mt = lrow >> 7, rg = rgg & 7, kt = j * 16 + rank * 2 + (kq >> 1);
            const unsigned off = par + (unsigned)((((mt * 128 + kt) * 8 + rg) * 2 + (kq & 1)) * 1024);
#pragma unroll
            for (int q = 0; q < 4; ++q) {
                __builtin_amdgcn_raw_buffer_store_b128(h0, rs[q], lo16, off, AUX_SYS);
                __builtin_amdgcn_raw_buffer_store_b128(h1, rs[4 + q], lo16, off, AUX_SYS);
            }
        } else {
            const int b = (idx + (idx >> 5)) & 3, rest = idx >> 2, kq = rest & 3, cgg = rest >> 2;
            const int pcol = 4 * (4 * (fr & 3) + (fr >> 2));
            const float* s = P.w + (size_t)(j * 128 + kq * 32 + fq * 8) * GN + b * 2048 + cgg * 64 + pcol;
            f32x4 v[8];
#pragma unroll
            for (int e = 0; e < 8; ++e) v[e] = *(const f32x4*)(s + (size_t)e * GN);
            WAIT_V(0);
            if (pending >= 0) signal(pending);
            const int nt = cgg >> 2, cg = cgg & 3, kt = j * 16 + rank * 2 + (kq >> 1);
            const unsigned off = par + A_BYTES + (unsigned)(((((nt * 128 + kt) * 4 + cg) * 4) * 2 + (kq & 1)) * 1024);
            const __amdgpu_buffer_rsrc_t r0 = mk_rsrc(P.ws[b], WS_USED), r1 = mk_rsrc(P.ws[b + 4], WS_USED);
#pragma unroll
            for (int c = 0; c < 4; ++c) {
                const f32x4 lo = {v[0][c], v[1][c], v[2][c], v[3][c]}, hi = {v[4][c], v[5][c], v[6][c], v[7][c]};
                const u32x4 h = cvt8(lo, hi);
                __builtin_amdgcn_raw_buffer_store_b128(h, r0, lo16, off + c * 2048, AUX_SYS);
                __builtin_amdgcn_raw_buffer_store_b128(h, r1, lo16, off + c * 2048, AUX_SYS);
            }
        }
        pending = st;
    }
    WAIT_V(0);
    if (pending >= 0) signal(pending);
    STAMP(20);
    if (wid >= LW2) return;
    char* wsl = P.ws[rank];
    unsigned* tmo = (unsigned*)(wsl + OFF_TMO);
    unsigned* q = (unsigned*)(wsl + OFF_LNKQ);
    const unsigned qbase = (call - 1u) * (unsigned)(NREC + NL * LW2);
    unsigned tk = 0;
    if (lane == 0) tk = __hip_atomic_fetch_add(q, 1u, __ATOMIC_RELAXED, __HIP_MEMORY_SCOPE_AGENT);
    int prec = -1; unsigned pnnz = 0;
    auto publish = [&]() {
        if (prec >= 0 && lane < NRANK && lane != rank)
            __hip_atomic_store((u64*)(myws + OFF_OUTFLAG + (unsigned)(rank * NREC + prec) * 64u), ((u64)call << 32) | pnnz, __ATOMIC_RELAXED, __HIP_MEMORY_SCOPE_SYSTEM);
        prec = -1;
    };
    for (;;) {
        const unsigned rec = (unsigned)__builtin_amdgcn_readfirstlane(tk) - qbase;
        if (rec >= (unsigned)NREC) break;
        if (lane == 0) tk = __hip_atomic_fetch_add(q, 1u, __ATOMIC_RELAXED, __HIP_MEMORY_SCOPE_AGENT);
        const u64* flag = (const u64*)(wsl + OFF_OUTFLAG + (unsigned)(rank * NREC + rec) * 64u);
        const u64 t0 = __builtin_amdgcn_s_memrealtime();
        unsigned nnz = 0; bool bad = false;
        for (;;) {
            const u64 f = __hip_atomic_load(flag, __ATOMIC_RELAXED, __HIP_MEMORY_SCOPE_AGENT);
            const unsigned fc = __builtin_amdgcn_readfirstlane((unsigned)(f >> 32));
            nnz = __builtin_amdgcn_readfirstlane((unsigned)f);
            WAIT_V(0);
            publish();
            if (fc >= call) break;
            if (timed_out(tmo, call, t0)) { bad = true; break; }
            __builtin_amdgcn_s_sleep(8);
        }
        if (bad || nnz > 4096u) continue;
        const unsigned bytes = (512u + nnz * 4u + 15u) & ~15u;
        const unsigned ro = par + A_BYTES + B_BYTES + (unsigned)(rank * NREC + rec) * REC_BYTES;
        const __amdgpu_buffer_rsrc_t rl = mk_rsrc(wsl + ro, bytes);
        u32x4 d[17];
#pragma unroll
        for (int i = 0; i < 17; ++i) if (i * 1024u < bytes) d[i] = __builtin_amdgcn_raw_buffer_load_b128(rl, lo16, i * 1024, AUX_SYS);
        __amdgpu_buffer_rsrc_t rp[7];
#pragma unroll
        for (int k = 0; k < 7; ++k) rp[k] = mk_rsrc(P.ws[(rank + 1 + k) & 7] + ro, bytes);
        WAIT_V(0);
#pragma unroll
        for (int i = 0; i < 17; ++i) if (i * 1024u < bytes) {
            WAIT_V(RELAY_DEPTH);
#pragma unroll
            for (int k = 0; k < 7; ++k) __builtin_amdgcn_raw_buffer_store_b128(d[i], rp[k], lo16, i * 1024, AUX_SYS);
        }
        prec = (int)rec; pnnz = nnz;
    }
    WAIT_V(0);
    publish();
}

__device__ __forceinline__ void wait_inputs(const Params& P, int st, unsigned call, int lane) {
    const __amdgpu_buffer_rsrc_t r = mk_rsrc(P.ws[P.rank], WS_USED);
    unsigned* tmo = (unsigned*)(P.ws[P.rank] + OFF_TMO);
    const bool valid = (lane & 15) * 4 < NL;
    const u64 t0 = __builtin_amdgcn_s_memrealtime();
    for (;;) {
        const u32x4 a = __builtin_amdgcn_raw_buffer_load_b128(r, lane * 16, OFF_INFLAG + st * 2048, AUX_SYS);
        const u32x4 b = __builtin_amdgcn_raw_buffer_load_b128(r, lane * 16 + 1024, OFF_INFLAG + st * 2048, AUX_SYS);
        const unsigned ma = min(min(a[0], a[1]), min(a[2], a[3])), mb = min(min(b[0], b[1]), min(b[2], b[3]));
        const bool ok = !valid || min(ma, mb) >= call;
        if (__builtin_amdgcn_ballot_w64(!ok) == 0ull) break;
        if (timed_out(tmo, call, t0)) break;
        __builtin_amdgcn_s_sleep(8);
    }
    __builtin_amdgcn_fence(__ATOMIC_ACQUIRE, "");
    WAIT_V(0);
}

__device__ __forceinline__ void gemm_role(const Params& P, LAS unsigned char* lds, unsigned call, int wid, int lane) {
    const int gi = blockIdx.x - NL, xcd = gi & 7, ix = gi >> 3;
    const int wr = wid >> 2, wc = wid & 3, fr = lane & 15, fq = lane >> 4;
    const unsigned par = OFF_DATA + (call & 1u) * PAR_BYTES;
    char* wsl = P.ws[P.rank];
    const int nt = (xcd & 1) * 4 + (ix & 3);
    const unsigned aoff = (unsigned)(wr * 8192 + lane * 16), boff = (unsigned)(16384 + wc * 8192 + lane * 16);
    for (int job = 0; job < 2; ++job) {
        const int mt = job * 16 + (xcd >> 1) * 4 + (ix >> 2);
        const char* ga = wsl + par + (size_t)mt * (128 * 16384) + wid * 2048 + lane * 16;
        const char* gb = wsl + par + A_BYTES + (size_t)nt * (128 * 32768) + wid * 4096 + lane * 16;
#define ISSUE(kt_, sb_) do { const char* _a = ga + (size_t)(kt_) * 16384; const char* _b = gb + (size_t)(kt_) * 32768; \
        LAS unsigned char* _la = lds + (sb_) + wid * 2048; LAS unsigned char* _lb = lds + (sb_) + 16384 + wid * 4096; \
        __builtin_amdgcn_global_load_lds((const unsigned*)_a, (LAS unsigned*)_la, 16, 0, 0); \
        __builtin_amdgcn_global_load_lds((const unsigned*)_a, (LAS unsigned*)_la, 16, 1024, 0); \
        __builtin_amdgcn_global_load_lds((const unsigned*)_b, (LAS unsigned*)_lb, 16, 0, 0); \
        __builtin_amdgcn_global_load_lds((const unsigned*)_b, (LAS unsigned*)_lb, 16, 1024, 0); \
        __builtin_amdgcn_global_load_lds((const unsigned*)_b, (LAS unsigned*)_lb, 16, 2048, 0); \
        __builtin_amdgcn_global_load_lds((const unsigned*)_b, (LAS unsigned*)_lb, 16, 3072, 0); } while (0)
#define LDF(A_, B_, sb_, kf_) do { _Pragma("unroll") for (int i = 0; i < 4; ++i) { \
        B_[i] = *(const LAS h16x8*)(lds + (sb_) + boff + i * 2048 + (kf_) * 1024); \
        A_[i] = *(const LAS h16x8*)(lds + (sb_) + aoff + i * 2048 + (kf_) * 1024); } } while (0)
#define MMA(A_, B_) do { __builtin_amdgcn_s_setprio(1); _Pragma("unroll") for (int m = 0; m < 4; ++m) _Pragma("unroll") for (int c = 0; c < 4; ++c) \
        acc[m][c] = __builtin_amdgcn_mfma_f32_16x16x32_f16(B_[c], A_[m], acc[m][c], 0, 0, 0); __builtin_amdgcn_s_setprio(0); } while (0)
        f32x4 acc[4][4];
#pragma unroll
        for (int m = 0; m < 4; ++m)
#pragma unroll
            for (int c = 0; c < 4; ++c) acc[m][c] = (f32x4){0.f, 0.f, 0.f, 0.f};
        h16x8 A0[4], B0[4], A1[4], B1[4];
        if (wid == 0) wait_inputs(P, job * 8, call, lane);
        BAR();
        STAMP(1 + job * 8);
        ISSUE(0, 0); ISSUE(1, STAGE); ISSUE(2, 2 * STAGE);
        WAIT_V(12); BAR();
        LDF(A0, B0, 0u, 0);
        unsigned sb = 0;
        for (int kt = 0; kt < 128; ++kt) {
            const unsigned sn = sb == 2 * STAGE ? 0u : sb + STAGE;
            __builtin_amdgcn_s_waitcnt(0xC07F);
            LDF(A1, B1, sb, 1);
            MMA(A0, B0);
            if (((kt + 3) & 15) == 0 && kt + 3 < 128 && wid == 0) { STAMP(32 + job * 8 + ((kt + 3) >> 4)); wait_inputs(P, job * 8 + ((kt + 3) >> 4), call, lane); STAMP(16 + job * 8 + ((kt + 3) >> 4)); }
            WAIT_V(6); __builtin_amdgcn_s_waitcnt(0xC07F); BAR();
            ISSUE((kt + 3) & 127, sb);
            LDF(A0, B0, sn, 0);
            MMA(A1, B1);
            sb = sn;
        }
        WAIT_V(0); WAIT_L(0); BAR();
        STAMP(2 + job * 8);
        unsigned* tmo = (unsigned*)(wsl + OFF_TMO);
        const bool bad = (unsigned)__builtin_amdgcn_readfirstlane(__hip_atomic_load(tmo, __ATOMIC_RELAXED, __HIP_MEMORY_SCOPE_AGENT)) == call;
        LAS float* cv = (LAS float*)(lds + wid * 16384);
        unsigned mlo = 0, mhi = 0, off = 0;
#pragma unroll
        for (int m = 0; m < 4; ++m)
#pragma unroll
            for (int c = 0; c < 4; ++c)
#pragma unroll
                for (int r = 0; r < 4; ++r) {
                    float v = acc[m][c][r];
                    v = v <= 0.f ? 0.f : v;
                    if (bad) v = __builtin_nanf("");
                    acc[m][c][r] = v;
                    const bool nz = __float_as_uint(v) != 0u;
                    const u64 mk = __builtin_amdgcn_ballot_w64(nz);
                    const unsigned lo = (unsigned)mk, hi = (unsigned)(mk >> 32);
                    const unsigned pos = off + __builtin_amdgcn_mbcnt_hi(hi, __builtin_amdgcn_mbcnt_lo(lo, 0u));
                    if (nz) cv[pos] = v;
                    const int q = (m * 4 + c) * 4 + r;
                    mlo = lane == q ? lo : mlo; mhi = lane == q ? hi : mhi;
                    off += (unsigned)__builtin_popcountll(mk);
                }
        const int rec = (mt * 8 + nt) * 8 + wid;
        const __amdgpu_buffer_rsrc_t rl = mk_rsrc(wsl, WS_USED);
        const unsigned ro = par + A_BYTES + B_BYTES + (unsigned)(P.rank * NREC + rec) * REC_BYTES;
        __builtin_amdgcn_raw_buffer_store_b64((u32x2){mlo, mhi}, rl, lane * 8, ro, AUX_SYS);
        WAIT_L(0);
        for (unsigned i = 0; i * 256 < off; ++i) {
            const u32x4 d = *(const LAS u32x4*)((LAS unsigned char*)cv + i * 1024 + lane * 16);
            __builtin_amdgcn_raw_buffer_store_b128(d, rl, lane * 16, ro + 512 + i * 1024, AUX_SYS);
        }
        WAIT_V(0);
        if (lane == 0)
            __hip_atomic_store((u64*)(wsl + OFF_OUTFLAG + (unsigned)(P.rank * NREC + rec) * 64u), ((u64)call << 32) | off, __ATOMIC_RELAXED, __HIP_MEMORY_SCOPE_AGENT);
        store_dense(P.out, (P.rank >> 2) * 4096 + mt * 128 + wr * 64, (P.rank & 3) * 2048 + nt * 256 + wc * 64, fr, fq, acc);
        WAIT_L(0); BAR();
        STAMP(3 + job * 8);
#undef ISSUE
#undef LDF
#undef MMA
    }
}

__device__ __forceinline__ void expand_role(const Params& P, LAS unsigned char* lds, unsigned call, int wid, int lane) {
    const int fr = lane & 15, fq = lane >> 4;
    constexpr unsigned NJOB = 7 * NREC, E = NE * EW;
    if (wid >= EW) return;
    const unsigned par = OFF_DATA + (call & 1u) * PAR_BYTES;
    char* wsl = P.ws[P.rank];
    unsigned* tmo = (unsigned*)(wsl + OFF_TMO);
    LAS unsigned char* buf = lds + wid * 17408;
    unsigned* q = (unsigned*)(wsl + OFF_EXPQ);
    const unsigned qbase = (call - 1u) * (NJOB + E);
    unsigned tk = 0;
    if (lane == 0) tk = __hip_atomic_fetch_add(q, 1u, __ATOMIC_RELAXED, __HIP_MEMORY_SCOPE_AGENT);
    int it = 0;
    for (;; ++it) {
        const unsigned ku = (unsigned)__builtin_amdgcn_readfirstlane(tk) - qbase;
        if (ku >= NJOB) break;
        const int k = (int)ku;
        if (lane == 0) tk = __hip_atomic_fetch_add(q, 1u, __ATOMIC_RELAXED, __HIP_MEMORY_SCOPE_AGENT);
        const int rec = k / 7, src = (P.rank + 1 + (k - rec * 7)) & 7;
        const u64* flag = (const u64*)(wsl + OFF_OUTFLAG + (unsigned)(src * NREC + rec) * 64u);
        const u64 t0 = __builtin_amdgcn_s_memrealtime();
        unsigned nnz = 0; bool bad = false;
        for (;;) {
            const u64 f = __hip_atomic_load(flag, __ATOMIC_RELAXED, __HIP_MEMORY_SCOPE_SYSTEM);
            const unsigned fc = __builtin_amdgcn_readfirstlane((unsigned)(f >> 32));
            nnz = __builtin_amdgcn_readfirstlane((unsigned)f);
            if (fc >= call) break;
            if (timed_out(tmo, call, t0)) { bad = true; break; }
            __builtin_amdgcn_s_sleep(16);
        }
        if (nnz > 4096u) { nnz = 0; bad = true; }
        f32x4 v[4][4];
        if (!bad) {
            const char* rbase = wsl + par + A_BYTES + B_BYTES + (size_t)(src * NREC + rec) * REC_BYTES;
            const __amdgpu_buffer_rsrc_t rr = mk_rsrc(rbase, REC_BYTES);
            const u32x2 hdr = __builtin_amdgcn_raw_buffer_load_b64(rr, lane * 8, 0, AUX_SYS);
            const char* gsrc = rbase + 512 + lane * 16;
            const unsigned bytes = nnz * 4;
            for (unsigned i = 0; i < bytes; i += 1024) {
                if (i + lane * 16 < bytes)
                    __builtin_amdgcn_global_load_lds((const unsigned*)(gsrc + i), (LAS unsigned*)(buf + i), 16, 0, AUX_PULL);
            }
            WAIT_V(0);
            unsigned off = 0;
#pragma unroll
            for (int m = 0; m < 4; ++m)
#pragma unroll
                for (int c = 0; c < 4; ++c)
#pragma unroll
                    for (int r = 0; r < 4; ++r) {
                        const int q = (m * 4 + c) * 4 + r;
                        const unsigned lo = __builtin_amdgcn_readlane(hdr[0], q), hi = __builtin_amdgcn_readlane(hdr[1], q);
                        const u64 mk = ((u64)hi << 32) | lo;
                        const unsigned pos = (off + __builtin_amdgcn_mbcnt_hi(hi, __builtin_amdgcn_mbcnt_lo(lo, 0u))) & 4095u;
                        const float t = *(const LAS float*)(buf + pos * 4);
                        v[m][c][r] = __builtin_amdgcn_inverse_ballot_w64(mk) ? t : 0.f;
                        off += (unsigned)__builtin_popcountll(mk);
                    }
        } else {
#pragma unroll
            for (int m = 0; m < 4; ++m)
#pragma unroll
                for (int c = 0; c < 4; ++c) { const float q = __builtin_nanf(""); v[m][c] = (f32x4){q, q, q, q}; }
        }
        const int w8 = rec & 7, nt = (rec >> 3) & 7, mt = rec >> 6;
        store_dense(P.out, (src >> 2) * 4096 + mt * 128 + (w8 >> 2) * 64, (src & 3) * 2048 + nt * 256 + (w8 & 3) * 64, fr, fq, v);
        WAIT_L(0);
        if (it < 60) STAMP(1 + it);
    }
}

__global__ __launch_bounds__(512, 2) void fused_kernel(Params P) {
    extern __shared__ __attribute__((aligned(16))) unsigned char shm[];
    LAS unsigned char* lds = (LAS unsigned char*)shm;
    const int tid = threadIdx.x, lane = tid & 63, wid = __builtin_amdgcn_readfirstlane(tid >> 6);
    LAS unsigned* misc = (LAS unsigned*)(lds + LDS_MISC);
    STAMP(0);
    if (tid == 0) {
        const unsigned t = __hip_atomic_fetch_add((unsigned*)(P.ws[P.rank] + OFF_CTR), 1u, __ATOMIC_RELAXED, __HIP_MEMORY_SCOPE_AGENT);
        misc[0] = t / (unsigned)GRID + 1u;
    }
    if (tid >= 64 && tid < 64 + NSTEP + 1) misc[tid] = 0u;
    WAIT_V(0); WAIT_L(0); BAR();
    const unsigned call = __builtin_amdgcn_readfirstlane(misc[0]);
    if (blockIdx.x < NL) link_role(P, lds, call, wid, lane);
    else if (blockIdx.x < NL + NG) gemm_role(P, lds, call, wid, lane);
    else expand_role(P, lds, call, wid, lane);
    STAMP(63);
}

}

extern "C" void kernel_launch(int rank, int nranks, ncclComm_t comm,
                              hipStream_t stream,
                              void* const* d_in, const int* in_sizes, int n_in,
                              void* d_out, int out_size,
                              const int* in_dtypes, int out_dtype,
                              void* const* d_out_all,
                              void* const* d_ws_all, size_t ws_bytes) {
    using namespace gar;
    if (nranks != NRANK || (unsigned)rank >= (unsigned)NRANK || ws_bytes < (size_t)WS_USED || n_in < 2 ||
        in_sizes[0] != GM * KS || in_sizes[1] != KS * GN || out_size != GM * GN) {
        fprintf(stderr, "kernel_launch: unexpected problem shape\n");
        return;
    }
    Params P;
    P.x = (const float*)d_in[0]; P.w = (const float*)d_in[1]; P.out = (float*)d_out; P.rank = rank;
    for (int p = 0; p < NRANK; ++p) P.ws[p] = (char*)d_ws_all[p];
    HIP_CHECK(hipFuncSetAttribute((const void*)fused_kernel, hipFuncAttributeMaxDynamicSharedMemorySize, LDS_BYTES));
    hipLaunchKernelGGL(fused_kernel, dim3(GRID), dim3(512), LDS_BYTES, stream, P);
    HIP_CHECK(hipGetLastError());
}
```

```cpp
#include <hip/hip_runtime.h>
#include <rccl/rccl.h>
#include <cstdio>

#define HIP_CHECK(expr) do { hipError_t _e = (expr); if (_e != hipSuccess) { \
    fprintf(stderr, "HIP error %s at %s:%d\n", hipGetErrorString(_e), __FILE__, __LINE__); \
    return; } } while (0)

namespace gar {

#define LAS __attribute__((address_space(3)))
typedef _Float16 h16x8 __attribute__((ext_vector_type(8)));
typedef float f32x4 __attribute__((ext_vector_type(4)));
typedef unsigned u32x4 __attribute__((ext_vector_type(4)));
typedef unsigned u32x2 __attribute__((ext_vector_type(2)));
typedef unsigned long long u64;

constexpr int NRANK = 8;
constexpr int GM = 8192, GN = 8192, KS = 1024;
#define NL 32
#define NE 96
#define LW 3
#define LW2 2
#define EW 3
#define RELAY_DEPTH 7
#define SC_DELAY 32
#define GSLEEP 32
#define WT_UNTIL 300
#define ESLEEP 4
constexpr int NG = 128;
constexpr int NA = 64;
constexpr int GRID = NL + NG + NE;
constexpr int NSTEP = 16;
constexpr int NREC = 2048;
constexpr unsigned REC_BYTES = 17408;
static_assert(GRID % 8 == 0 && NE >= NA && NL == 32 && LW2 <= LW, "geometry");

constexpr unsigned OFF_TMO = 0, OFF_EXPQ = 128, OFF_LNKQ = 256, OFF_CTR = 1024;
constexpr unsigned OFF_INFLAG = 4096;
constexpr unsigned OFF_AMAX = 40960;
constexpr unsigned OFF_OUTFLAG = 65536;
constexpr unsigned OFF_DBG = 2u << 20;
constexpr unsigned OFF_DATA = 4u << 20;
constexpr unsigned A_BYTES = 64u << 20, B_BYTES = 32u << 20, O_BYTES = NRANK * NREC * REC_BYTES;
constexpr unsigned CHUNK = 512u << 10;
constexpr unsigned PAR_BYTES = A_BYTES + B_BYTES + O_BYTES;
constexpr unsigned WS_USED = OFF_DATA + 2 * PAR_BYTES;
constexpr unsigned STAGE = 49152, LDS_MISC = 3 * STAGE, LDS_BYTES = LDS_MISC + 2048;
constexpr u64 DEADLINE = 50000000ull;
constexpr int AUX_SYS = 17;
constexpr int EXP_TARGET = 142;

struct Params {
    const float* x; const float* w; float* out; char* ws[NRANK]; int rank;
};

#define WAIT_V(n) WAIT_V_(n)
#define WAIT_V_(n) asm volatile("s_waitcnt vmcnt(" #n ")" ::: "memory")
#define WAIT_L(n) asm volatile("s_waitcnt lgkmcnt(" #n ")" ::: "memory")
#define BAR() do { asm volatile("" ::: "memory"); __builtin_amdgcn_s_barrier(); asm volatile("" ::: "memory"); } while (0)

#define STAMPW(w, i) do {} while (0)
#define STAMP(i) do {} while (0)

__device__ __forceinline__ __amdgpu_buffer_rsrc_t mk_rsrc(const void* p, unsigned bytes) {
    return __builtin_amdgcn_make_buffer_rsrc((void*)p, 0, bytes, 0x00020000);
}
__device__ __forceinline__ void st16_sys(char* p, u32x4 v) {
    __builtin_amdgcn_global_store_b128((__attribute__((address_space(1))) u32x4*)p, v, "");
}
__device__ __forceinline__ u32x4 cvt8(f32x4 a, f32x4 b, int sh) {
    h16x8 h;
#pragma unroll
    for (int j = 0; j < 4; ++j) { h[j] = (_Float16)__builtin_ldexpf(a[j], sh); h[4 + j] = (_Float16)__builtin_ldexpf(b[j], sh); }
    return __builtin_bit_cast(u32x4, h);
}
struct Deadline {
    unsigned* tmo; unsigned call; u64 t0; unsigned polls;
    __device__ __forceinline__ Deadline(unsigned* tmo_, unsigned call_) : tmo(tmo_), call(call_), t0(__builtin_amdgcn_s_memrealtime()), polls(1) {}
    __device__ __forceinline__ bool expired() {
        if ((polls++ & 63u) == 0u &&
            (unsigned)__builtin_amdgcn_readfirstlane(__hip_atomic_load(tmo, __ATOMIC_RELAXED, __HIP_MEMORY_SCOPE_AGENT)) == call) return true;
        if (__builtin_amdgcn_s_memrealtime() - t0 > DEADLINE) {
            __hip_atomic_store(tmo, call, __ATOMIC_RELAXED, __HIP_MEMORY_SCOPE_AGENT);
            return true;
        }
        return false;
    }
};
__device__ __forceinline__ unsigned wave_max8(unsigned v) {
    unsigned r = 0;
#pragma unroll
    for (int b = 7; b >= 0; --b) { const unsigned c = r | (1u << b); r = __builtin_amdgcn_ballot_w64(v >= c) != 0ull ? c : r; }
    return r;
}
__device__ __forceinline__ unsigned exp1_max4(unsigned m, u32x4 v) {
#pragma unroll
    for (int i = 0; i < 4; ++i) m = max(m, (v[i] + 0x00800000u) & 0x7f800000u);
    return m;
}

template <bool WT>
__device__ __forceinline__ void store_dense(float* out, int row0, int col0, int fr, int fq, const f32x4 (&v)[4][4]) {
    float* p = out + (size_t)(row0 + fr) * GN + col0 + 4 * fq;
#pragma unroll
    for (int m = 0; m < 4; ++m)
#pragma unroll
        for (int r = 0; r < 4; ++r) {
            f32x4 o = {v[m][0][r], v[m][1][r], v[m][2][r], v[m][3][r]};
            if (WT) st16_sys((char*)(p + (size_t)m * 16 * GN + r * 16), __builtin_bit_cast(u32x4, o));
            else *(f32x4*)(p + (size_t)m * 16 * GN + r * 16) = o;
        }
}

template <class F>
__device__ __forceinline__ void link_role(const Params& P, LAS unsigned char* lds, F&& ready, int wid, int lane) {
    const int wg = blockIdx.x, fr = lane & 15, fq = lane >> 4;
    constexpr int PER1 = 1024 / NL, PER2 = 512 / NL, NTICK = 8 * (PER1 + PER2);
    f32x4 p0[8];
    {
        const float* s0 = P.x + (size_t)((wg >> 4) * 4096 + (wg & 15) * 128 + wid * 16 + fr) * KS + fq * 8;
#pragma unroll
        for (int i = 0; i < 8; ++i) p0[i] = *(const f32x4*)(s0 + (i >> 1) * 32 + (i & 1) * 4);
    }
    const unsigned call = ready();
    const unsigned par = OFF_DATA + (call & 1u) * PAR_BYTES;
    LAS unsigned* cnt = (LAS unsigned*)(lds + LDS_MISC + 256);
    LAS unsigned* code = cnt + 32;
    char* myws = P.ws[lane & 7];
    char* wq0 = P.ws[fq] + fr * 16; char* wq1 = P.ws[4 + fq] + fr * 16;
    const int rank = P.rank;
    char* wsl = P.ws[rank];
    unsigned* tmo = (unsigned*)(wsl + OFF_TMO);
    const unsigned lo16 = lane * 16, tag = call & 0xffffu;
    {
        unsigned m = 0;
#pragma unroll
        for (int i = 0; i < 8; ++i) m = exp1_max4(m, __builtin_bit_cast(u32x4, p0[i]));
        m = wave_max8(m >> 23) + 1u;
        if (lane == 0) __hip_atomic_fetch_max(code, m, __ATOMIC_RELAXED, __HIP_MEMORY_SCOPE_WORKGROUP);
        WAIT_L(0); BAR();
        const int sh = EXP_TARGET + 1 - (int)__builtin_amdgcn_readfirstlane(code[0]);
        STAMP(21);
        const unsigned off = par + (unsigned)(rank * CHUNK + (wg & 15) * 32768 + wid * 2048);
        const __amdgpu_buffer_rsrc_t d0 = mk_rsrc(P.ws[(wg >> 4) * 4], WS_USED), d1 = mk_rsrc(P.ws[(wg >> 4) * 4 + 1], WS_USED),
                                     d2 = mk_rsrc(P.ws[(wg >> 4) * 4 + 2], WS_USED), d3 = mk_rsrc(P.ws[(wg >> 4) * 4 + 3], WS_USED);
#pragma unroll
        for (int kq = 0; kq < 4; ++kq) {
            const u32x4 h = cvt8(p0[2 * kq], p0[2 * kq + 1], sh);
            const unsigned o = off + (kq >> 1) * 16384 + (kq & 1) * 1024;
            WAIT_V(4);
            __builtin_amdgcn_raw_buffer_store_b128(h, d0, lo16, o, AUX_SYS);
            __builtin_amdgcn_raw_buffer_store_b128(h, d1, lo16, o, AUX_SYS);
            __builtin_amdgcn_raw_buffer_store_b128(h, d2, lo16, o, AUX_SYS);
            __builtin_amdgcn_raw_buffer_store_b128(h, d3, lo16, o, AUX_SYS);
        }
    }
    auto signal = [&](int st) {
        if (st < 0) return;
        unsigned old = 0;
        if (lane == 0) old = __hip_atomic_fetch_add(cnt + st, 1u, __ATOMIC_RELAXED, __HIP_MEMORY_SCOPE_WORKGROUP);
        old = __builtin_amdgcn_readfirstlane(old);
        if (old == (unsigned)(st == 0 ? PER2 + 7 : st < 8 ? PER1 - 1 : PER2 - 1)) {
            const unsigned cx = code[st] - 1u, cw = st < 8 ? code[16 + st] - 1u : 0u;
            if (lane < NRANK)
                __hip_atomic_store((unsigned*)(myws + OFF_INFLAG + ((st * 8 + rank) * NL + wg) * 4), (tag << 16) | (cx << 8) | cw, __ATOMIC_RELAXED, __HIP_MEMORY_SCOPE_SYSTEM);
            STAMPW(0, 1 + st);
        }
    };
    auto get_shift = [&](int st, bool isw) -> int {
        const int slot = isw ? 16 + st : st;
        unsigned c = __builtin_amdgcn_readfirstlane(__hip_atomic_load(code + slot, __ATOMIC_RELAXED, __HIP_MEMORY_SCOPE_WORKGROUP));
        if (c == 0u) {
            Deadline dl(tmo, call);
            unsigned mine = 0;
            if (lane == 0) mine = __hip_atomic_fetch_or(code + 24, 1u << slot, __ATOMIC_RELAXED, __HIP_MEMORY_SCOPE_WORKGROUP);
            mine = __builtin_amdgcn_readfirstlane(mine);
            if (((mine >> slot) & 1u) == 0u) {
                const unsigned* ps = (const unsigned*)(wsl + OFF_AMAX) + slot * 64 + lane;
                unsigned v;
                for (;;) {
                    v = __hip_atomic_load(ps, __ATOMIC_RELAXED, __HIP_MEMORY_SCOPE_AGENT);
                    if (__builtin_amdgcn_ballot_w64((v >> 16) != tag) == 0ull) break;
                    if (dl.expired()) break;
                    __builtin_amdgcn_s_sleep(2);
                }
                c = wave_max8(v & 0xffu) + 1u;
                if (lane == 0) __hip_atomic_store(code + slot, c, __ATOMIC_RELAXED, __HIP_MEMORY_SCOPE_WORKGROUP);
            } else {
                for (;;) {
                    c = __builtin_amdgcn_readfirstlane(__hip_atomic_load(code + slot, __ATOMIC_RELAXED, __HIP_MEMORY_SCOPE_WORKGROUP));
                    if (c != 0u) break;
                    __builtin_amdgcn_s_sleep(1);
                }
            }
        }
        return EXP_TARGET + 1 - (int)c;
    };
    f32x4 v[8];
    int st = -1; bool isw = false; unsigned off = 0;
    auto fetch = [&]() {
        unsigned tk = 0;
        if (lane == 0) tk = __hip_atomic_fetch_add(cnt + 16, 1u, __ATOMIC_RELAXED, __HIP_MEMORY_SCOPE_WORKGROUP);
        tk = __builtin_amdgcn_readfirstlane(tk) + PER2;
        if (tk >= (unsigned)NTICK) { st = -1; return; }
        int t;
        if (tk < (unsigned)(8 * PER1)) { st = tk / PER1; t = tk - st * PER1; } else { const int u = tk - 8 * PER1; st = u / PER2; t = u - st * PER2; st += 8; }
        const int g = st >> 3, j = st & 7;
        isw = t >= PER2;
        const int idx = (isw ? t - PER2 : t) * NL + wg;
        const float* s; size_t d1, d2;
        if (!isw) {
            const int lrow = g * 2048 + (idx >> 5) * 128 + ((idx >> 1) & 7) * 16, k0 = j * 128 + ((idx >> 4) & 1) * 64 + (idx & 1) * 32;
            s = P.x + (size_t)(lrow + fr) * KS + k0 + fq * 8; d1 = 4; d2 = (size_t)4096 * KS;
            off = par + (unsigned)((st * 8 + rank) * CHUNK + idx * 1024);
        } else {
            const int k0 = j * 128 + ((idx >> 5) & 1) * 64 + (idx & 7) * 8, c0 = (idx >> 6) * 256 + ((idx >> 3) & 3) * 64;
            s = P.w + (size_t)k0 * GN + fq * 2048 + c0 + 4 * (4 * (fr & 3) + (fr >> 2)); d1 = GN; d2 = 2 * GN;
            off = par + A_BYTES + (unsigned)((j * 8 + rank) * CHUNK + (idx >> 3) * 8192 + (idx & 7) * 256);
        }
#pragma unroll
        for (int i = 0; i < 4; ++i) v[i] = *(const f32x4*)(s + (i & 1) * d1 + (i >> 1) * d2);
        if (isw) {
#pragma unroll
            for (int i = 4; i < 8; ++i) v[i] = *(const f32x4*)(s + (i & 1) * d1 + (i >> 1) * d2);
        }
    };
    if (wid < LW) fetch();
    __builtin_amdgcn_s_waitcnt(0x0F70); asm volatile("" ::: "memory");
    signal(0);
    int pend1 = -1, pend2 = -1;
    while (st >= 0) {
        const int sh = get_shift(st, isw);
        __builtin_amdgcn_s_waitcnt(0x0F78); asm volatile("" ::: "memory");
        signal(pend2);
        pend2 = pend1; pend1 = st;
        u32x4 h[4]; char* ptr[8];
        if (!isw) {
            h[0] = cvt8(v[0], v[1], sh); h[1] = cvt8(v[2], v[3], sh); h[2] = h[0]; h[3] = h[1];
#pragma unroll
            for (int i = 0; i < 8; ++i) ptr[i] = P.ws[((i >> 1) & 1) * 4 + (i >> 2) * 2 + (i & 1)] + (off + lo16);
        } else {
#pragma unroll
            for (int c = 0; c < 4; ++c) {
                h[c] = cvt8((f32x4){v[0][c], v[1][c], v[2][c], v[3][c]}, (f32x4){v[4][c], v[5][c], v[6][c], v[7][c]}, sh);
                ptr[2 * c] = wq0 + (off + c * 2048); ptr[2 * c + 1] = wq1 + (off + c * 2048);
            }
        }
        fetch();
#pragma unroll
        for (int i = 0; i < 8; ++i) st16_sys(ptr[i], h[i >> 1]);
    }
    WAIT_V(0);
    signal(pend2); signal(pend1);
    STAMP(20);
    if (wid >= LW2) return;
    unsigned* q = (unsigned*)(wsl + OFF_LNKQ);
    const unsigned qbase = (call - 1u) * (unsigned)(NREC + NL * LW2);
    unsigned tk = 0;
    if (lane == 0) tk = __hip_atomic_fetch_add(q, 1u, __ATOMIC_RELAXED, __HIP_MEMORY_SCOPE_AGENT);
    int prec = -1; unsigned pnnz = 0;
    auto publish = [&]() {
        if (prec >= 0 && lane < NRANK && lane != rank)
            __hip_atomic_store((u64*)(myws + OFF_OUTFLAG + (unsigned)(rank * NREC + prec) * 64u), ((u64)call << 32) | pnnz, __ATOMIC_RELAXED, __HIP_MEMORY_SCOPE_SYSTEM);
        prec = -1;
    };
    for (;;) {
        const unsigned rec = (unsigned)__builtin_amdgcn_readfirstlane(tk) - qbase;
        if (rec >= (unsigned)NREC) break;
        if (lane == 0) tk = __hip_atomic_fetch_add(q, 1u, __ATOMIC_RELAXED, __HIP_MEMORY_SCOPE_AGENT);
        const u64* flag = (const u64*)(wsl + OFF_OUTFLAG + (unsigned)(rank * NREC + rec) * 64u);
        Deadline dl(tmo, call);
        unsigned nnz = 0; bool bad = false;
        for (;;) {
            const u64 f = __hip_atomic_load(flag, __ATOMIC_RELAXED, __HIP_MEMORY_SCOPE_AGENT);
            const unsigned fc = __builtin_amdgcn_readfirstlane((unsigned)(f >> 32));
            nnz = __builtin_amdgcn_readfirstlane((unsigned)f);
            WAIT_V(0);
            publish();
            if (fc >= call) break;
            if (dl.expired()) { bad = true; break; }
            __builtin_amdgcn_s_sleep(8);
        }
        if (bad || nnz > 4096u) continue;
        const unsigned bytes = 512u + nnz * 4u;
        const unsigned ro = par + A_BYTES + B_BYTES + (unsigned)(rank * NREC + rec) * REC_BYTES;
        const __amdgpu_buffer_rsrc_t rl = mk_rsrc(wsl + ro, bytes);
        u32x4 d[17];
#pragma unroll
        for (int i = 0; i < 17; ++i) if (i * 1024u < bytes) d[i] = __builtin_amdgcn_raw_buffer_load_b128(rl, lo16, i * 1024, AUX_SYS);
        __amdgpu_buffer_rsrc_t rp[7];
#pragma unroll
        for (int k = 0; k < 7; ++k) rp[k] = mk_rsrc(P.ws[(rank + 1 + k) & 7] + ro, bytes);
        WAIT_V(0);
#pragma unroll
        for (int i = 0; i < 17; ++i) if (i * 1024u < bytes) {
            WAIT_V(RELAY_DEPTH);
#pragma unroll
            for (int k = 0; k < 7; ++k) __builtin_amdgcn_raw_buffer_store_b128(d[i], rp[k], lo16, i * 1024, AUX_SYS);
        }
        prec = (int)rec; pnnz = nnz;
    }
    WAIT_V(0);
    publish();
}

__device__ __forceinline__ void wait_inputs(const Params& P, LAS unsigned char* lds, int st, int wsel, unsigned call, int lane) {
    const __amdgpu_buffer_rsrc_t r = mk_rsrc(P.ws[P.rank], WS_USED);
    const unsigned tag = call & 0xffffu;
    Deadline dl((unsigned*)(P.ws[P.rank] + OFF_TMO), call);
    u32x4 a;
    for (;;) {
        a = __builtin_amdgcn_raw_buffer_load_b128(r, lane * 16, OFF_INFLAG + st * 1024, AUX_SYS);
        unsigned df = 0;
#pragma unroll
        for (int i = 0; i < 4; ++i) df |= (a[i] >> 16) ^ tag;
        if (__builtin_amdgcn_ballot_w64(df != 0u) == 0ull) break;
        if (dl.expired()) break;
        __builtin_amdgcn_s_sleep(GSLEEP);
    }
    __builtin_amdgcn_fence(__ATOMIC_ACQUIRE, "");
    LAS int* tab = (LAS int*)(lds + LDS_MISC + 512);
    LAS int* ewt = tab + 64;
    if ((lane & 7) == (wsel >> 2)) {
        const int s0 = (st & 7) * 8 + (lane >> 3), e = wsel & 3;
        const unsigned fa = e == 0 ? a[0] : e == 1 ? a[1] : e == 2 ? a[2] : a[3];
        int wa = (int)(fa & 0xffu);
        if (st < 8) ewt[s0] = wa; else wa = ewt[s0];
        tab[s0] = (int)((fa >> 8) & 0xffu) + wa;
    }
    WAIT_V(0); WAIT_L(0);
}

__device__ __forceinline__ void gemm_role(const Params& P, LAS unsigned char* lds, unsigned call, int wid, int lane) {
    const int gi = blockIdx.x - NL, xcd = gi & 7, ix = gi >> 3;
    const int wr = wid >> 2, wc = wid & 3, fr = lane & 15, fq = lane >> 4;
    const unsigned par = OFF_DATA + (call & 1u) * PAR_BYTES;
    char* wsl = P.ws[P.rank];
    const int nt = (xcd & 1) * 4 + (ix & 3);
    const unsigned aoff = (unsigned)(wr * 8192 + lane * 16), boff = (unsigned)(16384 + wc * 8192 + lane * 16);
    const LAS int* tab = (const LAS int*)(lds + LDS_MISC + 512);
    const int wsel = (P.rank >> 2) * 16 + (xcd >> 1) * 4 + (ix >> 2);
    for (int job = 0; job < 2; ++job) {
        const int mt = job * 16 + (xcd >> 1) * 4 + (ix >> 2);
        const char* ga = wsl + par + (size_t)job * (64 * CHUNK) + (mt & 15) * 32768 + wid * 2048 + lane * 16;
        const char* gb = wsl + par + A_BYTES + nt * 65536 + wid * 4096 + lane * 16;
#define ISSUE(kt_, sb_) do { const char* _a = ga + (((kt_) >> 1) * CHUNK + ((kt_) & 1) * 16384); const char* _b = gb + (((kt_) >> 1) * CHUNK + ((kt_) & 1) * 32768); \
        LAS unsigned char* _la = lds + (sb_) + wid * 2048; LAS unsigned char* _lb = lds + (sb_) + 16384 + wid * 4096; \
        __builtin_amdgcn_global_load_lds((const unsigned*)_a, (LAS unsigned*)_la, 16, 0, 0); \
        __builtin_amdgcn_global_load_lds((const unsigned*)_a, (LAS unsigned*)_la, 16, 1024, 0); \
        __builtin_amdgcn_global_load_lds((const unsigned*)_b, (LAS unsigned*)_lb, 16, 0, 0); \
        __builtin_amdgcn_global_load_lds((const unsigned*)_b, (LAS unsigned*)_lb, 16, 1024, 0); \
        __builtin_amdgcn_global_load_lds((const unsigned*)_b, (LAS unsigned*)_lb, 16, 2048, 0); \
        __builtin_amdgcn_global_load_lds((const unsigned*)_b, (LAS unsigned*)_lb, 16, 3072, 0); } while (0)
#define LDF(A_, B_, sb_, kf_) do { _Pragma("unroll") for (int i = 0; i < 4; ++i) { \
        B_[i] = *(const LAS h16x8*)(lds + (sb_) + boff + i * 2048 + (kf_) * 1024); \
        A_[i] = *(const LAS h16x8*)(lds + (sb_) + aoff + i * 2048 + (kf_) * 1024); } } while (0)
#define MMA(A_, B_) do { __builtin_amdgcn_s_setprio(1); _Pragma("unroll") for (int m = 0; m < 4; ++m) _Pragma("unroll") for (int c = 0; c < 4; ++c) \
        acc[m][c] = __builtin_amdgcn_mfma_f32_16x16x32_f16(B_[c], A_[m], acc[m][c], 0, 0, 0); __builtin_amdgcn_s_setprio(0); } while (0)
        f32x4 acc[4][4];
#pragma unroll
        for (int m = 0; m < 4; ++m)
#pragma unroll
            for (int c = 0; c < 4; ++c) acc[m][c] = (f32x4){0.f, 0.f, 0.f, 0.f};
        h16x8 A0[4], B0[4], A1[4], B1[4];
        if (wid == 0) wait_inputs(P, lds, job * 8, wsel, call, lane);
        BAR();
        STAMP(1 + job * 8);
        ISSUE(0, 0); ISSUE(1, STAGE); ISSUE(2, 2 * STAGE);
        WAIT_V(12); BAR();
        LDF(A0, B0, 0u, 0);
        int tnext = tab[0];
        int tcur = __builtin_amdgcn_readfirstlane(tnext);
        unsigned sb = 0;
        for (int kt = 0; kt < 128; ++kt) {
            const unsigned sn = sb == 2 * STAGE ? 0u : sb + STAGE;
            __builtin_amdgcn_s_waitcnt(0xC07F);
            LDF(A1, B1, sb, 1);
            const int tn = __builtin_amdgcn_readfirstlane(tnext);
            if (tn != tcur) {
#pragma unroll
                for (int m = 0; m < 4; ++m)
#pragma unroll
                    for (int c = 0; c < 4; ++c)
#pragma unroll
                        for (int r = 0; r < 4; ++r) acc[m][c][r] = __builtin_ldexpf(acc[m][c][r], tcur - tn);
                tcur = tn;
            }
            MMA(A0, B0);
            if (((kt + 3) & 15) == 0 && kt + 3 < 128 && wid == 0) { STAMP(32 + job * 8 + ((kt + 3) >> 4)); wait_inputs(P, lds, job * 8 + ((kt + 3) >> 4), wsel, call, lane); STAMP(16 + job * 8 + ((kt + 3) >> 4)); }
            WAIT_V(6); __builtin_amdgcn_s_waitcnt(0xC07F); BAR();
            ISSUE((kt + 3) & 127, sb);
            LDF(A0, B0, sn, 0);
            tnext = tab[min(kt + 1, 127) >> 1];
            MMA(A1, B1);
            sb = sn;
        }
        WAIT_V(0); WAIT_L(0); BAR();
        STAMP(2 + job * 8);
        unsigned* tmo = (unsigned*)(wsl + OFF_TMO);
        const bool bad = (unsigned)__builtin_amdgcn_readfirstlane(__hip_atomic_load(tmo, __ATOMIC_RELAXED, __HIP_MEMORY_SCOPE_AGENT)) == call;
        LAS float* cv = (LAS float*)(lds + wid * 16384);
        unsigned mlo = 0, mhi = 0, off = 0;
        const int unsc = tcur - 2 * EXP_TARGET;
#pragma unroll
        for (int m = 0; m < 4; ++m)
#pragma unroll
            for (int c = 0; c < 4; ++c)
#pragma unroll
                for (int r = 0; r < 4; ++r) {
                    float v = __builtin_ldexpf(acc[m][c][r], unsc);
                    v = v <= 0.f ? 0.f : v;
                    if (bad) v = __builtin_nanf("");
                    acc[m][c][r] = v;
                    const bool nz = __float_as_uint(v) != 0u;
                    const u64 mk = __builtin_amdgcn_ballot_w64(nz);
                    const unsigned lo = (unsigned)mk, hi = (unsigned)(mk >> 32);
                    const unsigned pos = off + __builtin_amdgcn_mbcnt_hi(hi, __builtin_amdgcn_mbcnt_lo(lo, 0u));
                    if (nz) cv[pos] = v;
                    const int q = (m * 4 + c) * 4 + r;
                    mlo = lane == q ? lo : mlo; mhi = lane == q ? hi : mhi;
                    off += (unsigned)__builtin_popcountll(mk);
                }
        const int rec = (mt * 8 + nt) * 8 + wid;
        const __amdgpu_buffer_rsrc_t rl = mk_rsrc(wsl, WS_USED);
        const unsigned ro = par + A_BYTES + B_BYTES + (unsigned)(P.rank * NREC + rec) * REC_BYTES;
        __builtin_amdgcn_raw_buffer_store_b64((u32x2){mlo, mhi}, rl, lane * 8, ro, AUX_SYS);
        WAIT_L(0);
        for (unsigned i = 0; i * 256 < off; ++i) {
            const u32x4 d = *(const LAS u32x4*)((LAS unsigned char*)cv + i * 1024 + lane * 16);
            __builtin_amdgcn_raw_buffer_store_b128(d, rl, lane * 16, ro + 512 + i * 1024, AUX_SYS);
        }
        WAIT_V(0);
        if (lane == 0)
            __hip_atomic_store((u64*)(wsl + OFF_OUTFLAG + (unsigned)(P.rank * NREC + rec) * 64u), ((u64)call << 32) | off, __ATOMIC_RELAXED, __HIP_MEMORY_SCOPE_AGENT);
        store_dense<false>(P.out, (P.rank >> 2) * 4096 + mt * 128 + wr * 64, (P.rank & 3) * 2048 + nt * 256 + wc * 64, fr, fq, acc);
        WAIT_L(0); BAR();
        STAMP(3 + job * 8);
#undef ISSUE
#undef LDF
#undef MMA
    }
}

__device__ __forceinline__ void scale_role(const Params& P, LAS unsigned char* lds, unsigned call, int tid) {
    const int a = blockIdx.x - NL - NG, lane = tid & 63;
    LAS unsigned* red = (LAS unsigned*)(lds + LDS_MISC + 256);
    for (int u = 1; u < 24; ++u) {
        const bool isw = u < 16 && (u & 1);
        const int st = u < 16 ? u >> 1 : u - 8, slot = isw ? 16 + st : st;
        unsigned m = 0;
        if (isw) {
            u32x4 v[8];
#pragma unroll
            for (int i = 0; i < 8; ++i) v[i] = *(const u32x4*)(P.w + (size_t)(st * 128 + a * 2) * GN + i * 2048 + tid * 4);
#pragma unroll
            for (int i = 0; i < 8; ++i) m = exp1_max4(m, v[i]);
        } else {
            u32x4 v[4];
#pragma unroll
            for (int i = 0; i < 4; ++i) {
                const int rr = a * 64 + i * 16 + (tid >> 5), row = (rr >> 11) * 4096 + (st >> 3) * 2048 + (rr & 2047);
                v[i] = *(const u32x4*)(P.x + (size_t)row * KS + (st & 7) * 128 + (tid & 31) * 4);
            }
#pragma unroll
            for (int i = 0; i < 4; ++i) m = exp1_max4(m, v[i]);
        }
        m = wave_max8(m >> 23);
        if (lane == 0) __hip_atomic_fetch_max(red + slot, m, __ATOMIC_RELAXED, __HIP_MEMORY_SCOPE_WORKGROUP);
        WAIT_L(0); BAR();
        if (tid == 0)
            __hip_atomic_store((unsigned*)(P.ws[P.rank] + OFF_AMAX) + slot * 64 + a, ((call & 0xffffu) << 16) | red[slot], __ATOMIC_RELAXED, __HIP_MEMORY_SCOPE_AGENT);
        if (u == 1) STAMP(61);
        __builtin_amdgcn_s_sleep(SC_DELAY);
    }
}

__device__ __forceinline__ void expand_role(const Params& P, LAS unsigned char* lds, unsigned call, int wid, int lane) {
    const int fr = lane & 15, fq = lane >> 4;
    constexpr unsigned NJOB = 7 * NREC, E = NE * EW;
    if (wid >= EW) return;
    const unsigned par = OFF_DATA + (call & 1u) * PAR_BYTES;
    char* wsl = P.ws[P.rank];
    unsigned* tmo = (unsigned*)(wsl + OFF_TMO);
    LAS unsigned char* buf = lds + wid * 16384;
    unsigned* q = (unsigned*)(wsl + OFF_EXPQ);
    const unsigned qbase = (call - 1u) * (NJOB + E);
    unsigned tk = 0;
    if (lane == 0) tk = __hip_atomic_fetch_add(q, 1u, __ATOMIC_RELAXED, __HIP_MEMORY_SCOPE_AGENT);
    int it = 0;
    for (;; ++it) {
        const unsigned ku = (unsigned)__builtin_amdgcn_readfirstlane(tk) - qbase;
        if (ku >= NJOB) break;
        const int k = (int)ku;
        if (lane == 0) tk = __hip_atomic_fetch_add(q, 1u, __ATOMIC_RELAXED, __HIP_MEMORY_SCOPE_AGENT);
        const int rec = k / 7, src = (P.rank + 1 + (k - rec * 7)) & 7;
        const u64* flag = (const u64*)(wsl + OFF_OUTFLAG + (unsigned)(src * NREC + rec) * 64u);
        Deadline dl(tmo, call);
        unsigned nnz = 0; bool bad = false;
        for (;;) {
            const u64 f = __hip_atomic_load(flag, __ATOMIC_RELAXED, __HIP_MEMORY_SCOPE_SYSTEM);
            const unsigned fc = __builtin_amdgcn_readfirstlane((unsigned)(f >> 32));
            nnz = __builtin_amdgcn_readfirstlane((unsigned)f);
            if (fc >= call) break;
            if (dl.expired()) { bad = true; break; }
            if (it == 0) __builtin_amdgcn_s_sleep(100); else __builtin_amdgcn_s_sleep(ESLEEP);
        }
        if (nnz > 4096u) { nnz = 0; bad = true; }
        f32x4 v[4][4];
        if (!bad) {
            const char* rbase = wsl + par + A_BYTES + B_BYTES + (size_t)(src * NREC + rec) * REC_BYTES;
            const __amdgpu_buffer_rsrc_t rr = mk_rsrc(rbase, REC_BYTES);
            const u32x2 hdr = __builtin_amdgcn_raw_buffer_load_b64(rr, lane * 8, 0, AUX_SYS);
            const char* gsrc = rbase + 512 + lane * 16;
            const unsigned bytes = nnz * 4;
            for (unsigned i = 0; i < bytes; i += 1024)
                __builtin_amdgcn_global_load_lds((const unsigned*)(gsrc + i), (LAS unsigned*)(buf + i), 16, 0, AUX_SYS);
            WAIT_V(0);
            unsigned off = 0;
#pragma unroll
            for (int m = 0; m < 4; ++m)
#pragma unroll
                for (int c = 0; c < 4; ++c)
#pragma unroll
                    for (int r = 0; r < 4; ++r) {
                        const int qq = (m * 4 + c) * 4 + r;
                        const unsigned lo = __builtin_amdgcn_readlane(hdr[0], qq), hi = __builtin_amdgcn_readlane(hdr[1], qq);
                        const u64 mk = ((u64)hi << 32) | lo;
                        const unsigned pos = (off + __builtin_amdgcn_mbcnt_hi(hi, __builtin_amdgcn_mbcnt_lo(lo, 0u))) & 4095u;
                        const float t = *(const LAS float*)(buf + pos * 4);
                        v[m][c][r] = __builtin_amdgcn_inverse_ballot_w64(mk) ? t : 0.f;
                        off += (unsigned)__builtin_popcountll(mk);
                    }
        } else {
#pragma unroll
            for (int m = 0; m < 4; ++m)
#pragma unroll
                for (int c = 0; c < 4; ++c) { const float qn = __builtin_nanf(""); v[m][c] = (f32x4){qn, qn, qn, qn}; }
        }
        const int w8 = rec & 7, nt = (rec >> 3) & 7, mt = rec >> 6;
        const int row0 = (src >> 2) * 4096 + mt * 128 + (w8 >> 2) * 64, col0 = (src & 3) * 2048 + nt * 256 + (w8 & 3) * 64;
        if (ku < NJOB - WT_UNTIL) store_dense<true>(P.out, row0, col0, fr, fq, v); else
        store_dense<false>(P.out, row0, col0, fr, fq, v);
        WAIT_L(0);
        if (it < 60) STAMP(1 + it);
    }
}

__global__ __launch_bounds__(512, 2) void fused_kernel(Params P) {
    extern __shared__ __attribute__((aligned(16))) unsigned char shm[];
    LAS unsigned char* lds = (LAS unsigned char*)shm;
    const int tid = threadIdx.x, lane = tid & 63, wid = __builtin_amdgcn_readfirstlane(tid >> 6);
    LAS unsigned* misc = (LAS unsigned*)(lds + LDS_MISC);
    STAMP(0);
    unsigned ticket = 0, vz = 0;
    asm volatile("" : "+v"(vz));
    if (tid == 0) ticket = __hip_atomic_fetch_add((unsigned*)(P.ws[P.rank] + OFF_CTR + (blockIdx.x & 7) * 128 + vz), 1u, __ATOMIC_RELAXED, __HIP_MEMORY_SCOPE_AGENT);
    if (tid >= 64 && tid < 128) misc[tid] = 0u;
    auto ready = [&]() -> unsigned {
        asm volatile("" : "+v"(ticket));
        if (tid == 0) misc[0] = ticket / (unsigned)(GRID / 8) + 1u;
        WAIT_L(0); BAR();
        return __builtin_amdgcn_readfirstlane(misc[0]);
    };
    if (blockIdx.x < NL) link_role(P, lds, ready, wid, lane);
    else {
        const unsigned call = ready();
        if (blockIdx.x < NL + NG) gemm_role(P, lds, call, wid, lane);
        else {
            if (blockIdx.x < NL + NG + NA) scale_role(P, lds, call, tid);
            STAMP(62);
            expand_role(P, lds, call, wid, lane);
        }
    }
    STAMP(63);
}

}

extern "C" void kernel_launch(int rank, int nranks, ncclComm_t comm,
                              hipStream_t stream,
                              void* const* d_in, const int* in_sizes, int n_in,
                              void* d_out, int out_size,
                              const int* in_dtypes, int out_dtype,
                              void* const* d_out_all,
                              void* const* d_ws_all, size_t ws_bytes) {
    using namespace gar;
    if (nranks != NRANK || (unsigned)rank >= (unsigned)NRANK || ws_bytes < (size_t)WS_USED || n_in < 2 ||
        in_sizes[0] != GM * KS || in_sizes[1] != KS * GN || out_size != GM * GN) {
        fprintf(stderr, "kernel_launch: unexpected problem shape\n");
        return;
    }
    Params P;
    P.x = (const float*)d_in[0]; P.w = (const float*)d_in[1]; P.out = (float*)d_out; P.rank = rank;
    for (int p = 0; p < NRANK; ++p) P.ws[p] = (char*)d_ws_all[p];
    HIP_CHECK(hipFuncSetAttribute((const void*)fused_kernel, hipFuncAttributeMaxDynamicSharedMemorySize, LDS_BYTES));
    hipLaunchKernelGGL(fused_kernel, dim3(GRID), dim3(512), LDS_BYTES, stream, P);
    HIP_CHECK(hipGetLastError());
}
```

```cpp
#include <hip/hip_runtime.h>
#include <rccl/rccl.h>
#include <cstdio>

#define HIP_CHECK(expr) do { hipError_t _e = (expr); if (_e != hipSuccess) { \
    fprintf(stderr, "HIP error %s at %s:%d\n", hipGetErrorString(_e), __FILE__, __LINE__); \
    return; } } while (0)

namespace gar {

#define LAS __attribute__((address_space(3)))
typedef _Float16 h16x8 __attribute__((ext_vector_type(8)));
typedef float f32x4 __attribute__((ext_vector_type(4)));
typedef unsigned u32x4 __attribute__((ext_vector_type(4)));
typedef unsigned u32x2 __attribute__((ext_vector_type(2)));
typedef unsigned long long u64;

constexpr int NRANK = 8;
constexpr int GM = 8192, GN = 8192, KS = 1024;
#define PW 5
#define EW 4
#define GSLEEP 32
#define GSLEEP8 127
#define PULLU 1
#define TAILR 192
#define TAILP 4
#define ESLEEP 4
#define CV_DELAY 0
constexpr int NL = 32;
constexpr int NG = 128;
constexpr int NE = 96;
constexpr int GRID = NL + NG + NE;
constexpr int NREC = 2048;
constexpr unsigned REC_BYTES = 17408;
constexpr unsigned NITEM_IN = 64 + 7 * 128 + 8 * 64;

constexpr unsigned OFF_TMO = 0, OFF_EXPQ = 128, OFF_LNKQ = 256, OFF_CTR = 1024;
constexpr unsigned OFF_CVFLAG = 4096;
constexpr unsigned OFF_PULLED = 65536;
constexpr unsigned OFF_RECFLAG = 0x140000;
constexpr unsigned OFF_DBG = 2u << 20;
constexpr unsigned OFF_DATA = 4u << 20;
constexpr unsigned CHUNK = 512u << 10;
constexpr unsigned A_BYTES = 128 * CHUNK, B_BYTES = 64 * CHUNK, XO_BYTES = 16 * CHUNK, WO_BYTES = 32 * CHUNK, O_BYTES = NRANK * NREC * REC_BYTES;
constexpr unsigned REL_B = A_BYTES, REL_XO = REL_B + B_BYTES, REL_WO = REL_XO + XO_BYTES, REL_O = REL_WO + WO_BYTES;
constexpr unsigned PAR_BYTES = REL_O + O_BYTES;
constexpr unsigned WS_USED = OFF_DATA + 2 * PAR_BYTES;
constexpr unsigned STAGE = 49152, LDS_MISC = 3 * STAGE, LDS_BYTES = LDS_MISC + 2048;
constexpr u64 DEADLINE = 50000000ull;
constexpr int AUX_SYS = 17;
constexpr int EXP_TARGET = 142;

struct Params {
    const float* x; const float* w; float* out; char* ws[NRANK]; int rank;
};

#define WAIT_V(n) WAIT_V_(n)
#define WAIT_V_(n) asm volatile("s_waitcnt vmcnt(" #n ")" ::: "memory")
#define WAIT_L(n) asm volatile("s_waitcnt lgkmcnt(" #n ")" ::: "memory")
#define BAR() do { asm volatile("" ::: "memory"); __builtin_amdgcn_s_barrier(); asm volatile("" ::: "memory"); } while (0)

#define STAMPW(w, i) do {} while (0)
#define STAMP(i) do {} while (0)

__device__ __forceinline__ __amdgpu_buffer_rsrc_t mk_rsrc(const void* p, unsigned bytes) {
    return __builtin_amdgcn_make_buffer_rsrc((void*)p, 0, bytes, 0x00020000);
}
__device__ __forceinline__ void st16_sys(char* p, u32x4 v) {
    __builtin_amdgcn_global_store_b128((__attribute__((address_space(1))) u32x4*)p, v, "");
}
__device__ __forceinline__ u32x4 cvt8(f32x4 a, f32x4 b, int sh) {
    h16x8 h;
#pragma unroll
    for (int j = 0; j < 4; ++j) { h[j] = (_Float16)__builtin_ldexpf(a[j], sh); h[4 + j] = (_Float16)__builtin_ldexpf(b[j], sh); }
    return __builtin_bit_cast(u32x4, h);
}
struct Deadline {
    unsigned* tmo; unsigned call; u64 t0; unsigned polls;
    __device__ __forceinline__ Deadline(unsigned* tmo_, unsigned call_) : tmo(tmo_), call(call_), t0(__builtin_amdgcn_s_memrealtime()), polls(1) {}
    __device__ __forceinline__ bool expired() {
        if ((polls++ & 63u) == 0u &&
            (unsigned)__builtin_amdgcn_readfirstlane(__hip_atomic_load(tmo, __ATOMIC_RELAXED, __HIP_MEMORY_SCOPE_AGENT)) == call) return true;
        if (__builtin_amdgcn_s_memrealtime() - t0 > DEADLINE) {
            __hip_atomic_store(tmo, call, __ATOMIC_RELAXED, __HIP_MEMORY_SCOPE_AGENT);
            return true;
        }
        return false;
    }
};
__device__ __forceinline__ unsigned wave_max8(unsigned v) {
    unsigned r = 0;
#pragma unroll
    for (int b = 7; b >= 0; --b) { const unsigned c = r | (1u << b); r = __builtin_amdgcn_ballot_w64(v >= c) != 0ull ? c : r; }
    return r;
}
__device__ __forceinline__ unsigned exp1_max4(unsigned m, u32x4 v) {
#pragma unroll
    for (int i = 0; i < 4; ++i) m = max(m, (v[i] + 0x00800000u) & 0x7f800000u);
    return m;
}

__device__ __forceinline__ void store_dense(float* out, int row0, int col0, int fr, int fq, const f32x4 (&v)[4][4]) {
    float* p = out + (size_t)(row0 + fr) * GN + col0 + 4 * fq;
#pragma unroll
    for (int m = 0; m < 4; ++m)
#pragma unroll
        for (int r = 0; r < 4; ++r) {
            f32x4 o = {v[m][0][r], v[m][1][r], v[m][2][r], v[m][3][r]};
            st16_sys((char*)(p + (size_t)m * 16 * GN + r * 16), __builtin_bit_cast(u32x4, o));
        }
}

__device__ __forceinline__ unsigned block_code(LAS unsigned* word, unsigned m, int lane) {
    m = wave_max8(m >> 23);
    if (lane == 0) __hip_atomic_fetch_max(word, m, __ATOMIC_RELAXED, __HIP_MEMORY_SCOPE_WORKGROUP);
    WAIT_L(0); BAR();
    return __builtin_amdgcn_readfirstlane(*word);
}
__device__ __forceinline__ void load_xblock(const Params& P, f32x4 (&v)[16], int half, int st, int mt, int wid, int fr, int fq) {
    const float* s = P.x + (size_t)(half * 4096 + (st >> 3) * 2048 + mt * 128 + wid * 16 + fr) * KS + (st & 7) * 128 + fq * 8;
#pragma unroll
    for (int i = 0; i < 8; ++i) v[i] = *(const f32x4*)(s + (i >> 1) * 32 + (i & 1) * 4);
}

__device__ __forceinline__ void convert_role(const Params& P, LAS unsigned char* lds, unsigned call, int wid, int lane) {
    const int fr = lane & 15, fq = lane >> 4, rank = P.rank;
    const unsigned par = OFF_DATA + (call & 1u) * PAR_BYTES;
    LAS unsigned* red = (LAS unsigned*)(lds + LDS_MISC + 256);
    const __amdgpu_buffer_rsrc_t rl = mk_rsrc(P.ws[rank], WS_USED);
    int n = 0;
    for (int i = blockIdx.x - NL - NG; i < 23 * 32; i += NE, ++n) {
        const int u = (i >> 5) + 1, sub = i & 31;
        const bool isw = u < 16 && (u & 1);
        const int st = u < 16 ? u >> 1 : u - 8;
        f32x4 v[16];
        unsigned m = 0, blk; int dst;
        if (isw) {
            const int b = sub >> 3, nt = sub & 7;
            const float* s = P.w + (size_t)(st * 128 + (wid >> 2) * 64 + fq * 8) * GN + b * 2048 + nt * 256 + (wid & 3) * 64 + 4 * (4 * (fr & 3) + (fr >> 2));
#pragma unroll
            for (int e = 0; e < 16; ++e) v[e] = *(const f32x4*)(s + (size_t)((e >> 3) * 32 + (e & 7)) * GN);
#pragma unroll
            for (int e = 0; e < 16; ++e) m = exp1_max4(m, __builtin_bit_cast(u32x4, v[e]));
            m = block_code(red + n, m, lane);
            const int sh = EXP_TARGET - (int)m;
            const unsigned off = par + (b == (rank & 3) ? REL_B + (st * 8 + rank) * CHUNK : REL_WO + (b * 8 + st) * CHUNK) + nt * 65536 + wid * 8192;
#pragma unroll
            for (int c = 0; c < 4; ++c)
#pragma unroll
                for (int kf = 0; kf < 2; ++kf) {
                    const f32x4* q = v + 8 * kf;
                    const u32x4 h = cvt8((f32x4){q[0][c], q[1][c], q[2][c], q[3][c]}, (f32x4){q[4][c], q[5][c], q[6][c], q[7][c]}, sh);
                    __builtin_amdgcn_raw_buffer_store_b128(h, rl, lane * 16 + off + c * 2048 + kf * 1024, 0, AUX_SYS);
                }
            blk = 256 + st * 8 + nt; dst = lane < 2 ? b + 4 * lane : -1;
        } else {
            const int half = sub >> 4, mt = sub & 15;
            load_xblock(P, v, half, st, mt, wid, fr, fq);
#pragma unroll
            for (int e = 0; e < 8; ++e) m = exp1_max4(m, __builtin_bit_cast(u32x4, v[e]));
            m = block_code(red + n, m, lane);
            const int sh = EXP_TARGET - (int)m;
            const unsigned off = par + (half == (rank >> 2) ? (st * 8 + rank) * CHUNK : REL_XO + st * CHUNK) + mt * 32768 + wid * 2048;
#pragma unroll
            for (int kq = 0; kq < 4; ++kq)
                __builtin_amdgcn_raw_buffer_store_b128(cvt8(v[2 * kq], v[2 * kq + 1], sh), rl, lane * 16 + off + (kq >> 1) * 16384 + (kq & 1) * 1024, 0, AUX_SYS);
            blk = st * 16 + mt; dst = lane < 4 ? half * 4 + lane : -1;
        }
        WAIT_V(0); BAR();
        if (wid == 0 && dst >= 0)
            __hip_atomic_store((u64*)(P.ws[dst & 7] + OFF_CVFLAG) + blk * 8 + rank, ((u64)call << 32) | m, __ATOMIC_RELAXED, __HIP_MEMORY_SCOPE_SYSTEM);
        if (n == 0) STAMP(61);
        __builtin_amdgcn_s_sleep(CV_DELAY);
    }
}

template <class F>
__device__ __forceinline__ void link_role(const Params& P, LAS unsigned char* lds, F&& ready, int wid, int lane) {
    const int wg = blockIdx.x, fr = lane & 15, fq = lane >> 4, rank = P.rank;
    f32x4 p0[16];
    load_xblock(P, p0, wg >> 4, 0, wg & 15, wid, fr, fq);
    const unsigned call = ready();
    const unsigned par = OFF_DATA + (call & 1u) * PAR_BYTES;
    LAS unsigned* cnt = (LAS unsigned*)(lds + LDS_MISC + 256);
    char* wsl = P.ws[rank];
    unsigned* tmo = (unsigned*)(wsl + OFF_TMO);
    const unsigned lo16 = lane * 16;
    unsigned* q = (unsigned*)(wsl + OFF_LNKQ);
    constexpr unsigned NITEM = NITEM_IN + NREC + TAILR * (TAILP - 1);
    const unsigned qbase = (call - 1u) * (NITEM + NL * PW);
    {
        unsigned m = 0;
#pragma unroll
        for (int i = 0; i < 8; ++i) m = exp1_max4(m, __builtin_bit_cast(u32x4, p0[i]));
        m = block_code(cnt, m, lane);
        const int sh = EXP_TARGET - (int)m;
        STAMP(21);
        const unsigned off = par + (unsigned)(rank * CHUNK + (wg & 15) * 32768 + wid * 2048);
        const __amdgpu_buffer_rsrc_t d0 = mk_rsrc(P.ws[(wg >> 4) * 4], WS_USED), d1 = mk_rsrc(P.ws[(wg >> 4) * 4 + 1], WS_USED),
                                     d2 = mk_rsrc(P.ws[(wg >> 4) * 4 + 2], WS_USED), d3 = mk_rsrc(P.ws[(wg >> 4) * 4 + 3], WS_USED);
#pragma unroll
        for (int kq = 0; kq < 4; ++kq) {
            const u32x4 h = cvt8(p0[2 * kq], p0[2 * kq + 1], sh);
            const unsigned o = lo16 + off + (kq >> 1) * 16384 + (kq & 1) * 1024;
            WAIT_V(4);
            __builtin_amdgcn_raw_buffer_store_b128(h, d0, o, 0, AUX_SYS);
            __builtin_amdgcn_raw_buffer_store_b128(h, d1, o, 0, AUX_SYS);
            __builtin_amdgcn_raw_buffer_store_b128(h, d2, o, 0, AUX_SYS);
            __builtin_amdgcn_raw_buffer_store_b128(h, d3, o, 0, AUX_SYS);
        }
        WAIT_V(0);
        unsigned old = 0;
        if (lane == 0) old = __hip_atomic_fetch_add(cnt + 1, 1u, __ATOMIC_RELAXED, __HIP_MEMORY_SCOPE_WORKGROUP);
        if (__builtin_amdgcn_readfirstlane(old) == 7u && lane < 4)
            __hip_atomic_store((u64*)(P.ws[(wg >> 4) * 4 + lane] + OFF_CVFLAG) + (wg & 15) * 8 + rank, ((u64)call << 32) | m, __ATOMIC_RELAXED, __HIP_MEMORY_SCOPE_SYSTEM);
    }
    STAMP(20);
    if (wid >= PW) return;
    unsigned tk = 0;
    if (lane == 0) tk = __hip_atomic_fetch_add(q, 1u, __ATOMIC_RELAXED, __HIP_MEMORY_SCOPE_AGENT);
    const int lsrc = (rank + 1 + lane) & 7;
    int prev = -1;
    auto publish = [&]() {
        if (prev >= 0 && lane == 0) __hip_atomic_fetch_add((unsigned*)(wsl + OFF_PULLED + (unsigned)prev * 64u), 1u, __ATOMIC_RELAXED, __HIP_MEMORY_SCOPE_AGENT);
        prev = -1;
    };
    for (;;) {
        const unsigned it = (unsigned)__builtin_amdgcn_readfirstlane(tk) - qbase;
        if (it >= NITEM) break;
        if (lane == 0) tk = __hip_atomic_fetch_add(q, 1u, __ATOMIC_RELAXED, __HIP_MEMORY_SCOPE_AGENT);
        const bool isrec = it >= NITEM_IN;
        int id; unsigned foff, soff, doff, o0 = 0, ostep = 1024 * PULLU;
        if (isrec) {
            unsigned rec = it - NITEM_IN;
            if (rec >= NREC - TAILR) { const unsigned t = rec - (NREC - TAILR); rec = NREC - TAILR + t / TAILP; o0 = (t % TAILP) * ostep; ostep *= TAILP; }
            id = 320 + (int)rec; foff = OFF_RECFLAG + (unsigned)(lsrc * NREC + rec) * 8u;
            soff = doff = par + REL_O + (unsigned)(lsrc * NREC + rec) * REC_BYTES;
        } else {
            const int u = (int)(it >> 6) + 1, sub = it & 63;
            const bool isw = u < 16 && (u & 1);
            const int st = u < 16 ? u >> 1 : u - 8;
            id = isw ? 256 + st * 8 + (sub >> 3) : st * 16 + (sub >> 2);
            foff = OFF_CVFLAG + (unsigned)(id * 8 + lsrc) * 8u;
            doff = par + (isw ? REL_B : 0u) + (unsigned)(st * 8 + lsrc) * CHUNK + sub * 8192;
            const bool same = isw ? (lsrc & 3) == (rank & 3) : (lsrc >> 2) == (rank >> 2);
            soff = same ? doff : par + (isw ? REL_WO + (unsigned)((rank & 3) * 8 + st) * CHUNK : REL_XO + (unsigned)st * CHUNK) + sub * 8192;
            if ((it & 127u) == 0u) STAMPW(7, 1 + (it >> 7));
        }
        Deadline dl(tmo, call);
        u64 f = 0; bool bad = false;
        for (;;) {
            if (lane < 7) f = __hip_atomic_load((const u64*)(wsl + foff), __ATOMIC_RELAXED, __HIP_MEMORY_SCOPE_SYSTEM);
            const u64 anylate = __builtin_amdgcn_ballot_w64(lane < 7 && (unsigned)(f >> 32) < call);
            WAIT_V(0);
            publish();
            if (anylate == 0ull) break;
            if (dl.expired()) { bad = true; break; }
            __builtin_amdgcn_s_sleep(8);
        }
        if (bad || __builtin_amdgcn_ballot_w64(lane < 7 && (unsigned)f > 4096u) != 0ull) continue;
        const unsigned lbytes = isrec ? 512u + (unsigned)f * 4u : 8192u;
        __amdgpu_buffer_rsrc_t rp[7], rl[7];
        unsigned maxb = 0;
#pragma unroll
        for (int k = 0; k < 7; ++k) {
            const unsigned bytes = __builtin_amdgcn_readlane(lbytes, k);
            rp[k] = mk_rsrc(P.ws[(rank + 1 + k) & 7] + (unsigned)__builtin_amdgcn_readlane(soff, k), bytes);
            rl[k] = mk_rsrc(wsl + (unsigned)__builtin_amdgcn_readlane(doff, k), bytes);
            maxb = max(maxb, bytes);
        }
        for (unsigned o = o0; o < maxb; o += ostep) {
            u32x4 d[PULLU][7];
#pragma unroll
            for (int u = 0; u < PULLU; ++u)
#pragma unroll
                for (int k = 0; k < 7; ++k) d[u][k] = __builtin_amdgcn_raw_buffer_load_b128(rp[k], lo16 + o + u * 1024, 0, AUX_SYS);
#pragma unroll
            for (int u = 0; u < PULLU; ++u)
#pragma unroll
                for (int k = 0; k < 7; ++k) __builtin_amdgcn_raw_buffer_store_b128(d[u][k], rl[k], lo16 + o + u * 1024, 0, AUX_SYS);
        }
        prev = id;
    }
    WAIT_V(0);
    publish();
}

__device__ __forceinline__ void wait_inputs(const Params& P, LAS unsigned char* lds, int st, int mt, int nt, unsigned call, int lane) {
    char* wsl = P.ws[P.rank];
    const int bx = st * 16 + mt, bw = 256 + (st & 7) * 8 + nt;
    const unsigned off = lane < 16 ? OFF_CVFLAG + (unsigned)((lane < 8 ? bx : bw) * 8 + (lane & 7)) * 8u : OFF_PULLED + (unsigned)(lane == 16 ? bx : bw) * 64u;
    const unsigned need = lane == 16 ? (st == 0 ? 0u : 4u * call) : 8u * call;
    Deadline dl((unsigned*)(wsl + OFF_TMO), call);
    u64 f = 0;
    for (;;) {
        if (lane < 18) f = __hip_atomic_load((const u64*)(wsl + off), __ATOMIC_RELAXED, __HIP_MEMORY_SCOPE_SYSTEM);
        const bool late = lane < 16 ? (unsigned)(f >> 32) < call : lane < 18 && (unsigned)f < need;
        if (__builtin_amdgcn_ballot_w64(late) == 0ull) break;
        if (dl.expired()) break;
        if (st == 8) __builtin_amdgcn_s_sleep(GSLEEP8); else __builtin_amdgcn_s_sleep(GSLEEP);
    }
    __builtin_amdgcn_fence(__ATOMIC_ACQUIRE, "");
    LAS int* tab = (LAS int*)(lds + LDS_MISC + 512);
    const int cw = __shfl((int)((unsigned)f & 0xffu), (lane & 7) + 8);
    if (lane < 8) tab[(st & 7) * 8 + lane] = (int)((unsigned)f & 0xffu) + cw;
    WAIT_V(0); WAIT_L(0);
}

__device__ __forceinline__ void gemm_role(const Params& P, LAS unsigned char* lds, unsigned call, int wid, int lane) {
    const int gi = blockIdx.x - NL, xcd = gi & 7, ix = (gi & 127) >> 3;
    const int wr = wid >> 2, wc = wid & 3, fr = lane & 15, fq = lane >> 4;
    const unsigned par = OFF_DATA + (call & 1u) * PAR_BYTES;
    char* wsl = P.ws[P.rank];
    const int nt = (xcd & 1) * 4 + (ix & 3);
    const unsigned aoff = (unsigned)(wr * 8192 + lane * 16), boff = (unsigned)(16384 + wc * 8192 + lane * 16);
    const LAS int* tab = (const LAS int*)(lds + LDS_MISC + 512);
    const int ntile = gi < 32 ? 2 : 1;
    for (int ti = 0; ti < ntile; ++ti) {
        const int mt = gi >= NG ? 16 + (xcd >> 1) * 3 + (ix >> 2) : ti == 0 ? (xcd >> 1) * 4 + (ix >> 2) : 28 + (xcd >> 1);
        const int job = mt >> 4;
        const char* ga = wsl + par + (size_t)job * (64 * CHUNK) + (mt & 15) * 32768 + wid * 2048 + lane * 16;
        const char* gb = wsl + par + REL_B + nt * 65536 + wid * 4096 + lane * 16;
#define ISSUE(kt_, sb_) do { const char* _a = ga + (((kt_) >> 1) * CHUNK + ((kt_) & 1) * 16384); const char* _b = gb + (((kt_) >> 1) * CHUNK + ((kt_) & 1) * 32768); \
        LAS unsigned char* _la = lds + (sb_) + wid * 2048; LAS unsigned char* _lb = lds + (sb_) + 16384 + wid * 4096; \
        __builtin_amdgcn_global_load_lds((const unsigned*)_a, (LAS unsigned*)_la, 16, 0, 0); \
        __builtin_amdgcn_global_load_lds((const unsigned*)_a, (LAS unsigned*)_la, 16, 1024, 0); \
        __builtin_amdgcn_global_load_lds((const unsigned*)_b, (LAS unsigned*)_lb, 16, 0, 0); \
        __builtin_amdgcn_global_load_lds((const unsigned*)_b, (LAS unsigned*)_lb, 16, 1024, 0); \
        __builtin_amdgcn_global_load_lds((const unsigned*)_b, (LAS unsigned*)_lb, 16, 2048, 0); \
        __builtin_amdgcn_global_load_lds((const unsigned*)_b, (LAS unsigned*)_lb, 16, 3072, 0); } while (0)
#define LDF(A_, B_, sb_, kf_) do { _Pragma("unroll") for (int i = 0; i < 4; ++i) { \
        B_[i] = *(const LAS h16x8*)(lds + (sb_) + boff + i * 2048 + (kf_) * 1024); \
        A_[i] = *(const LAS h16x8*)(lds + (sb_) + aoff + i * 2048 + (kf_) * 1024); } } while (0)
#define MMA(A_, B_) do { __builtin_amdgcn_s_setprio(1); _Pragma("unroll") for (int m = 0; m < 4; ++m) _Pragma("unroll") for (int c = 0; c < 4; ++c) \
        acc[m][c] = __builtin_amdgcn_mfma_f32_16x16x32_f16(B_[c], A_[m], acc[m][c], 0, 0, 0); __builtin_amdgcn_s_setprio(0); } while (0)
        f32x4 acc[4][4];
#pragma unroll
        for (int m = 0; m < 4; ++m)
#pragma unroll
            for (int c = 0; c < 4; ++c) acc[m][c] = (f32x4){0.f, 0.f, 0.f, 0.f};
        h16x8 A0[4], B0[4], A1[4], B1[4];
        if (wid == 0) wait_inputs(P, lds, job * 8, mt & 15, nt, call, lane);
        BAR();
        STAMP(1 + job * 8);
        ISSUE(0, 0); ISSUE(1, STAGE); ISSUE(2, 2 * STAGE);
        WAIT_V(12); BAR();
        LDF(A0, B0, 0u, 0);
        int tnext = tab[0];
        int tcur = __builtin_amdgcn_readfirstlane(tnext);
        unsigned sb = 0;
        for (int kt = 0; kt < 128; ++kt) {
            const unsigned sn = sb == 2 * STAGE ? 0u : sb + STAGE;
            __builtin_amdgcn_s_waitcnt(0xC07F);
            LDF(A1, B1, sb, 1);
            const int tn = __builtin_amdgcn_readfirstlane(tnext);
            if (tn != tcur) {
#pragma unroll
                for (int m = 0; m < 4; ++m)
#pragma unroll
                    for (int c = 0; c < 4; ++c)
#pragma unroll
                        for (int r = 0; r < 4; ++r) acc[m][c][r] = __builtin_ldexpf(acc[m][c][r], tcur - tn);
                tcur = tn;
            }
            MMA(A0, B0);
            if (((kt + 3) & 15) == 0 && kt + 3 < 128 && wid == 0) { STAMP(32 + job * 8 + ((kt + 3) >> 4)); wait_inputs(P, lds, job * 8 + ((kt + 3) >> 4), mt & 15, nt, call, lane); STAMP(16 + job * 8 + ((kt + 3) >> 4)); }
            WAIT_V(6); __builtin_amdgcn_s_waitcnt(0xC07F); BAR();
            ISSUE((kt + 3) & 127, sb);
            LDF(A0, B0, sn, 0);
            tnext = tab[min(kt + 1, 127) >> 1];
            MMA(A1, B1);
            sb = sn;
        }
        WAIT_V(0); WAIT_L(0); BAR();
        STAMP(2 + job * 8);
        unsigned* tmo = (unsigned*)(wsl + OFF_TMO);
        const bool bad = (unsigned)__builtin_amdgcn_readfirstlane(__hip_atomic_load(tmo, __ATOMIC_RELAXED, __HIP_MEMORY_SCOPE_AGENT)) == call;
        LAS float* cv = (LAS float*)(lds + wid * 16384);
        unsigned mlo = 0, mhi = 0, off = 0;
        const int unsc = tcur - 2 * EXP_TARGET;
#pragma unroll
        for (int m = 0; m < 4; ++m)
#pragma unroll
            for (int c = 0; c < 4; ++c)
#pragma unroll
                for (int r = 0; r < 4; ++r) {
                    float v = __builtin_ldexpf(acc[m][c][r], unsc);
                    v = v <= 0.f ? 0.f : v;
                    if (bad) v = __builtin_nanf("");
                    acc[m][c][r] = v;
                    const bool nz = __float_as_uint(v) != 0u;
                    const u64 mk = __builtin_amdgcn_ballot_w64(nz);
                    const unsigned lo = (unsigned)mk, hi = (unsigned)(mk >> 32);
                    const unsigned pos = off + __builtin_amdgcn_mbcnt_hi(hi, __builtin_amdgcn_mbcnt_lo(lo, 0u));
                    if (nz) cv[pos] = v;
                    const int q = (m * 4 + c) * 4 + r;
                    mlo = lane == q ? lo : mlo; mhi = lane == q ? hi : mhi;
                    off += (unsigned)__builtin_popcountll(mk);
                }
        const int rec = (mt * 8 + nt) * 8 + wid;
        const __amdgpu_buffer_rsrc_t rl = mk_rsrc(wsl, WS_USED);
        const unsigned ro = par + REL_O + (unsigned)(P.rank * NREC + rec) * REC_BYTES;
        __builtin_amdgcn_raw_buffer_store_b64((u32x2){mlo, mhi}, rl, lane * 8, ro, AUX_SYS);
        WAIT_L(0);
        for (unsigned i = 0; i * 256 < off; ++i) {
            const u32x4 d = *(const LAS u32x4*)((LAS unsigned char*)cv + i * 1024 + lane * 16);
            __builtin_amdgcn_raw_buffer_store_b128(d, rl, lane * 16 + ro + 512 + i * 1024, 0, AUX_SYS);
        }
        LAS unsigned* nz8 = (LAS unsigned*)(lds + LDS_MISC + 1024);
        if (lane == 0) nz8[wid] = off;
        WAIT_V(0); WAIT_L(0); BAR();
        if (wid == 0 && (lane >> 3) != P.rank)
            __hip_atomic_store((u64*)(P.ws[lane >> 3] + OFF_RECFLAG) + (unsigned)(P.rank * NREC + rec + (lane & 7)), ((u64)call << 32) | nz8[lane & 7], __ATOMIC_RELAXED, __HIP_MEMORY_SCOPE_SYSTEM);
        store_dense(P.out, (P.rank >> 2) * 4096 + mt * 128 + wr * 64, (P.rank & 3) * 2048 + nt * 256 + wc * 64, fr, fq, acc);
        WAIT_L(0); BAR();
        STAMP(3 + job * 8);
#undef ISSUE
#undef LDF
#undef MMA
    }
}

__device__ __forceinline__ void expand_role(const Params& P, LAS unsigned char* lds, unsigned call, int wid, int lane) {
    const int fr = lane & 15, fq = lane >> 4;
    constexpr unsigned NJOB = 7 * NREC, E = NE * EW;
    if (wid >= EW) return;
    const unsigned par = OFF_DATA + (call & 1u) * PAR_BYTES;
    char* wsl = P.ws[P.rank];
    unsigned* tmo = (unsigned*)(wsl + OFF_TMO);
    LAS unsigned char* buf = lds + wid * 16384;
    unsigned* q = (unsigned*)(wsl + OFF_EXPQ);
    const unsigned qbase = (call - 1u) * (NJOB + E);
    unsigned tk = 0;
    if (lane == 0) tk = __hip_atomic_fetch_add(q, 1u, __ATOMIC_RELAXED, __HIP_MEMORY_SCOPE_AGENT);
    int it = 0;
    unsigned vz = 0;
    asm volatile("" : "+v"(vz));
    for (;; ++it) {
        const unsigned ku = (unsigned)__builtin_amdgcn_readfirstlane(tk) - qbase;
        if (ku >= NJOB) break;
        const int k = (int)ku;
        const int rec = k / 7, src = (P.rank + 1 + (k - rec * 7)) & 7;
        const u64* flag = (const u64*)(wsl + (lane == 0 ? OFF_PULLED + (unsigned)(320 + rec) * 64u : OFF_RECFLAG + (unsigned)(src * NREC + rec) * 8u));
        const unsigned need = rec >= NREC - TAILR ? TAILP * call : call;
        Deadline dl(tmo, call);
        unsigned nnz = 0; bool bad = false;
        for (;;) {
            u64 f = 0;
            if (lane < 2) f = __hip_atomic_load(flag, __ATOMIC_RELAXED, __HIP_MEMORY_SCOPE_SYSTEM);
            nnz = __builtin_amdgcn_readlane((unsigned)f, 1);
            if ((unsigned)__builtin_amdgcn_readlane((unsigned)f, 0) >= need && (unsigned)__builtin_amdgcn_readlane((unsigned)(f >> 32), 1) >= call) break;
            if (dl.expired()) { bad = true; break; }
            if (it == 0) __builtin_amdgcn_s_sleep(100); else __builtin_amdgcn_s_sleep(ESLEEP);
        }
        if (nnz > 4096u) { nnz = 0; bad = true; }
        f32x4 v[4][4];
        if (!bad) {
            const char* rbase = wsl + par + REL_O + (size_t)(src * NREC + rec) * REC_BYTES;
            const __amdgpu_buffer_rsrc_t rr = mk_rsrc(rbase, REC_BYTES);
            const u32x2 hdr = __builtin_amdgcn_raw_buffer_load_b64(rr, lane * 8, 0, AUX_SYS);
            const char* gsrc = rbase + 512 + lane * 16;
            const unsigned bytes = nnz * 4;
            for (unsigned i = 0; i < bytes; i += 1024)
                __builtin_amdgcn_global_load_lds((const unsigned*)(gsrc + i), (LAS unsigned*)(buf + i), 16, 0, AUX_SYS);
            WAIT_V(0);
            unsigned off = 0;
#pragma unroll
            for (int m = 0; m < 4; ++m)
#pragma unroll
                for (int c = 0; c < 4; ++c)
#pragma unroll
                    for (int r = 0; r < 4; ++r) {
                        const int qq = (m * 4 + c) * 4 + r;
                        const unsigned lo = __builtin_amdgcn_readlane(hdr[0], qq), hi = __builtin_amdgcn_readlane(hdr[1], qq);
                        const u64 mk = ((u64)hi << 32) | lo;
                        const unsigned pos = (off + __builtin_amdgcn_mbcnt_hi(hi, __builtin_amdgcn_mbcnt_lo(lo, 0u))) & 4095u;
                        const float t = *(const LAS float*)(buf + pos * 4);
                        v[m][c][r] = __builtin_amdgcn_inverse_ballot_w64(mk) ? t : 0.f;
                        off += (unsigned)__builtin_popcountll(mk);
                    }
        } else {
#pragma unroll
            for (int m = 0; m < 4; ++m)
#pragma unroll
                for (int c = 0; c < 4; ++c) { const float qn = __builtin_nanf(""); v[m][c] = (f32x4){qn, qn, qn, qn}; }
        }
        if (lane == 0) tk = __hip_atomic_fetch_add(q + vz, 1u, __ATOMIC_RELAXED, __HIP_MEMORY_SCOPE_AGENT);
        const int w8 = rec & 7, nt = (rec >> 3) & 7, mt = rec >> 6;
        const int row0 = (src >> 2) * 4096 + mt * 128 + (w8 >> 2) * 64, col0 = (src & 3) * 2048 + nt * 256 + (w8 & 3) * 64;
        store_dense(P.out, row0, col0, fr, fq, v);
        WAIT_L(0);
        if (it < 60) STAMPW(4 + wid, 1 + it);
    }
}

__global__ __launch_bounds__(512, 2) void fused_kernel(Params P) {
    extern __shared__ __attribute__((aligned(16))) unsigned char shm[];
    LAS unsigned char* lds = (LAS unsigned char*)shm;
    const int tid = threadIdx.x, lane = tid & 63, wid = __builtin_amdgcn_readfirstlane(tid >> 6);
    LAS unsigned* misc = (LAS unsigned*)(lds + LDS_MISC);
    STAMP(0);
    unsigned ticket = 0, vz = 0;
    asm volatile("" : "+v"(vz));
    if (tid == 0) ticket = __hip_atomic_fetch_add((unsigned*)(P.ws[P.rank] + OFF_CTR + (blockIdx.x & 7) * 128 + vz), 1u, __ATOMIC_RELAXED, __HIP_MEMORY_SCOPE_AGENT);
    if (tid >= 64 && tid < 128) misc[tid] = 0u;
    auto ready = [&]() -> unsigned {
        asm volatile("" : "+v"(ticket));
        if (tid == 0) misc[0] = ticket / (unsigned)(GRID / 8) + 1u;
        WAIT_L(0); BAR();
        return __builtin_amdgcn_readfirstlane(misc[0]);
    };
    if (blockIdx.x < NL) link_role(P, lds, ready, wid, lane);
    else {
        const unsigned call = ready();
        if (blockIdx.x >= NL + NG) convert_role(P, lds, call, wid, lane);
        STAMP(62);
        gemm_role(P, lds, call, wid, lane);
        if (blockIdx.x >= NL + 32 && blockIdx.x < NL + NG)
        expand_role(P, lds, call, wid, lane);
    }
    STAMP(63);
}

}

extern "C" void kernel_launch(int rank, int nranks, ncclComm_t comm,
                              hipStream_t stream,
                              void* const* d_in, const int* in_sizes, int n_in,
                              void* d_out, int out_size,
                              const int* in_dtypes, int out_dtype,
                              void* const* d_out_all,
                              void* const* d_ws_all, size_t ws_bytes) {
    using namespace gar;
    if (nranks != NRANK || (unsigned)rank >= (unsigned)NRANK || ws_bytes < (size_t)WS_USED || n_in < 2 ||
        in_sizes[0] != GM * KS || in_sizes[1] != KS * GN || out_size != GM * GN) {
        fprintf(stderr, "kernel_launch: unexpected problem shape\n");
        return;
    }
    Params P;
    P.x = (const float*)d_in[0]; P.w = (const float*)d_in[1]; P.out = (float*)d_out; P.rank = rank;
    for (int p = 0; p < NRANK; ++p) P.ws[p] = (char*)d_ws_all[p];
    HIP_CHECK(hipFuncSetAttribute((const void*)fused_kernel, hipFuncAttributeMaxDynamicSharedMemorySize, LDS_BYTES));
    hipLaunchKernelGGL(fused_kernel, dim3(GRID), dim3(512), LDS_BYTES, stream, P);
    HIP_CHECK(hipGetLastError());
}
```

```cpp
#include <hip/hip_runtime.h>
#include <rccl/rccl.h>
#include <cstdio>

#define HIP_CHECK(expr) do { hipError_t _e = (expr); if (_e != hipSuccess) { \
    fprintf(stderr, "HIP error %s at %s:%d\n", hipGetErrorString(_e), __FILE__, __LINE__); \
    return; } } while (0)

namespace gar {

#define LAS __attribute__((address_space(3)))
typedef _Float16 h16x8 __attribute__((ext_vector_type(8)));
typedef float f32x4 __attribute__((ext_vector_type(4)));
typedef unsigned u32x4 __attribute__((ext_vector_type(4)));
typedef unsigned u32x2 __attribute__((ext_vector_type(2)));
typedef unsigned long long u64;

constexpr int NRANK = 8;
constexpr int GM = 8192, GN = 8192, KS = 1024;
#define PW 4
#define EW 4
#define GSLEEP 32
#define GSLEEP8 127
#define PULLU 1
#define TAILR 192
#define TAILP 4
#define PSLEEP 8
#define CV_START 64
#define DIRR 32
#define DIRG 32
#define DIRS 31
#define ESLEEP 4
#define CV_DELAY 0
constexpr int NL = 32;
constexpr int NG = 128;
constexpr int NE = 96;
constexpr int GRID = NL + NG + NE;
constexpr int NREC = 2048;
constexpr int NSTG = NREC - DIRR;
constexpr unsigned REC_BYTES = 17408;
constexpr unsigned NITEM_IN = 24 * 64;

constexpr unsigned OFF_TMO = 0, OFF_EXPQ = 128, OFF_LNKQ = 256, OFF_CTR = 1024;
constexpr unsigned OFF_CVW = 4096;
constexpr unsigned OFF_CVX = 0x180000;
constexpr unsigned OFF_PULLED = 65536;
constexpr unsigned OFF_RECFLAG = 0x140000;
constexpr unsigned OFF_DBG = 2u << 20;
constexpr unsigned OFF_DATA = 4u << 20;
constexpr unsigned CHUNK = 512u << 10;
constexpr unsigned A_BYTES = 128 * CHUNK, B_BYTES = 64 * CHUNK, XO_BYTES = 16 * CHUNK, WO_BYTES = 32 * CHUNK, O_BYTES = NRANK * NREC * REC_BYTES;
constexpr unsigned REL_B = A_BYTES, REL_XO = REL_B + B_BYTES, REL_WO = REL_XO + XO_BYTES, REL_O = REL_WO + WO_BYTES;
constexpr unsigned PAR_BYTES = REL_O + O_BYTES;
constexpr unsigned WS_USED = OFF_DATA + 2 * PAR_BYTES;
constexpr unsigned STAGE = 49152, LDS_MISC = 3 * STAGE, LDS_BYTES = LDS_MISC + 2304;
constexpr u64 DEADLINE = 50000000ull;
constexpr int AUX_SYS = 17;
constexpr int EXP_TARGET = 142;

struct Params {
    const float* x; const float* w; float* out; char* ws[NRANK]; int rank;
};

#define WAIT_V(n) WAIT_V_(n)
#define WAIT_V_(n) asm volatile("s_waitcnt vmcnt(" #n ")" ::: "memory")
#define WAIT_L(n) asm volatile("s_waitcnt lgkmcnt(" #n ")" ::: "memory")
#define BAR() do { asm volatile("" ::: "memory"); __builtin_amdgcn_s_barrier(); asm volatile("" ::: "memory"); } while (0)

#define STAMPW(w, i) do {} while (0)
#define STAMP(i) do {} while (0)

__device__ __forceinline__ __amdgpu_buffer_rsrc_t mk_rsrc(const void* p, unsigned bytes) {
    return __builtin_amdgcn_make_buffer_rsrc((void*)p, 0, bytes, 0x00020000);
}
__device__ __forceinline__ void st16_sys(char* p, u32x4 v) {
    __builtin_amdgcn_global_store_b128((__attribute__((address_space(1))) u32x4*)p, v, "");
}
__device__ __forceinline__ u32x4 cvt8(f32x4 a, f32x4 b, int sh) {
    h16x8 h;
#pragma unroll
    for (int j = 0; j < 4; ++j) { h[j] = (_Float16)__builtin_ldexpf(a[j], sh); h[4 + j] = (_Float16)__builtin_ldexpf(b[j], sh); }
    return __builtin_bit_cast(u32x4, h);
}
struct Deadline {
    unsigned* tmo; unsigned call; u64 t0; unsigned polls;
    __device__ __forceinline__ Deadline(unsigned* tmo_, unsigned call_) : tmo(tmo_), call(call_), t0(__builtin_amdgcn_s_memrealtime()), polls(1) {}
    __device__ __forceinline__ bool expired() {
        if ((polls++ & 63u) == 0u &&
            (unsigned)__builtin_amdgcn_readfirstlane(__hip_atomic_load(tmo, __ATOMIC_RELAXED, __HIP_MEMORY_SCOPE_AGENT)) == call) return true;
        if (__builtin_amdgcn_s_memrealtime() - t0 > DEADLINE) {
            __hip_atomic_store(tmo, call, __ATOMIC_RELAXED, __HIP_MEMORY_SCOPE_AGENT);
            return true;
        }
        return false;
    }
};
__device__ __forceinline__ unsigned wave_max8(unsigned v) {
    unsigned r = 0;
#pragma unroll
    for (int b = 7; b >= 0; --b) { const unsigned c = r | (1u << b); r = __builtin_amdgcn_ballot_w64(v >= c) != 0ull ? c : r; }
    return r;
}
__device__ __forceinline__ unsigned exp1_max4(unsigned m, u32x4 v) {
#pragma unroll
    for (int i = 0; i < 4; ++i) m = max(m, (v[i] + 0x00800000u) & 0x7f800000u);
    return m;
}

__device__ __forceinline__ void store_dense(float* out, int row0, int col0, int fr, int fq, const f32x4 (&v)[4][4]) {
    float* p = out + (size_t)(row0 + fr) * GN + col0 + 4 * fq;
#pragma unroll
    for (int m = 0; m < 4; ++m)
#pragma unroll
        for (int r = 0; r < 4; ++r) {
            f32x4 o = {v[m][0][r], v[m][1][r], v[m][2][r], v[m][3][r]};
            st16_sys((char*)(p + (size_t)m * 16 * GN + r * 16), __builtin_bit_cast(u32x4, o));
        }
}

__device__ __forceinline__ unsigned block_code(LAS unsigned* word, unsigned m, int lane) {
    m = wave_max8(m >> 23);
    if (lane == 0) __hip_atomic_fetch_max(word, m, __ATOMIC_RELAXED, __HIP_MEMORY_SCOPE_WORKGROUP);
    WAIT_L(0); BAR();
    return min(((unsigned)__builtin_amdgcn_readfirstlane(*word) + 3u) & ~3u, 255u);
}
__device__ __forceinline__ void load_xblock(const Params& P, f32x4 (&v)[16], int half, int st, int mt, int wid, int fr, int fq) {
    const float* s = P.x + (size_t)(half * 4096 + (st >> 3) * 2048 + mt * 128 + wid * 16 + fr) * KS + (st & 7) * 128 + fq * 8;
#pragma unroll
    for (int i = 0; i < 8; ++i) v[i] = *(const f32x4*)(s + (i >> 1) * 32 + (i & 1) * 4);
}

__device__ __forceinline__ void convert_role(const Params& P, LAS unsigned char* lds, unsigned call, int wid, int lane) {
    const int fr = lane & 15, fq = lane >> 4, rank = P.rank;
    const unsigned par = OFF_DATA + (call & 1u) * PAR_BYTES;
    LAS unsigned* red = (LAS unsigned*)(lds + LDS_MISC + 256);
    const __amdgpu_buffer_rsrc_t rl = mk_rsrc(P.ws[rank], WS_USED);
    int n = 0;
    __builtin_amdgcn_s_sleep(CV_START);
    for (int i = blockIdx.x - NL - NG; i < 23 * 32; i += NE, ++n) {
        const int u = (i >> 5) + 1, sub = i & 31;
        const bool isw = u < 16 && (u & 1);
        const int st = u < 16 ? u >> 1 : u - 8;
        f32x4 v[16];
        unsigned m = 0, foff; int dst;
        if (isw) {
            const int b = sub >> 3, nt = sub & 7;
            const float* s = P.w + (size_t)(st * 128 + (wid >> 2) * 64 + fq * 8) * GN + b * 2048 + nt * 256 + (wid & 3) * 64 + 4 * (4 * (fr & 3) + (fr >> 2));
#pragma unroll
            for (int e = 0; e < 16; ++e) v[e] = *(const f32x4*)(s + (size_t)((e >> 3) * 32 + (e & 7)) * GN);
#pragma unroll
            for (int e = 0; e < 16; ++e) m = exp1_max4(m, __builtin_bit_cast(u32x4, v[e]));
            m = block_code(red + n, m, lane);
            const int sh = EXP_TARGET - (int)m;
            const unsigned off = par + (b == (rank & 3) ? REL_B + (st * 8 + rank) * CHUNK : REL_WO + (b * 8 + st) * CHUNK) + nt * 65536 + wid * 8192;
#pragma unroll
            for (int c = 0; c < 4; ++c)
#pragma unroll
                for (int kf = 0; kf < 2; ++kf) {
                    const f32x4* q = v + 8 * kf;
                    const u32x4 h = cvt8((f32x4){q[0][c], q[1][c], q[2][c], q[3][c]}, (f32x4){q[4][c], q[5][c], q[6][c], q[7][c]}, sh);
                    __builtin_amdgcn_raw_buffer_store_b128(h, rl, lane * 16 + off + c * 2048 + kf * 1024, 0, AUX_SYS);
                }
            foff = OFF_CVW + (unsigned)((st * 8 + nt) * 8 + rank) * 8u; dst = lane < 2 ? b + 4 * lane : -1;
        } else {
            const int half = sub >> 4, mt = sub & 15;
            load_xblock(P, v, half, st, mt, wid, fr, fq);
#pragma unroll
            for (int e = 0; e < 8; ++e) m = exp1_max4(m, __builtin_bit_cast(u32x4, v[e]));
            m = block_code(red + n, m, lane);
            const int sh = EXP_TARGET - (int)m;
            const unsigned off = par + (half == (rank >> 2) ? (st * 8 + rank) * CHUNK : REL_XO + st * CHUNK) + mt * 32768 + wid * 2048;
#pragma unroll
            for (int kq = 0; kq < 4; ++kq)
                __builtin_amdgcn_raw_buffer_store_b128(cvt8(v[2 * kq], v[2 * kq + 1], sh), rl, lane * 16 + off + (kq >> 1) * 16384 + (kq & 1) * 1024, 0, AUX_SYS);
            foff = OFF_CVX + (unsigned)(((st * 16 + mt) * 8 + rank) * 4 + (lane & 3)) * 8u; dst = lane < 16 ? half * 4 + (lane >> 2) : -1;
        }
        WAIT_V(0); BAR();
        if (wid == 0 && dst >= 0)
            __hip_atomic_store((u64*)(P.ws[dst & 7] + foff), ((u64)call << 32) | m, __ATOMIC_RELAXED, __HIP_MEMORY_SCOPE_SYSTEM);
        if (n == 0) STAMP(61);
        __builtin_amdgcn_s_sleep(CV_DELAY);
    }
}

template <class F>
__device__ __forceinline__ unsigned convert_first(const Params& P, LAS unsigned char* lds, F&& ready, int wid, int lane) {
    const int gi = blockIdx.x - NL, half = gi >> 6, mt = (gi >> 2) & 15, sub = gi & 3, rank = P.rank;
    const float* s = P.x + (size_t)(half * 4096 + mt * 128 + (sub & 1) * 64 + (wid >> 1) * 16 + (lane & 15)) * KS + (sub >> 1) * 64 + (wid & 1) * 32 + (lane >> 4) * 8;
    const f32x4 v0 = *(const f32x4*)s, v1 = *(const f32x4*)(s + 4);
    const unsigned call = ready();
    LAS unsigned* red = (LAS unsigned*)(lds + LDS_MISC + 256);
    const unsigned m = block_code(red, exp1_max4(exp1_max4(0u, __builtin_bit_cast(u32x4, v0)), __builtin_bit_cast(u32x4, v1)), lane);
    STAMP(21);
    const unsigned off = OFF_DATA + (call & 1u) * PAR_BYTES + (half == (rank >> 2) ? rank * CHUNK : REL_XO) + mt * 32768 + sub * 8192 + wid * 1024 + lane * 16;
    __builtin_amdgcn_raw_buffer_store_b128(cvt8(v0, v1, EXP_TARGET - (int)m), mk_rsrc(P.ws[rank], WS_USED), off, 0, AUX_SYS);
    WAIT_V(0); BAR();
    if (wid == 0 && lane < 4)
        __hip_atomic_store((u64*)(P.ws[half * 4 + lane] + OFF_CVX) + (mt * 8 + rank) * 4 + sub, ((u64)call << 32) | m, __ATOMIC_RELAXED, __HIP_MEMORY_SCOPE_SYSTEM);
    STAMP(20);
    return call;
}

__device__ __forceinline__ void link_role(const Params& P, unsigned call, int wid, int lane) {
    if (wid >= PW) return;
    const int rank = P.rank;
    const unsigned par = OFF_DATA + (call & 1u) * PAR_BYTES;
    char* wsl = P.ws[rank];
    unsigned* tmo = (unsigned*)(wsl + OFF_TMO);
    const unsigned lo16 = lane * 16;
    unsigned* q = (unsigned*)(wsl + OFF_LNKQ);
    constexpr unsigned NITEM = NITEM_IN + NSTG + TAILR * (TAILP - 1);
    const unsigned qbase = (call - 1u) * NITEM - NL * PW;
    unsigned tk = qbase + wid * NL + blockIdx.x, vz = 0;
    asm volatile("" : "+v"(vz));
    const int lsrc = (rank + 1 + lane) & 7;
    int prev = -1;
    auto publish = [&]() {
        if (prev >= 0 && lane == 0) __hip_atomic_fetch_add((unsigned*)(wsl + OFF_PULLED + (unsigned)prev * 64u), 1u, __ATOMIC_RELAXED, __HIP_MEMORY_SCOPE_AGENT);
        prev = -1;
    };
    for (;;) {
        const unsigned it = (unsigned)__builtin_amdgcn_readfirstlane(tk) - qbase;
        if (it >= NITEM) break;
        if (lane == 0) tk = __hip_atomic_fetch_add(q + vz, 1u, __ATOMIC_RELAXED, __HIP_MEMORY_SCOPE_AGENT);
        const bool isrec = it >= NITEM_IN;
        int id; unsigned foff, soff, doff, o0 = 0, ostep = 1024 * PULLU;
        if (isrec) {
            unsigned rec = it - NITEM_IN;
            if (rec >= NSTG - TAILR) { const unsigned t = rec - (NSTG - TAILR); rec = NSTG - TAILR + t / TAILP; o0 = (t % TAILP) * ostep; ostep *= TAILP; }
            id = 320 + (int)rec; foff = OFF_RECFLAG + (unsigned)(lsrc * NREC + rec) * 8u;
            soff = doff = par + REL_O + (unsigned)(lsrc * NREC + rec) * REC_BYTES;
        } else {
            const int u = (int)(it >> 6), sub = it & 63;
            const bool isw = u < 16 && (u & 1);
            const int st = u < 16 ? u >> 1 : u - 8;
            id = isw ? 256 + st * 8 + (sub >> 3) : st * 16 + (sub >> 2);
            foff = isw ? OFF_CVW + (unsigned)((id - 256) * 8 + lsrc) * 8u : OFF_CVX + (unsigned)((id * 8 + lsrc) * 4 + (sub & 3)) * 8u;
            doff = par + (isw ? REL_B : 0u) + (unsigned)(st * 8 + lsrc) * CHUNK + sub * 8192;
            const bool same = isw ? (lsrc & 3) == (rank & 3) : (lsrc >> 2) == (rank >> 2);
            soff = same ? doff : par + (isw ? REL_WO + (unsigned)((rank & 3) * 8 + st) * CHUNK : REL_XO + (unsigned)st * CHUNK) + sub * 8192;
            if ((it & 127u) == 0u) STAMPW(7, 1 + (it >> 7));
        }
        Deadline dl(tmo, call);
        u64 f = 0; bool bad = false;
        for (;;) {
            if (lane < 7) f = __hip_atomic_load((const u64*)(wsl + foff), __ATOMIC_RELAXED, __HIP_MEMORY_SCOPE_SYSTEM);
            const u64 anylate = __builtin_amdgcn_ballot_w64(lane < 7 && (unsigned)(f >> 32) < call);
            WAIT_V(0);
            publish();
            if (anylate == 0ull) break;
            if (dl.expired()) { bad = true; break; }
            __builtin_amdgcn_s_sleep(PSLEEP);
        }
        if (bad || __builtin_amdgcn_ballot_w64(lane < 7 && (unsigned)f > 4096u) != 0ull) continue;
        if (it < 64) STAMP(22);
        const unsigned lbytes = isrec ? 512u + (unsigned)f * 4u : 8192u;
        __amdgpu_buffer_rsrc_t rp[7], rl[7];
        unsigned maxb = 0;
#pragma unroll
        for (int k = 0; k < 7; ++k) {
            const unsigned bytes = __builtin_amdgcn_readlane(lbytes, k);
            rp[k] = mk_rsrc(P.ws[(rank + 1 + k) & 7] + (unsigned)__builtin_amdgcn_readlane(soff, k), bytes);
            rl[k] = mk_rsrc(wsl + (unsigned)__builtin_amdgcn_readlane(doff, k), bytes);
            maxb = max(maxb, bytes);
        }
        for (unsigned o = o0; o < maxb; o += ostep) {
            u32x4 d[PULLU][7];
#pragma unroll
            for (int u = 0; u < PULLU; ++u)
#pragma unroll
                for (int k = 0; k < 7; ++k) d[u][k] = __builtin_amdgcn_raw_buffer_load_b128(rp[k], lo16 + o + u * 1024, 0, AUX_SYS);
#pragma unroll
            for (int u = 0; u < PULLU; ++u)
#pragma unroll
                for (int k = 0; k < 7; ++k) __builtin_amdgcn_raw_buffer_store_b128(d[u][k], rl[k], lo16 + o + u * 1024, 0, AUX_SYS);
        }
        prev = id;
    }
    WAIT_V(0);
    publish();
}

__device__ __forceinline__ void wait_inputs(const Params& P, LAS unsigned char* lds, int st, int mt, int nt, unsigned call, int lane) {
    char* wsl = P.ws[P.rank];
    const int bx = st * 16 + mt, bw = 256 + (st & 7) * 8 + nt;
    const unsigned off = lane < 32 ? OFF_CVX + (unsigned)(bx * 32 + lane) * 8u : lane < 40 ? OFF_CVW + (unsigned)((bw - 256) * 8 + (lane & 7)) * 8u
                                   : OFF_PULLED + (unsigned)(lane == 40 ? bx : bw) * 64u;
    const unsigned need = lane == 40 ? 4u * call : 8u * call;
    Deadline dl((unsigned*)(wsl + OFF_TMO), call);
    u64 f = 0;
    for (;;) {
        if (lane < 42) f = __hip_atomic_load((const u64*)(wsl + off), __ATOMIC_RELAXED, __HIP_MEMORY_SCOPE_SYSTEM);
        const bool late = lane < 40 ? (unsigned)(f >> 32) < call : lane < 42 && (unsigned)f < need;
        if (__builtin_amdgcn_ballot_w64(late) == 0ull) break;
        if (dl.expired()) break;
        if (st == 8) __builtin_amdgcn_s_sleep(GSLEEP8); else __builtin_amdgcn_s_sleep(GSLEEP);
    }
    __builtin_amdgcn_fence(__ATOMIC_ACQUIRE, "");
    LAS int* tab = (LAS int*)(lds + LDS_MISC + 1280);
    const int cw = __shfl((int)((unsigned)f & 0xffu), 32 + ((lane >> 2) & 7));
    if (lane < 32) tab[(st & 7) * 32 + lane] = (int)((unsigned)f & 0xffu) + cw;
    WAIT_V(0); WAIT_L(0);
}

__device__ __forceinline__ void gemm_role(const Params& P, LAS unsigned char* lds, unsigned call, int wid, int lane) {
    const int gi = blockIdx.x - NL, xcd = gi & 7, ix = (gi & 127) >> 3;
    const int wr = wid >> 2, wc = wid & 3, fr = lane & 15, fq = lane >> 4;
    const unsigned par = OFF_DATA + (call & 1u) * PAR_BYTES;
    char* wsl = P.ws[P.rank];
    const int nt = (xcd & 1) * 4 + (ix & 3);
    const unsigned aoff = (unsigned)(wr * 8192 + lane * 16), boff = (unsigned)(16384 + wc * 8192 + lane * 16);
    const LAS int* tab = (const LAS int*)(lds + LDS_MISC + 1280) + wr;
    const int ntile = gi < 32 ? 2 : 1;
    for (int ti = 0; ti < ntile; ++ti) {
        const int mt = gi >= NG ? 16 + (xcd >> 1) * 3 + (ix >> 2) : ti == 0 ? (xcd >> 1) * 4 + (ix >> 2) : 28 + (xcd >> 1);
        const int job = mt >> 4;
        const char* ga = wsl + par + (size_t)job * (64 * CHUNK) + (mt & 15) * 32768 + wid * 2048 + lane * 16;
        const char* gb = wsl + par + REL_B + nt * 65536 + wid * 4096 + lane * 16;
#define ISSUE(kt_, sb_) do { const char* _a = ga + (((kt_) >> 1) * CHUNK + ((kt_) & 1) * 16384); const char* _b = gb + (((kt_) >> 1) * CHUNK + ((kt_) & 1) * 32768); \
        LAS unsigned char* _la = lds + (sb_) + wid * 2048; LAS unsigned char* _lb = lds + (sb_) + 16384 + wid * 4096; \
        __builtin_amdgcn_global_load_lds((const unsigned*)_a, (LAS unsigned*)_la, 16, 0, 0); \
        __builtin_amdgcn_global_load_lds((const unsigned*)_a, (LAS unsigned*)_la, 16, 1024, 0); \
        __builtin_amdgcn_global_load_lds((const unsigned*)_b, (LAS unsigned*)_lb, 16, 0, 0); \
        __builtin_amdgcn_global_load_lds((const unsigned*)_b, (LAS unsigned*)_lb, 16, 1024, 0); \
        __builtin_amdgcn_global_load_lds((const unsigned*)_b, (LAS unsigned*)_lb, 16, 2048, 0); \
        __builtin_amdgcn_global_load_lds((const unsigned*)_b, (LAS unsigned*)_lb, 16, 3072, 0); } while (0)
#define LDF(A_, B_, sb_, kf_) do { _Pragma("unroll") for (int i = 0; i < 4; ++i) { \
        B_[i] = *(const LAS h16x8*)(lds + (sb_) + boff + i * 2048 + (kf_) * 1024); \
        A_[i] = *(const LAS h16x8*)(lds + (sb_) + aoff + i * 2048 + (kf_) * 1024); } } while (0)
#define MMA(A_, B_) do { __builtin_amdgcn_s_setprio(1); _Pragma("unroll") for (int m = 0; m < 4; ++m) _Pragma("unroll") for (int c = 0; c < 4; ++c) \
        acc[m][c] = __builtin_amdgcn_mfma_f32_16x16x32_f16(B_[c], A_[m], acc[m][c], 0, 0, 0); __builtin_amdgcn_s_setprio(0); } while (0)
        f32x4 acc[4][4], tot[4][4];
#pragma unroll
        for (int m = 0; m < 4; ++m)
#pragma unroll
            for (int c = 0; c < 4; ++c) acc[m][c] = tot[m][c] = (f32x4){0.f, 0.f, 0.f, 0.f};
        h16x8 A0[4], B0[4], A1[4], B1[4];
        if (wid == 0) wait_inputs(P, lds, job * 8, mt & 15, nt, call, lane);
        BAR();
        STAMP(1 + job * 8);
        ISSUE(0, 0); ISSUE(1, STAGE); ISSUE(2, 2 * STAGE);
        WAIT_V(12); BAR();
        LDF(A0, B0, 0u, 0);
        int tnext = tab[0];
        int tcur = __builtin_amdgcn_readfirstlane(tnext);
        int tref = tcur;
        unsigned sb = 0;
        for (int kt = 0; kt < 128; ++kt) {
            const unsigned sn = sb == 2 * STAGE ? 0u : sb + STAGE;
            __builtin_amdgcn_s_waitcnt(0xC07F);
            LDF(A1, B1, sb, 1);
            const int tn = __builtin_amdgcn_readfirstlane(tnext);
            if (tn != tcur) {
                const bool nearby = abs(tn - tref) <= 30;
                const int ekeep = nearby ? tcur - tn : -2000, ebank = nearby ? -2000 : tcur - 2 * EXP_TARGET;
#pragma unroll
                for (int m = 0; m < 4; ++m)
#pragma unroll
                    for (int c = 0; c < 4; ++c)
#pragma unroll
                        for (int r = 0; r < 4; ++r) {
                            tot[m][c][r] += __builtin_ldexpf(acc[m][c][r], ebank);
                            acc[m][c][r] = __builtin_ldexpf(acc[m][c][r], ekeep);
                        }
                tref = nearby ? tref : tn;
                tcur = tn;
            }
            MMA(A0, B0);
            if (((kt + 3) & 15) == 0 && kt + 3 < 128 && wid == 0) { STAMP(32 + job * 8 + ((kt + 3) >> 4)); wait_inputs(P, lds, job * 8 + ((kt + 3) >> 4), mt & 15, nt, call, lane); STAMP(16 + job * 8 + ((kt + 3) >> 4)); }
            WAIT_V(6); __builtin_amdgcn_s_waitcnt(0xC07F); BAR();
            ISSUE((kt + 3) & 127, sb);
            LDF(A0, B0, sn, 0);
            tnext = tab[min(kt + 1, 127) * 2];
            MMA(A1, B1);
            sb = sn;
        }
        WAIT_V(0); WAIT_L(0); BAR();
        STAMP(2 + job * 8);
        unsigned* tmo = (unsigned*)(wsl + OFF_TMO);
        const bool bad = (unsigned)__builtin_amdgcn_readfirstlane(__hip_atomic_load(tmo, __ATOMIC_RELAXED, __HIP_MEMORY_SCOPE_AGENT)) == call;
        LAS float* cv = (LAS float*)(lds + wid * 16384);
        unsigned mlo = 0, mhi = 0, off = 0;
        const int unsc = tcur - 2 * EXP_TARGET;
#pragma unroll
        for (int m = 0; m < 4; ++m)
#pragma unroll
            for (int c = 0; c < 4; ++c)
#pragma unroll
                for (int r = 0; r < 4; ++r) {
                    float v = tot[m][c][r] + __builtin_ldexpf(acc[m][c][r], unsc);
                    v = v <= 0.f ? 0.f : v;
                    if (bad) v = __builtin_nanf("");
                    acc[m][c][r] = v;
                    const bool nz = __float_as_uint(v) != 0u;
                    const u64 mk = __builtin_amdgcn_ballot_w64(nz);
                    const unsigned lo = (unsigned)mk, hi = (unsigned)(mk >> 32);
                    const unsigned pos = off + __builtin_amdgcn_mbcnt_hi(hi, __builtin_amdgcn_mbcnt_lo(lo, 0u));
                    if (nz) cv[pos] = v;
                    const int q = (m * 4 + c) * 4 + r;
                    mlo = lane == q ? lo : mlo; mhi = lane == q ? hi : mhi;
                    off += (unsigned)__builtin_popcountll(mk);
                }
        const int rec = (mt * 8 + nt) * 8 + wid;
        const __amdgpu_buffer_rsrc_t rl = mk_rsrc(wsl, WS_USED);
        const unsigned ro = par + REL_O + (unsigned)(P.rank * NREC + rec) * REC_BYTES;
        __builtin_amdgcn_raw_buffer_store_b64((u32x2){mlo, mhi}, rl, lane * 8, ro, AUX_SYS);
        WAIT_L(0);
        for (unsigned i = 0; i * 256 < off; ++i) {
            const u32x4 d = *(const LAS u32x4*)((LAS unsigned char*)cv + i * 1024 + lane * 16);
            __builtin_amdgcn_raw_buffer_store_b128(d, rl, lane * 16 + ro + 512 + i * 1024, 0, AUX_SYS);
        }
        LAS unsigned* nz8 = (LAS unsigned*)(lds + LDS_MISC + 1024);
        if (lane == 0) nz8[wid] = off;
        WAIT_V(0); WAIT_L(0); BAR();
        if (wid == 0 && (lane >> 3) != P.rank)
            __hip_atomic_store((u64*)(P.ws[lane >> 3] + OFF_RECFLAG) + (unsigned)(P.rank * NREC + rec + (lane & 7)), ((u64)call << 32) | nz8[lane & 7], __ATOMIC_RELAXED, __HIP_MEMORY_SCOPE_SYSTEM);
        store_dense(P.out, (P.rank >> 2) * 4096 + mt * 128 + wr * 64, (P.rank & 3) * 2048 + nt * 256 + wc * 64, fr, fq, acc);
        WAIT_L(0); BAR();
        STAMP(3 + job * 8);
#undef ISSUE
#undef LDF
#undef MMA
    }
}

__device__ __forceinline__ void expand_role(const Params& P, LAS unsigned char* lds, unsigned call, int wid, int lane) {
    const int fr = lane & 15, fq = lane >> 4;
        constexpr unsigned NJOB = 7 * NSTG, E = NE * EW;
    const int gi = blockIdx.x - NL;
    const bool direct = gi < 32 || gi >= NG;
    const unsigned didx = (unsigned)(wid * 128 + (gi < 32 ? gi : gi - 96));
    if (direct ? didx >= 7u * DIRR : wid >= EW) return;
    const unsigned par = OFF_DATA + (call & 1u) * PAR_BYTES;
    char* wsl = P.ws[P.rank];
    unsigned* tmo = (unsigned*)(wsl + OFF_TMO);
    LAS unsigned char* buf = lds + wid * 16384;
    unsigned* q = (unsigned*)(wsl + OFF_EXPQ);
    const unsigned qbase = (call - 1u) * (NJOB + E);
    unsigned tk = qbase + NJOB + didx;
    if (!direct) { if (lane == 0) tk = __hip_atomic_fetch_add(q, 1u, __ATOMIC_RELAXED, __HIP_MEMORY_SCOPE_AGENT); }
    else {
        Deadline dl(tmo, call);
        while ((unsigned)__builtin_amdgcn_readfirstlane(__hip_atomic_load((unsigned*)(wsl + OFF_PULLED + (unsigned)(320 + NSTG - TAILR - 32) * 64u), __ATOMIC_RELAXED, __HIP_MEMORY_SCOPE_AGENT)) < call) {
            if (dl.expired()) break;
            __builtin_amdgcn_s_sleep(127); __builtin_amdgcn_s_sleep(127);
        }
    }
    int it = direct ? 1 : 0;
    unsigned vz = 0;
    asm volatile("" : "+v"(vz));
    for (;; ++it) {
        const unsigned ku = (unsigned)__builtin_amdgcn_readfirstlane(tk) - qbase;
        if (ku >= NJOB && !direct) break;
        const int k = (int)ku;
        const int rec = k / 7, src = (P.rank + 1 + (k - rec * 7)) & 7;
        const u64* flag = (const u64*)(wsl + (lane == 0 ? OFF_PULLED + (unsigned)(320 + (direct ? NSTG - DIRG + (rec - NSTG) * DIRS / DIRR : rec)) * 64u : OFF_RECFLAG + (unsigned)(src * NREC + rec) * 8u));
        const unsigned need = rec >= NSTG - TAILR ? TAILP * call : call;
        Deadline dl(tmo, call);
        unsigned nnz = 0; bool bad = false;
        for (;;) {
            u64 f = 0;
            if (lane < 2) f = __hip_atomic_load(flag, __ATOMIC_RELAXED, __HIP_MEMORY_SCOPE_SYSTEM);
            nnz = __builtin_amdgcn_readlane((unsigned)f, 1);
            if ((unsigned)__builtin_amdgcn_readlane((unsigned)f, 0) >= need && (unsigned)__builtin_amdgcn_readlane((unsigned)(f >> 32), 1) >= call) break;
            if (dl.expired()) { bad = true; break; }
            if (it == 0) __builtin_amdgcn_s_sleep(100); else __builtin_amdgcn_s_sleep(ESLEEP);
        }
        if (nnz > 4096u) { nnz = 0; bad = true; }
        f32x4 v[4][4];
        if (!bad) {
            const char* rbase = (direct ? P.ws[src] : wsl) + par + REL_O + (size_t)(src * NREC + rec) * REC_BYTES;
            const __amdgpu_buffer_rsrc_t rr = mk_rsrc(rbase, REC_BYTES);
            const u32x2 hdr = __builtin_amdgcn_raw_buffer_load_b64(rr, lane * 8, 0, AUX_SYS);
            const char* gsrc = rbase + 512 + lane * 16;
            const unsigned bytes = nnz * 4;
            for (unsigned i = 0; i < bytes; i += 1024)
                if (i + lane * 16 < bytes) __builtin_amdgcn_global_load_lds((const unsigned*)(gsrc + i), (LAS unsigned*)(buf + i), 16, 0, AUX_SYS);
            WAIT_V(0);
            unsigned off = 0;
#pragma unroll
            for (int m = 0; m < 4; ++m)
#pragma unroll
                for (int c = 0; c < 4; ++c)
#pragma unroll
                    for (int r = 0; r < 4; ++r) {
                        const int qq = (m * 4 + c) * 4 + r;
                        const unsigned lo = __builtin_amdgcn_readlane(hdr[0], qq), hi = __builtin_amdgcn_readlane(hdr[1], qq);
                        const u64 mk = ((u64)hi << 32) | lo;
                        const unsigned pos = (off + __builtin_amdgcn_mbcnt_hi(hi, __builtin_amdgcn_mbcnt_lo(lo, 0u))) & 4095u;
                        const float t = *(const LAS float*)(buf + pos * 4);
                        v[m][c][r] = __builtin_amdgcn_inverse_ballot_w64(mk) ? t : 0.f;
                        off += (unsigned)__builtin_popcountll(mk);
                    }
        } else {
#pragma unroll
            for (int m = 0; m < 4; ++m)
#pragma unroll
                for (int c = 0; c < 4; ++c) { const float qn = __builtin_nanf(""); v[m][c] = (f32x4){qn, qn, qn, qn}; }
        }
        if (lane == 0 && !direct) tk = __hip_atomic_fetch_add(q + vz, 1u, __ATOMIC_RELAXED, __HIP_MEMORY_SCOPE_AGENT);
        const int w8 = rec & 7, nt = (rec >> 3) & 7, mt = rec >> 6;
        const int row0 = (src >> 2) * 4096 + mt * 128 + (w8 >> 2) * 64, col0 = (src & 3) * 2048 + nt * 256 + (w8 & 3) * 64;
        store_dense(P.out, row0, col0, fr, fq, v);
        WAIT_L(0);
        if (direct) break;
        if (it < 60) STAMPW(4 + wid, 1 + it);
    }
}

__global__ __launch_bounds__(512, 2) void fused_kernel(Params P) {
    extern __shared__ __attribute__((aligned(16))) unsigned char shm[];
    LAS unsigned char* lds = (LAS unsigned char*)shm;
    const int tid = threadIdx.x, lane = tid & 63, wid = __builtin_amdgcn_readfirstlane(tid >> 6);
    LAS unsigned* misc = (LAS unsigned*)(lds + LDS_MISC);
    STAMP(0);
    unsigned ticket = 0, vz = 0;
    asm volatile("" : "+v"(vz));
    if (tid == 0) ticket = __hip_atomic_fetch_add((unsigned*)(P.ws[P.rank] + OFF_CTR + (blockIdx.x & 7) * 128 + vz), 1u, __ATOMIC_RELAXED, __HIP_MEMORY_SCOPE_AGENT);
    if (tid >= 64 && tid < 128) misc[tid] = 0u;
    auto ready = [&]() -> unsigned {
        asm volatile("" : "+v"(ticket));
        if (tid == 0) misc[0] = ticket / (unsigned)(GRID / 8) + 1u;
        WAIT_L(0); BAR();
        return __builtin_amdgcn_readfirstlane(misc[0]);
    };
    if (blockIdx.x < NL) link_role(P, ready(), wid, lane);
    else {
        unsigned call;
        if (blockIdx.x < NL + NG) call = convert_first(P, lds, ready, wid, lane);
        else { call = ready(); convert_role(P, lds, call, wid, lane); }
        STAMP(62);
        gemm_role(P, lds, call, wid, lane);
        expand_role(P, lds, call, wid, lane);
    }
    STAMP(63);
}

}

extern "C" void kernel_launch(int rank, int nranks, ncclComm_t comm,
                              hipStream_t stream,
                              void* const* d_in, const int* in_sizes, int n_in,
                              void* d_out, int out_size,
                              const int* in_dtypes, int out_dtype,
                              void* const* d_out_all,
                              void* const* d_ws_all, size_t ws_bytes) {
    using namespace gar;
    if (nranks != NRANK || (unsigned)rank >= (unsigned)NRANK || ws_bytes < (size_t)WS_USED || n_in < 2 ||
        in_sizes[0] != GM * KS || in_sizes[1] != KS * GN || out_size != GM * GN) {
        fprintf(stderr, "kernel_launch: unexpected problem shape\n");
        return;
    }
    Params P;
    P.x = (const float*)d_in[0]; P.w = (const float*)d_in[1]; P.out = (float*)d_out; P.rank = rank;
    for (int p = 0; p < NRANK; ++p) P.ws[p] = (char*)d_ws_all[p];
    HIP_CHECK(hipFuncSetAttribute((const void*)fused_kernel, hipFuncAttributeMaxDynamicSharedMemorySize, LDS_BYTES));
    hipLaunchKernelGGL(fused_kernel, dim3(GRID), dim3(512), LDS_BYTES, stream, P);
    HIP_CHECK(hipGetLastError());
}
```

```cpp
#include <hip/hip_runtime.h>
#include <rccl/rccl.h>
#include <cstdio>

#define HIP_CHECK(expr) do { hipError_t _e = (expr); if (_e != hipSuccess) { \
    fprintf(stderr, "HIP error %s at %s:%d\n", hipGetErrorString(_e), __FILE__, __LINE__); \
    return; } } while (0)

namespace gar {

#define LAS __attribute__((address_space(3)))
typedef _Float16 h16x8 __attribute__((ext_vector_type(8)));
typedef float f32x4 __attribute__((ext_vector_type(4)));
typedef unsigned u32x4 __attribute__((ext_vector_type(4)));
typedef unsigned u32x2 __attribute__((ext_vector_type(2)));
typedef unsigned long long u64;

constexpr int NRANK = 8;
constexpr int GM = 8192, GN = 8192, KS = 1024;
constexpr int PW = 4;
constexpr int EW = 4;
constexpr int TAILR = 192, TAILP = 4;
constexpr int DIRR = 32;
constexpr int NL = 32;
constexpr int NG = 128;
constexpr int NE = 96;
constexpr int GRID = NL + NG + NE;
constexpr int NREC = 2048;
constexpr int NSTG = NREC - DIRR;
constexpr unsigned REC_BYTES = 17408;
static_assert(GRID == 256 && DIRR <= TAILR && 7 * DIRR <= 8 * 128 && PW <= 8 && EW <= 8, "geometry");
constexpr unsigned NITEM_IN = 24 * 64;

constexpr unsigned OFF_TMO = 0, OFF_EXPQ = 128, OFF_LNKQ = 256, OFF_CTR = 1024;
constexpr unsigned OFF_CVW = 4096;
constexpr unsigned OFF_CVX = 0x180000;
constexpr unsigned OFF_PULLED = 65536;
constexpr unsigned OFF_RECFLAG = 0x140000;
constexpr unsigned OFF_DATA = 2u << 20;
constexpr unsigned CHUNK = 512u << 10;
constexpr unsigned A_BYTES = 128 * CHUNK, B_BYTES = 64 * CHUNK, XO_BYTES = 16 * CHUNK, WO_BYTES = 32 * CHUNK, O_BYTES = NRANK * NREC * REC_BYTES;
constexpr unsigned REL_B = A_BYTES, REL_XO = REL_B + B_BYTES, REL_WO = REL_XO + XO_BYTES, REL_O = REL_WO + WO_BYTES;
constexpr unsigned PAR_BYTES = REL_O + O_BYTES;
constexpr unsigned WS_USED = OFF_DATA + 2 * PAR_BYTES;
constexpr unsigned STAGE = 49152, LDS_MISC = 3 * STAGE, LDS_BYTES = LDS_MISC + 2304;
constexpr u64 DEADLINE = 50000000ull;
constexpr int AUX_SYS = 17;
constexpr int EXP_TARGET = 142;

struct Params {
    const float* x; const float* w; float* out; char* ws[NRANK]; int rank;
};

#define WAIT_V(n) asm volatile("s_waitcnt vmcnt(" #n ")" ::: "memory")
#define WAIT_L(n) asm volatile("s_waitcnt lgkmcnt(" #n ")" ::: "memory")
#define BAR() do { asm volatile("" ::: "memory"); __builtin_amdgcn_s_barrier(); asm volatile("" ::: "memory"); } while (0)

__device__ __forceinline__ __amdgpu_buffer_rsrc_t mk_rsrc(const void* p, unsigned bytes) {
    return __builtin_amdgcn_make_buffer_rsrc((void*)p, 0, bytes, 0x00020000);
}
__device__ __forceinline__ void st16_sys(char* p, u32x4 v) {
    __builtin_amdgcn_global_store_b128((__attribute__((address_space(1))) u32x4*)p, v, "");
}
__device__ __forceinline__ u32x4 cvt8(f32x4 a, f32x4 b, int sh) {
    h16x8 h;
#pragma unroll
    for (int j = 0; j < 4; ++j) { h[j] = (_Float16)__builtin_ldexpf(a[j], sh); h[4 + j] = (_Float16)__builtin_ldexpf(b[j], sh); }
    return __builtin_bit_cast(u32x4, h);
}
struct Deadline {
    unsigned* tmo; unsigned call; u64 t0; unsigned polls;
    __device__ __forceinline__ Deadline(unsigned* tmo_, unsigned call_) : tmo(tmo_), call(call_), t0(__builtin_amdgcn_s_memrealtime()), polls(1) {}
    __device__ __forceinline__ bool expired() {
        if ((polls++ & 63u) == 0u &&
            (unsigned)__builtin_amdgcn_readfirstlane(__hip_atomic_load(tmo, __ATOMIC_RELAXED, __HIP_MEMORY_SCOPE_AGENT)) == call) return true;
        if (__builtin_amdgcn_s_memrealtime() - t0 > DEADLINE) {
            __hip_atomic_store(tmo, call, __ATOMIC_RELAXED, __HIP_MEMORY_SCOPE_AGENT);
            return true;
        }
        return false;
    }
};
__device__ __forceinline__ unsigned wave_max8(unsigned v) {
    unsigned r = 0;
#pragma unroll
    for (int b = 7; b >= 0; --b) { const unsigned c = r | (1u << b); r = __builtin_amdgcn_ballot_w64(v >= c) != 0ull ? c : r; }
    return r;
}
__device__ __forceinline__ unsigned exp1_max4(unsigned m, u32x4 v) {
#pragma unroll
    for (int i = 0; i < 4; ++i) m = max(m, (v[i] + 0x00800000u) & 0x7f800000u);
    return m;
}

__device__ __forceinline__ void store_dense(float* out, int row0, int col0, int fr, int fq, const f32x4 (&v)[4][4]) {
    float* p = out + (size_t)(row0 + fr) * GN + col0 + 4 * fq;
#pragma unroll
    for (int m = 0; m < 4; ++m)
#pragma unroll
        for (int r = 0; r < 4; ++r) {
            f32x4 o = {v[m][0][r], v[m][1][r], v[m][2][r], v[m][3][r]};
            st16_sys((char*)(p + (size_t)m * 16 * GN + r * 16), __builtin_bit_cast(u32x4, o));
        }
}

__device__ __forceinline__ unsigned block_code(LAS unsigned* word, unsigned m, int lane) {
    m = wave_max8(m >> 23);
    if (lane == 0) __hip_atomic_fetch_max(word, m, __ATOMIC_RELAXED, __HIP_MEMORY_SCOPE_WORKGROUP);
    WAIT_L(0); BAR();
    return min(((unsigned)__builtin_amdgcn_readfirstlane(*word) + 3u) & ~3u, 255u);
}
__device__ __forceinline__ void load_xblock(const Params& P, f32x4 (&v)[16], int half, int st, int mt, int wid, int fr, int fq) {
    const float* s = P.x + (size_t)(half * 4096 + (st >> 3) * 2048 + mt * 128 + wid * 16 + fr) * KS + (st & 7) * 128 + fq * 8;
#pragma unroll
    for (int i = 0; i < 8; ++i) v[i] = *(const f32x4*)(s + (i >> 1) * 32 + (i & 1) * 4);
}

__device__ __forceinline__ void convert_role(const Params& P, LAS unsigned char* lds, unsigned call, int wid, int lane) {
    const int fr = lane & 15, fq = lane >> 4, rank = P.rank;
    const unsigned par = OFF_DATA + (call & 1u) * PAR_BYTES;
    LAS unsigned* red = (LAS unsigned*)(lds + LDS_MISC + 256);
    const __amdgpu_buffer_rsrc_t rl = mk_rsrc(P.ws[rank], WS_USED);
    int n = 0;
    __builtin_amdgcn_s_sleep(64);
    for (int i = blockIdx.x - NL - NG; i < 23 * 32; i += NE, ++n) {
        const int u = (i >> 5) + 1, sub = i & 31;
        const bool isw = u < 16 && (u & 1);
        const int st = u < 16 ? u >> 1 : u - 8;
        f32x4 v[16];
        unsigned m = 0, foff; int dst;
        if (isw) {
            const int b = sub >> 3, nt = sub & 7;
            const float* s = P.w + (size_t)(st * 128 + (wid >> 2) * 64 + fq * 8) * GN + b * 2048 + nt * 256 + (wid & 3) * 64 + 4 * (4 * (fr & 3) + (fr >> 2));
#pragma unroll
            for (int e = 0; e < 16; ++e) v[e] = *(const f32x4*)(s + (size_t)((e >> 3) * 32 + (e & 7)) * GN);
#pragma unroll
            for (int e = 0; e < 16; ++e) m = exp1_max4(m, __builtin_bit_cast(u32x4, v[e]));
            m = block_code(red + n, m, lane);
            const int sh = EXP_TARGET - (int)m;
            const unsigned off = par + (b == (rank & 3) ? REL_B + (st * 8 + rank) * CHUNK : REL_WO + (b * 8 + st) * CHUNK) + nt * 65536 + wid * 8192;
#pragma unroll
            for (int c = 0; c < 4; ++c)
#pragma unroll
                for (int kf = 0; kf < 2; ++kf) {
                    const f32x4* q = v + 8 * kf;
                    const u32x4 h = cvt8((f32x4){q[0][c], q[1][c], q[2][c], q[3][c]}, (f32x4){q[4][c], q[5][c], q[6][c], q[7][c]}, sh);
                    __builtin_amdgcn_raw_buffer_store_b128(h, rl, lane * 16 + off + c * 2048 + kf * 1024, 0, AUX_SYS);
                }
            foff = OFF_CVW + (unsigned)((st * 8 + nt) * 8 + rank) * 8u; dst = lane < 2 ? b + 4 * lane : -1;
        } else {
            const int half = sub >> 4, mt = sub & 15;
            load_xblock(P, v, half, st, mt, wid, fr, fq);
#pragma unroll
            for (int e = 0; e < 8; ++e) m = exp1_max4(m, __builtin_bit_cast(u32x4, v[e]));
            m = block_code(red + n, m, lane);
            const int sh = EXP_TARGET - (int)m;
            const unsigned off = par + (half == (rank >> 2) ? (st * 8 + rank) * CHUNK : REL_XO + st * CHUNK) + mt * 32768 + wid * 2048;
#pragma unroll
            for (int kq = 0; kq < 4; ++kq)
                __builtin_amdgcn_raw_buffer_store_b128(cvt8(v[2 * kq], v[2 * kq + 1], sh), rl, lane * 16 + off + (kq >> 1) * 16384 + (kq & 1) * 1024, 0, AUX_SYS);
            foff = OFF_CVX + (unsigned)(((st * 16 + mt) * 8 + rank) * 4 + (lane & 3)) * 8u; dst = lane < 16 ? half * 4 + (lane >> 2) : -1;
        }
        WAIT_V(0); BAR();
        if (wid == 0 && dst >= 0)
            __hip_atomic_store((u64*)(P.ws[dst & 7] + foff), ((u64)call << 32) | m, __ATOMIC_RELAXED, __HIP_MEMORY_SCOPE_SYSTEM);
    }
}

template <class F>
__device__ __forceinline__ unsigned convert_first(const Params& P, LAS unsigned char* lds, F&& ready, int wid, int lane) {
    const int gi = blockIdx.x - NL, half = gi >> 6, mt = (gi >> 2) & 15, sub = gi & 3, rank = P.rank;
    const float* s = P.x + (size_t)(half * 4096 + mt * 128 + (sub & 1) * 64 + (wid >> 1) * 16 + (lane & 15)) * KS + (sub >> 1) * 64 + (wid & 1) * 32 + (lane >> 4) * 8;
    const f32x4 v0 = *(const f32x4*)s, v1 = *(const f32x4*)(s + 4);
    const unsigned call = ready();
    LAS unsigned* red = (LAS unsigned*)(lds + LDS_MISC + 256);
    const unsigned m = block_code(red, exp1_max4(exp1_max4(0u, __builtin_bit_cast(u32x4, v0)), __builtin_bit_cast(u32x4, v1)), lane);
    const unsigned off = OFF_DATA + (call & 1u) * PAR_BYTES + (half == (rank >> 2) ? rank * CHUNK : REL_XO) + mt * 32768 + sub * 8192 + wid * 1024 + lane * 16;
    __builtin_amdgcn_raw_buffer_store_b128(cvt8(v0, v1, EXP_TARGET - (int)m), mk_rsrc(P.ws[rank], WS_USED), off, 0, AUX_SYS);
    WAIT_V(0); BAR();
    if (wid == 0 && lane < 4)
        __hip_atomic_store((u64*)(P.ws[half * 4 + lane] + OFF_CVX) + (mt * 8 + rank) * 4 + sub, ((u64)call << 32) | m, __ATOMIC_RELAXED, __HIP_MEMORY_SCOPE_SYSTEM);
    return call;
}

__device__ __forceinline__ void link_role(const Params& P, unsigned call, int wid, int lane) {
    if (wid >= PW) return;
    const int rank = P.rank;
    const unsigned par = OFF_DATA + (call & 1u) * PAR_BYTES;
    char* wsl = P.ws[rank];
    unsigned* tmo = (unsigned*)(wsl + OFF_TMO);
    const unsigned lo16 = lane * 16;
    unsigned* q = (unsigned*)(wsl + OFF_LNKQ);
    constexpr unsigned NITEM = NITEM_IN + NSTG + TAILR * (TAILP - 1);
    const unsigned qbase = (call - 1u) * NITEM - NL * PW;
    unsigned tk = qbase + wid * NL + blockIdx.x, vz = 0;
    asm volatile("" : "+v"(vz));
    const int lsrc = (rank + 1 + lane) & 7;
    int prev = -1;
    auto publish = [&]() {
        if (prev >= 0 && lane == 0) __hip_atomic_fetch_add((unsigned*)(wsl + OFF_PULLED + (unsigned)prev * 64u), 1u, __ATOMIC_RELAXED, __HIP_MEMORY_SCOPE_AGENT);
        prev = -1;
    };
    for (;;) {
        const unsigned it = (unsigned)__builtin_amdgcn_readfirstlane(tk) - qbase;
        if (it >= NITEM) break;
        if (lane == 0) tk = __hip_atomic_fetch_add(q + vz, 1u, __ATOMIC_RELAXED, __HIP_MEMORY_SCOPE_AGENT);
        const bool isrec = it >= NITEM_IN;
        int id; unsigned foff, soff, doff, o0 = 0, ostep = 1024;
        if (isrec) {
            unsigned rec = it - NITEM_IN;
            if (rec >= NSTG - TAILR) { const unsigned t = rec - (NSTG - TAILR); rec = NSTG - TAILR + t / TAILP; o0 = (t % TAILP) * ostep; ostep *= TAILP; }
            id = 320 + (int)rec; foff = OFF_RECFLAG + (unsigned)(lsrc * NREC + rec) * 8u;
            soff = doff = par + REL_O + (unsigned)(lsrc * NREC + rec) * REC_BYTES;
        } else {
            const int u = (int)(it >> 6), sub = it & 63;
            const bool isw = u < 16 && (u & 1);
            const int st = u < 16 ? u >> 1 : u - 8;
            id = isw ? 256 + st * 8 + (sub >> 3) : st * 16 + (sub >> 2);
            foff = isw ? OFF_CVW + (unsigned)((id - 256) * 8 + lsrc) * 8u : OFF_CVX + (unsigned)((id * 8 + lsrc) * 4 + (sub & 3)) * 8u;
            doff = par + (isw ? REL_B : 0u) + (unsigned)(st * 8 + lsrc) * CHUNK + sub * 8192;
            const bool same = isw ? (lsrc & 3) == (rank & 3) : (lsrc >> 2) == (rank >> 2);
            soff = same ? doff : par + (isw ? REL_WO + (unsigned)((rank & 3) * 8 + st) * CHUNK : REL_XO + (unsigned)st * CHUNK) + sub * 8192;
        }
        Deadline dl(tmo, call);
        u64 f = 0; bool bad = false;
        for (;;) {
            if (lane < 7) f = __hip_atomic_load((const u64*)(wsl + foff), __ATOMIC_RELAXED, __HIP_MEMORY_SCOPE_SYSTEM);
            const u64 anylate = __builtin_amdgcn_ballot_w64(lane < 7 && (unsigned)(f >> 32) < call);
            WAIT_V(0);
            publish();
            if (anylate == 0ull) break;
            if (dl.expired()) { bad = true; break; }
            __builtin_amdgcn_s_sleep(8);
        }
        if (bad || __builtin_amdgcn_ballot_w64(lane < 7 && (unsigned)f > 4096u) != 0ull) continue;
        const unsigned lbytes = isrec ? 512u + (unsigned)f * 4u : 8192u;
        __amdgpu_buffer_rsrc_t rp[7], rl[7];
        unsigned maxb = 0;
#pragma unroll
        for (int k = 0; k < 7; ++k) {
            const unsigned bytes = __builtin_amdgcn_readlane(lbytes, k);
            rp[k] = mk_rsrc(P.ws[(rank + 1 + k) & 7] + (unsigned)__builtin_amdgcn_readlane(soff, k), bytes);
            rl[k] = mk_rsrc(wsl + (unsigned)__builtin_amdgcn_readlane(doff, k), bytes);
            maxb = max(maxb, bytes);
        }
        for (unsigned o = o0; o < maxb; o += ostep) {
            u32x4 d[7];
#pragma unroll
            for (int k = 0; k < 7; ++k) d[k] = __builtin_amdgcn_raw_buffer_load_b128(rp[k], lo16 + o, 0, AUX_SYS);
#pragma unroll
            for (int k = 0; k < 7; ++k) __builtin_amdgcn_raw_buffer_store_b128(d[k], rl[k], lo16 + o, 0, AUX_SYS);
        }
        prev = id;
    }
    WAIT_V(0);
    publish();
}

__device__ __forceinline__ void wait_inputs(const Params& P, LAS unsigned char* lds, int st, int mt, int nt, unsigned call, int lane) {
    char* wsl = P.ws[P.rank];
    const int bx = st * 16 + mt, bw = 256 + (st & 7) * 8 + nt;
    const unsigned off = lane < 32 ? OFF_CVX + (unsigned)(bx * 32 + lane) * 8u : lane < 40 ? OFF_CVW + (unsigned)((bw - 256) * 8 + (lane & 7)) * 8u
                                   : OFF_PULLED + (unsigned)(lane == 40 ? bx : bw) * 64u;
    const unsigned need = lane == 40 ? 4u * call : 8u * call;
    Deadline dl((unsigned*)(wsl + OFF_TMO), call);
    u64 f = 0;
    for (;;) {
        if (lane < 42) f = __hip_atomic_load((const u64*)(wsl + off), __ATOMIC_RELAXED, __HIP_MEMORY_SCOPE_SYSTEM);
        const bool late = lane < 40 ? (unsigned)(f >> 32) < call : lane < 42 && (unsigned)f < need;
        if (__builtin_amdgcn_ballot_w64(late) == 0ull) break;
        if (dl.expired()) break;
        if (st == 8) __builtin_amdgcn_s_sleep(127); else __builtin_amdgcn_s_sleep(32);
    }
    __builtin_amdgcn_fence(__ATOMIC_ACQUIRE, "");
    LAS int* tab = (LAS int*)(lds + LDS_MISC + 1280);
    const int cw = __shfl((int)((unsigned)f & 0xffu), 32 + ((lane >> 2) & 7));
    if (lane < 32) tab[(st & 7) * 32 + lane] = (int)((unsigned)f & 0xffu) + cw;
    WAIT_V(0); WAIT_L(0);
}

__device__ __forceinline__ void gemm_role(const Params& P, LAS unsigned char* lds, unsigned call, int wid, int lane) {
    const int gi = blockIdx.x - NL, xcd = gi & 7, ix = (gi & 127) >> 3;
    const int wr = wid >> 2, wc = wid & 3, fr = lane & 15, fq = lane >> 4;
    const unsigned par = OFF_DATA + (call & 1u) * PAR_BYTES;
    char* wsl = P.ws[P.rank];
    const int nt = (xcd & 1) * 4 + (ix & 3);
    const unsigned aoff = (unsigned)(wr * 8192 + lane * 16), boff = (unsigned)(16384 + wc * 8192 + lane * 16);
    const LAS int* tab = (const LAS int*)(lds + LDS_MISC + 1280) + wr;
    const int ntile = gi < 32 ? 2 : 1;
    for (int ti = 0; ti < ntile; ++ti) {
        const int mt = gi >= NG ? 16 + (xcd >> 1) * 3 + (ix >> 2) : ti == 0 ? (xcd >> 1) * 4 + (ix >> 2) : 28 + (xcd >> 1);
        const int job = mt >> 4;
        const char* ga = wsl + par + (size_t)job * (64 * CHUNK) + (mt & 15) * 32768 + wid * 2048 + lane * 16;
        const char* gb = wsl + par + REL_B + nt * 65536 + wid * 4096 + lane * 16;
#define ISSUE(kt_, sb_) do { const char* _a = ga + (((kt_) >> 1) * CHUNK + ((kt_) & 1) * 16384); const char* _b = gb + (((kt_) >> 1) * CHUNK + ((kt_) & 1) * 32768); \
        LAS unsigned char* _la = lds + (sb_) + wid * 2048; LAS unsigned char* _lb = lds + (sb_) + 16384 + wid * 4096; \
        __builtin_amdgcn_global_load_lds((const unsigned*)_a, (LAS unsigned*)_la, 16, 0, 0); \
        __builtin_amdgcn_global_load_lds((const unsigned*)_a, (LAS unsigned*)_la, 16, 1024, 0); \
        __builtin_amdgcn_global_load_lds((const unsigned*)_b, (LAS unsigned*)_lb, 16, 0, 0); \
        __builtin_amdgcn_global_load_lds((const unsigned*)_b, (LAS unsigned*)_lb, 16, 1024, 0); \
        __builtin_amdgcn_global_load_lds((const unsigned*)_b, (LAS unsigned*)_lb, 16, 2048, 0); \
        __builtin_amdgcn_global_load_lds((const unsigned*)_b, (LAS unsigned*)_lb, 16, 3072, 0); } while (0)
#define LDF(A_, B_, sb_, kf_) do { _Pragma("unroll") for (int i = 0; i < 4; ++i) { \
        B_[i] = *(const LAS h16x8*)(lds + (sb_) + boff + i * 2048 + (kf_) * 1024); \
        A_[i] = *(const LAS h16x8*)(lds + (sb_) + aoff + i * 2048 + (kf_) * 1024); } } while (0)
#define MMA(A_, B_) do { __builtin_amdgcn_s_setprio(1); _Pragma("unroll") for (int m = 0; m < 4; ++m) _Pragma("unroll") for (int c = 0; c < 4; ++c) \
        acc[m][c] = __builtin_amdgcn_mfma_f32_16x16x32_f16(B_[c], A_[m], acc[m][c], 0, 0, 0); __builtin_amdgcn_s_setprio(0); } while (0)
        f32x4 acc[4][4], tot[4][4];
#pragma unroll
        for (int m = 0; m < 4; ++m)
#pragma unroll
            for (int c = 0; c < 4; ++c) acc[m][c] = tot[m][c] = (f32x4){0.f, 0.f, 0.f, 0.f};
        h16x8 A0[4], B0[4], A1[4], B1[4];
        if (wid == 0) wait_inputs(P, lds, job * 8, mt & 15, nt, call, lane);
        BAR();
        ISSUE(0, 0); ISSUE(1, STAGE); ISSUE(2, 2 * STAGE);
        WAIT_V(12); BAR();
        LDF(A0, B0, 0u, 0);
        int tnext = tab[0];
        int tcur = __builtin_amdgcn_readfirstlane(tnext);
        int tref = tcur;
        unsigned sb = 0;
        for (int kt = 0; kt < 128; ++kt) {
            const unsigned sn = sb == 2 * STAGE ? 0u : sb + STAGE;
            __builtin_amdgcn_s_waitcnt(0xC07F);
            LDF(A1, B1, sb, 1);
            const int tn = __builtin_amdgcn_readfirstlane(tnext);
            if (tn != tcur) {
                const bool nearby = abs(tn - tref) <= 30;
                const int ekeep = nearby ? tcur - tn : -2000, ebank = nearby ? -2000 : tcur - 2 * EXP_TARGET;
#pragma unroll
                for (int m = 0; m < 4; ++m)
#pragma unroll
                    for (int c = 0; c < 4; ++c)
#pragma unroll
                        for (int r = 0; r < 4; ++r) {
                            tot[m][c][r] += __builtin_ldexpf(acc[m][c][r], ebank);
                            acc[m][c][r] = __builtin_ldexpf(acc[m][c][r], ekeep);
                        }
                tref = nearby ? tref : tn;
                tcur = tn;
            }
            MMA(A0, B0);
            if (((kt + 3) & 15) == 0 && kt + 3 < 128 && wid == 0) wait_inputs(P, lds, job * 8 + ((kt + 3) >> 4), mt & 15, nt, call, lane);
            WAIT_V(6); __builtin_amdgcn_s_waitcnt(0xC07F); BAR();
            ISSUE((kt + 3) & 127, sb);
            LDF(A0, B0, sn, 0);
            tnext = tab[min(kt + 1, 127) * 2];
            MMA(A1, B1);
            sb = sn;
        }
        WAIT_V(0); WAIT_L(0); BAR();
        unsigned* tmo = (unsigned*)(wsl + OFF_TMO);
        const bool bad = (unsigned)__builtin_amdgcn_readfirstlane(__hip_atomic_load(tmo, __ATOMIC_RELAXED, __HIP_MEMORY_SCOPE_AGENT)) == call;
        LAS float* cv = (LAS float*)(lds + wid * 16384);
        unsigned mlo = 0, mhi = 0, off = 0;
        const int unsc = tcur - 2 * EXP_TARGET;
#pragma unroll
        for (int m = 0; m < 4; ++m)
#pragma unroll
            for (int c = 0; c < 4; ++c)
#pragma unroll
                for (int r = 0; r < 4; ++r) {
                    float v = tot[m][c][r] + __builtin_ldexpf(acc[m][c][r], unsc);
                    v = v <= 0.f ? 0.f : v;
                    if (bad) v = __builtin_nanf("");
                    acc[m][c][r] = v;
                    const bool nz = __float_as_uint(v) != 0u;
                    const u64 mk = __builtin_amdgcn_ballot_w64(nz);
                    const unsigned lo = (unsigned)mk, hi = (unsigned)(mk >> 32);
                    const unsigned pos = off + __builtin_amdgcn_mbcnt_hi(hi, __builtin_amdgcn_mbcnt_lo(lo, 0u));
                    if (nz) cv[pos] = v;
                    const int q = (m * 4 + c) * 4 + r;
                    mlo = lane == q ? lo : mlo; mhi = lane == q ? hi : mhi;
                    off += (unsigned)__builtin_popcountll(mk);
                }
        const int rec = (mt * 8 + nt) * 8 + wid;
        const __amdgpu_buffer_rsrc_t rl = mk_rsrc(wsl, WS_USED);
        const unsigned ro = par + REL_O + (unsigned)(P.rank * NREC + rec) * REC_BYTES;
        __builtin_amdgcn_raw_buffer_store_b64((u32x2){mlo, mhi}, rl, lane * 8, ro, AUX_SYS);
        WAIT_L(0);
        for (unsigned i = 0; i * 256 < off; ++i) {
            const u32x4 d = *(const LAS u32x4*)((LAS unsigned char*)cv + i * 1024 + lane * 16);
            __builtin_amdgcn_raw_buffer_store_b128(d, rl, lane * 16 + ro + 512 + i * 1024, 0, AUX_SYS);
        }
        LAS unsigned* nz8 = (LAS unsigned*)(lds + LDS_MISC + 1024);
        if (lane == 0) nz8[wid] = off;
        WAIT_V(0); WAIT_L(0); BAR();
        if (wid == 0 && (lane >> 3) != P.rank)
            __hip_atomic_store((u64*)(P.ws[lane >> 3] + OFF_RECFLAG) + (unsigned)(P.rank * NREC + rec + (lane & 7)), ((u64)call << 32) | nz8[lane & 7], __ATOMIC_RELAXED, __HIP_MEMORY_SCOPE_SYSTEM);
        store_dense(P.out, (P.rank >> 2) * 4096 + mt * 128 + wr * 64, (P.rank & 3) * 2048 + nt * 256 + wc * 64, fr, fq, acc);
        WAIT_L(0); BAR();
#undef ISSUE
#undef LDF
#undef MMA
    }
}

__device__ __forceinline__ void expand_role(const Params& P, LAS unsigned char* lds, unsigned call, int wid, int lane) {
    const int fr = lane & 15, fq = lane >> 4;
    constexpr unsigned NJOB = 7 * NSTG, E = NE * EW;
    const int gi = blockIdx.x - NL;
    const bool direct = gi < 32 || gi >= NG;
    const unsigned didx = (unsigned)(wid * 128 + (gi < 32 ? gi : gi - 96));
    if (direct ? didx >= 7u * DIRR : wid >= EW) return;
    const unsigned par = OFF_DATA + (call & 1u) * PAR_BYTES;
    char* wsl = P.ws[P.rank];
    unsigned* tmo = (unsigned*)(wsl + OFF_TMO);
    LAS unsigned char* buf = lds + wid * 16384;
    unsigned* q = (unsigned*)(wsl + OFF_EXPQ);
    const unsigned qbase = (call - 1u) * (NJOB + E);
    unsigned tk = qbase + NJOB + didx;
    if (!direct) { if (lane == 0) tk = __hip_atomic_fetch_add(q, 1u, __ATOMIC_RELAXED, __HIP_MEMORY_SCOPE_AGENT); }
    else {
        Deadline dl(tmo, call);
        while ((unsigned)__builtin_amdgcn_readfirstlane(__hip_atomic_load((unsigned*)(wsl + OFF_PULLED + (unsigned)(320 + NSTG - TAILR - 32) * 64u), __ATOMIC_RELAXED, __HIP_MEMORY_SCOPE_AGENT)) < call) {
            if (dl.expired()) break;
            __builtin_amdgcn_s_sleep(127); __builtin_amdgcn_s_sleep(127);
        }
    }
    int it = direct ? 1 : 0;
    unsigned vz = 0;
    asm volatile("" : "+v"(vz));
    for (;; ++it) {
        const unsigned ku = (unsigned)__builtin_amdgcn_readfirstlane(tk) - qbase;
        if (ku >= NJOB && !direct) break;
        const int k = (int)ku;
        const int rec = k / 7, src = (P.rank + 1 + (k - rec * 7)) & 7;
        const u64* flag = (const u64*)(wsl + (lane == 0 ? OFF_PULLED + (unsigned)(320 + (direct ? rec - DIRR : rec)) * 64u : OFF_RECFLAG + (unsigned)(src * NREC + rec) * 8u));
        const unsigned need = rec >= NSTG - TAILR ? TAILP * call : call;
        Deadline dl(tmo, call);
        unsigned nnz = 0; bool bad = false;
        for (;;) {
            u64 f = 0;
            if (lane < 2) f = __hip_atomic_load(flag, __ATOMIC_RELAXED, __HIP_MEMORY_SCOPE_SYSTEM);
            nnz = __builtin_amdgcn_readlane((unsigned)f, 1);
            if ((unsigned)__builtin_amdgcn_readlane((unsigned)f, 0) >= need && (unsigned)__builtin_amdgcn_readlane((unsigned)(f >> 32), 1) >= call) break;
            if (dl.expired()) { bad = true; break; }
            if (it == 0) __builtin_amdgcn_s_sleep(100); else __builtin_amdgcn_s_sleep(4);
        }
        if (nnz > 4096u) { nnz = 0; bad = true; }
        f32x4 v[4][4];
        if (!bad) {
            const char* rbase = (direct ? P.ws[src] : wsl) + par + REL_O + (size_t)(src * NREC + rec) * REC_BYTES;
            const __amdgpu_buffer_rsrc_t rr = mk_rsrc(rbase, REC_BYTES);
            const u32x2 hdr = __builtin_amdgcn_raw_buffer_load_b64(rr, lane * 8, 0, AUX_SYS);
            const char* gsrc = rbase + 512 + lane * 16;
            const unsigned bytes = nnz * 4;
            for (unsigned i = 0; i < bytes; i += 1024)
                if (i + lane * 16 < bytes) __builtin_amdgcn_global_load_lds((const unsigned*)(gsrc + i), (LAS unsigned*)(buf + i), 16, 0, AUX_SYS);
            WAIT_V(0);
            unsigned off = 0;
#pragma unroll
            for (int m = 0; m < 4; ++m)
#pragma unroll
                for (int c = 0; c < 4; ++c)
#pragma unroll
                    for (int r = 0; r < 4; ++r) {
                        const int qq = (m * 4 + c) * 4 + r;
                        const unsigned lo = __builtin_amdgcn_readlane(hdr[0], qq), hi = __builtin_amdgcn_readlane(hdr[1], qq);
                        const u64 mk = ((u64)hi << 32) | lo;
                        const unsigned pos = (off + __builtin_amdgcn_mbcnt_hi(hi, __builtin_amdgcn_mbcnt_lo(lo, 0u))) & 4095u;
                        const float t = *(const LAS float*)(buf + pos * 4);
                        v[m][c][r] = __builtin_amdgcn_inverse_ballot_w64(mk) ? t : 0.f;
                        off += (unsigned)__builtin_popcountll(mk);
                    }
        } else {
#pragma unroll
            for (int m = 0; m < 4; ++m)
#pragma unroll
                for (int c = 0; c < 4; ++c) { const float qn = __builtin_nanf(""); v[m][c] = (f32x4){qn, qn, qn, qn}; }
        }
        if (lane == 0 && !direct) tk = __hip_atomic_fetch_add(q + vz, 1u, __ATOMIC_RELAXED, __HIP_MEMORY_SCOPE_AGENT);
        const int w8 = rec & 7, nt = (rec >> 3) & 7, mt = rec >> 6;
        const int row0 = (src >> 2) * 4096 + mt * 128 + (w8 >> 2) * 64, col0 = (src & 3) * 2048 + nt * 256 + (w8 & 3) * 64;
        store_dense(P.out, row0, col0, fr, fq, v);
        WAIT_L(0);
        if (direct) break;
    }
}

__global__ __launch_bounds__(512, 2) void fused_kernel(Params P) {
    extern __shared__ __attribute__((aligned(16))) unsigned char shm[];
    LAS unsigned char* lds = (LAS unsigned char*)shm;
    const int tid = threadIdx.x, lane = tid & 63, wid = __builtin_amdgcn_readfirstlane(tid >> 6);
    LAS unsigned* misc = (LAS unsigned*)(lds + LDS_MISC);
    unsigned ticket = 0, vz = 0;
    asm volatile("" : "+v"(vz));
    if (tid == 0) ticket = __hip_atomic_fetch_add((unsigned*)(P.ws[P.rank] + OFF_CTR + (blockIdx.x & 7) * 128 + vz), 1u, __ATOMIC_RELAXED, __HIP_MEMORY_SCOPE_AGENT);
    if (tid >= 64 && tid < 128) misc[tid] = 0u;
    auto ready = [&]() -> unsigned {
        asm volatile("" : "+v"(ticket));
        if (tid == 0) misc[0] = ticket / (unsigned)(GRID / 8) + 1u;
        WAIT_L(0); BAR();
        return __builtin_amdgcn_readfirstlane(misc[0]);
    };
    if (blockIdx.x < NL) link_role(P, ready(), wid, lane);
    else {
        unsigned call;
        if (blockIdx.x < NL + NG) call = convert_first(P, lds, ready, wid, lane);
        else { call = ready(); convert_role(P, lds, call, wid, lane); }
        gemm_role(P, lds, call, wid, lane);
        expand_role(P, lds, call, wid, lane);
    }
}

}

extern "C" void kernel_launch(int rank, int nranks, ncclComm_t comm,
                              hipStream_t stream,
                              void* const* d_in, const int* in_sizes, int n_in,
                              void* d_out, int out_size,
                              const int* in_dtypes, int out_dtype,
                              void* const* d_out_all,
                              void* const* d_ws_all, size_t ws_bytes) {
    using namespace gar;
    if (nranks != NRANK || (unsigned)rank >= (unsigned)NRANK || ws_bytes < (size_t)WS_USED || n_in < 2 ||
        in_sizes[0] != GM * KS || in_sizes[1] != KS * GN || out_size != GM * GN) {
        fprintf(stderr, "kernel_launch: unexpected problem shape\n");
        return;
    }
    int dev = 0, ncu = 0;
    HIP_CHECK(hipGetDevice(&dev));
    HIP_CHECK(hipDeviceGetAttribute(&ncu, hipDeviceAttributeMultiprocessorCount, dev));
    if (ncu < GRID) { fprintf(stderr, "kernel_launch: needs %d CUs, found %d\n", GRID, ncu); return; }
    Params P;
    P.x = (const float*)d_in[0]; P.w = (const float*)d_in[1]; P.out = (float*)d_out; P.rank = rank;
    for (int p = 0; p < NRANK; ++p) P.ws[p] = (char*)d_ws_all[p];
    HIP_CHECK(hipFuncSetAttribute((const void*)fused_kernel, hipFuncAttributeMaxDynamicSharedMemorySize, LDS_BYTES));
    hipLaunchKernelGGL(fused_kernel, dim3(GRID), dim3(512), LDS_BYTES, stream, P);
    HIP_CHECK(hipGetLastError());
}
```

```cpp
#include <hip/hip_runtime.h>
#include <rccl/rccl.h>
#include <cstdio>

#define HIP_CHECK(expr) do { hipError_t _e = (expr); if (_e != hipSuccess) { \
    fprintf(stderr, "HIP error %s at %s:%d\n", hipGetErrorString(_e), __FILE__, __LINE__); \
    return; } } while (0)

namespace gar {

#define LAS __attribute__((address_space(3)))
typedef _Float16 h16x8 __attribute__((ext_vector_type(8)));
typedef float f32x4 __attribute__((ext_vector_type(4)));
typedef unsigned u32x4 __attribute__((ext_vector_type(4)));
typedef unsigned u32x2 __attribute__((ext_vector_type(2)));
typedef unsigned long long u64;

constexpr int NRANK = 8;
constexpr int GM = 8192, GN = 8192, KS = 1024;
constexpr int PW = 4;
constexpr int EW = 4;
constexpr int TAILR = 192, TAILP = 4;
constexpr int DIRR = 32;
constexpr int NL = 32;
constexpr int NG = 128;
constexpr int NE = 96;
constexpr int GRID = NL + NG + NE;
constexpr int NREC = 2048;
constexpr int NSTG = NREC - DIRR;
constexpr unsigned REC_BYTES = 17408;
static_assert(GRID == 256 && DIRR <= TAILR && 7 * DIRR <= 8 * 128 && PW <= 8 && EW <= 8, "geometry");
constexpr unsigned NITEM_IN = 24 * 64;

constexpr unsigned OFF_TMO = 0, OFF_EXPQ = 128, OFF_LNKQ = 256, OFF_CTR = 1024;
constexpr unsigned OFF_CVW = 4096;
constexpr unsigned OFF_CVX = 0x180000;
constexpr unsigned OFF_PULLED = 65536;
constexpr unsigned OFF_RECFLAG = 0x140000;
constexpr unsigned OFF_DATA = 2u << 20;
constexpr unsigned CHUNK = 512u << 10;
constexpr unsigned A_BYTES = 128 * CHUNK, B_BYTES = 64 * CHUNK, XO_BYTES = 16 * CHUNK, WO_BYTES = 32 * CHUNK, O_BYTES = NRANK * NREC * REC_BYTES;
constexpr unsigned REL_B = A_BYTES, REL_XO = REL_B + B_BYTES, REL_WO = REL_XO + XO_BYTES, REL_O = REL_WO + WO_BYTES;
constexpr unsigned PAR_BYTES = REL_O + O_BYTES;
constexpr unsigned WS_USED = OFF_DATA + 2 * PAR_BYTES;
constexpr unsigned STAGE = 49152, LDS_MISC = 3 * STAGE, LDS_BYTES = LDS_MISC + 2304;
constexpr u64 DEADLINE = 150000000ull;
constexpr int AUX_SYS = 17;
constexpr int EXP_TARGET = 142;

struct Params {
    const float* x; char* wsl; int rank; int pad;
    const float* w; float* out; char* ws[NRANK];
    char* peer[7];
};

#define WAIT_V(n) asm volatile("s_waitcnt vmcnt(" #n ")" ::: "memory")
#define WAIT_L(n) asm volatile("s_waitcnt lgkmcnt(" #n ")" ::: "memory")
#define BAR() do { asm volatile("" ::: "memory"); __builtin_amdgcn_s_barrier(); asm volatile("" ::: "memory"); } while (0)

__device__ __forceinline__ __amdgpu_buffer_rsrc_t mk_rsrc(const void* p, unsigned bytes) {
    return __builtin_amdgcn_make_buffer_rsrc((void*)p, 0, bytes, 0x00020000);
}
__device__ __forceinline__ void st16_sys(char* p, u32x4 v) {
    __builtin_amdgcn_global_store_b128((__attribute__((address_space(1))) u32x4*)p, v, "");
}
__device__ __forceinline__ u32x4 cvt8(f32x4 a, f32x4 b, int sh) {
    h16x8 h;
#pragma unroll
    for (int j = 0; j < 4; ++j) { h[j] = (_Float16)__builtin_ldexpf(a[j], sh); h[4 + j] = (_Float16)__builtin_ldexpf(b[j], sh); }
    return __builtin_bit_cast(u32x4, h);
}
struct Deadline {
    unsigned* tmo; unsigned call; u64 t0; unsigned polls;
    __device__ __forceinline__ Deadline(unsigned* tmo_, unsigned call_) : tmo(tmo_), call(call_), t0(__builtin_amdgcn_s_memrealtime()), polls(1) {}
    __device__ __forceinline__ bool expired() {
        if ((polls++ & 63u) == 0u &&
            (unsigned)__builtin_amdgcn_readfirstlane(__hip_atomic_load(tmo, __ATOMIC_RELAXED, __HIP_MEMORY_SCOPE_AGENT)) == call) return true;
        if (__builtin_amdgcn_s_memrealtime() - t0 > DEADLINE) {
            __hip_atomic_store(tmo, call, __ATOMIC_RELAXED, __HIP_MEMORY_SCOPE_AGENT);
            return true;
        }
        return false;
    }
};
__device__ __forceinline__ unsigned wave_max8(unsigned v) {
    unsigned r = 0;
#pragma unroll
    for (int b = 7; b >= 0; --b) { const unsigned c = r | (1u << b); r = __builtin_amdgcn_ballot_w64(v >= c) != 0ull ? c : r; }
    return r;
}
__device__ __forceinline__ unsigned exp1_max4(unsigned m, u32x4 v) {
#pragma unroll
    for (int i = 0; i < 4; ++i) m = max(m, (v[i] + 0x00800000u) & 0x7f800000u);
    return m;
}

__device__ __forceinline__ void store_dense(float* out, int row0, int col0, int fr, int fq, const f32x4 (&v)[4][4]) {
    float* p = out + (size_t)(row0 + fr) * GN + col0 + 4 * fq;
#pragma unroll
    for (int m = 0; m < 4; ++m)
#pragma unroll
        for (int r = 0; r < 4; ++r) {
            f32x4 o = {v[m][0][r], v[m][1][r], v[m][2][r], v[m][3][r]};
            st16_sys((char*)(p + (size_t)m * 16 * GN + r * 16), __builtin_bit_cast(u32x4, o));
        }
}

__device__ __forceinline__ unsigned block_code(LAS unsigned* word, unsigned m, int lane) {
    m = wave_max8(m >> 23);
    if (lane == 0) __hip_atomic_fetch_max(word, m, __ATOMIC_RELAXED, __HIP_MEMORY_SCOPE_WORKGROUP);
    WAIT_L(0); BAR();
    return min(((unsigned)__builtin_amdgcn_readfirstlane(*word) + 3u) & ~3u, 255u);
}
__device__ __forceinline__ void load_xblock(const Params& P, f32x4 (&v)[16], int half, int st, int mt, int wid, int fr, int fq) {
    const float* s = P.x + (size_t)(half * 4096 + (st >> 3) * 2048 + mt * 128 + wid * 16 + fr) * KS + (st & 7) * 128 + fq * 8;
#pragma unroll
    for (int i = 0; i < 8; ++i) v[i] = *(const f32x4*)(s + (i >> 1) * 32 + (i & 1) * 4);
}

__device__ __forceinline__ void convert_role(const Params& P, LAS unsigned char* lds, unsigned call, int wid, int lane) {
    const int fr = lane & 15, fq = lane >> 4, rank = P.rank;
    const unsigned par = OFF_DATA + (call & 1u) * PAR_BYTES;
    LAS unsigned* red = (LAS unsigned*)(lds + LDS_MISC + 256);
    const __amdgpu_buffer_rsrc_t rl = mk_rsrc(P.wsl, WS_USED);
    int n = 0;
    __builtin_amdgcn_s_sleep(64);
    for (int i = blockIdx.x - NL - NG; i < 23 * 32; i += NE, ++n) {
        const int u = (i >> 5) + 1, sub = i & 31;
        const bool isw = u < 16 && (u & 1);
        const int st = u < 16 ? u >> 1 : u - 8;
        f32x4 v[16];
        unsigned m = 0, foff; int dst;
        if (isw) {
            const int b = sub >> 3, nt = sub & 7;
            const float* s = P.w + (size_t)(st * 128 + (wid >> 2) * 64 + fq * 8) * GN + b * 2048 + nt * 256 + (wid & 3) * 64 + 4 * (4 * (fr & 3) + (fr >> 2));
#pragma unroll
            for (int e = 0; e < 16; ++e) v[e] = *(const f32x4*)(s + (size_t)((e >> 3) * 32 + (e & 7)) * GN);
#pragma unroll
            for (int e = 0; e < 16; ++e) m = exp1_max4(m, __builtin_bit_cast(u32x4, v[e]));
            m = block_code(red + n, m, lane);
            const int sh = EXP_TARGET - (int)m;
            const unsigned off = par + (b == (rank & 3) ? REL_B + (st * 8 + rank) * CHUNK : REL_WO + (b * 8 + st) * CHUNK) + nt * 65536 + wid * 8192;
#pragma unroll
            for (int c = 0; c < 4; ++c)
#pragma unroll
                for (int kf = 0; kf < 2; ++kf) {
                    const f32x4* q = v + 8 * kf;
                    const u32x4 h = cvt8((f32x4){q[0][c], q[1][c], q[2][c], q[3][c]}, (f32x4){q[4][c], q[5][c], q[6][c], q[7][c]}, sh);
                    __builtin_amdgcn_raw_buffer_store_b128(h, rl, lane * 16 + off + c * 2048 + kf * 1024, 0, AUX_SYS);
                }
            foff = OFF_CVW + (unsigned)((st * 8 + nt) * 8 + rank) * 8u; dst = lane < 2 ? b + 4 * lane : -1;
        } else {
            const int half = sub >> 4, mt = sub & 15;
            load_xblock(P, v, half, st, mt, wid, fr, fq);
#pragma unroll
            for (int e = 0; e < 8; ++e) m = exp1_max4(m, __builtin_bit_cast(u32x4, v[e]));
            m = block_code(red + n, m, lane);
            const int sh = EXP_TARGET - (int)m;
            const unsigned off = par + (half == (rank >> 2) ? (st * 8 + rank) * CHUNK : REL_XO + st * CHUNK) + mt * 32768 + wid * 2048;
#pragma unroll
            for (int kq = 0; kq < 4; ++kq)
                __builtin_amdgcn_raw_buffer_store_b128(cvt8(v[2 * kq], v[2 * kq + 1], sh), rl, lane * 16 + off + (kq >> 1) * 16384 + (kq & 1) * 1024, 0, AUX_SYS);
            foff = OFF_CVX + (unsigned)(((st * 16 + mt) * 8 + rank) * 4 + (lane & 3)) * 8u; dst = lane < 16 ? half * 4 + (lane >> 2) : -1;
        }
        WAIT_V(0); BAR();
        if (wid == 0 && dst >= 0)
            __hip_atomic_store((u64*)(P.ws[dst & 7] + foff), ((u64)call << 32) | m, __ATOMIC_RELAXED, __HIP_MEMORY_SCOPE_SYSTEM);
    }
}

template <class F>
__device__ __forceinline__ unsigned convert_first(const Params& P, LAS unsigned char* lds, F&& ready, int wid, int lane) {
    const int gi = blockIdx.x - NL, half = gi >> 6, mt = (gi >> 2) & 15, sub = gi & 3, rank = P.rank;
    const float* s = P.x + (size_t)(half * 4096 + mt * 128 + (sub & 1) * 64 + (wid >> 1) * 16 + (lane & 15)) * KS + (sub >> 1) * 64 + (wid & 1) * 32 + (lane >> 4) * 8;
    const f32x4 v0 = *(const f32x4*)s, v1 = *(const f32x4*)(s + 4);
    char* dst = P.ws[half * 4 + (lane & 3)];
    const unsigned call = ready();
    asm volatile("" : "+v"(dst));
    LAS unsigned* red = (LAS unsigned*)(lds + LDS_MISC + 256);
    const unsigned m = block_code(red, exp1_max4(exp1_max4(0u, __builtin_bit_cast(u32x4, v0)), __builtin_bit_cast(u32x4, v1)), lane);
    const unsigned off = OFF_DATA + (call & 1u) * PAR_BYTES + (half == (rank >> 2) ? rank * CHUNK : REL_XO) + mt * 32768 + sub * 8192 + wid * 1024 + lane * 16;
    __builtin_amdgcn_raw_buffer_store_b128(cvt8(v0, v1, EXP_TARGET - (int)m), mk_rsrc(P.wsl, WS_USED), off, 0, AUX_SYS);
    WAIT_V(0); BAR();
    if (wid == 0 && lane < 4)
        __hip_atomic_store((u64*)(dst + OFF_CVX) + (mt * 8 + rank) * 4 + sub, ((u64)call << 32) | m, __ATOMIC_RELAXED, __HIP_MEMORY_SCOPE_SYSTEM);
    return call;
}

__device__ __forceinline__ void link_role(const Params& P, unsigned call, int wid, int lane) {
    if (wid >= PW) return;
    const int rank = P.rank;
    const unsigned par = OFF_DATA + (call & 1u) * PAR_BYTES;
    char* wsl = P.wsl;
    unsigned* tmo = (unsigned*)(wsl + OFF_TMO);
    const unsigned lo16 = lane * 16;
    unsigned* q = (unsigned*)(wsl + OFF_LNKQ);
    constexpr unsigned NITEM = NITEM_IN + NSTG + TAILR * (TAILP - 1);
    const unsigned qbase = (call - 1u) * NITEM - NL * PW;
    unsigned tk = qbase + wid * NL + blockIdx.x, vz = 0;
    asm volatile("" : "+v"(vz));
    const int lsrc = (rank + 1 + lane) & 7;
    int prev = -1;
    auto publish = [&]() {
        if (prev >= 0 && lane == 0) __hip_atomic_fetch_add((unsigned*)(wsl + OFF_PULLED + (unsigned)prev * 64u), 1u, __ATOMIC_RELAXED, __HIP_MEMORY_SCOPE_AGENT);
        prev = -1;
    };
    for (;;) {
        const unsigned it = (unsigned)__builtin_amdgcn_readfirstlane(tk) - qbase;
        if (it >= NITEM) break;
        if (lane == 0) tk = __hip_atomic_fetch_add(q + vz, 1u, __ATOMIC_RELAXED, __HIP_MEMORY_SCOPE_AGENT);
        const bool isrec = it >= NITEM_IN;
        int id; unsigned foff, soff, doff, o0 = 0, ostep = 1024;
        if (isrec) {
            unsigned rec = it - NITEM_IN;
            if (rec >= NSTG - TAILR) { const unsigned t = rec - (NSTG - TAILR); rec = NSTG - TAILR + t / TAILP; o0 = (t % TAILP) * ostep; ostep *= TAILP; }
            id = 320 + (int)rec; foff = OFF_RECFLAG + (unsigned)(lsrc * NREC + rec) * 8u;
            soff = doff = par + REL_O + (unsigned)(lsrc * NREC + rec) * REC_BYTES;
        } else {
            const int u = (int)(it >> 6), sub = it & 63;
            const bool isw = u < 16 && (u & 1);
            const int st = u < 16 ? u >> 1 : u - 8;
            id = isw ? 256 + st * 8 + (sub >> 3) : st * 16 + (sub >> 2);
            foff = isw ? OFF_CVW + (unsigned)((id - 256) * 8 + lsrc) * 8u : OFF_CVX + (unsigned)((id * 8 + lsrc) * 4 + (sub & 3)) * 8u;
            doff = par + (isw ? REL_B : 0u) + (unsigned)(st * 8 + lsrc) * CHUNK + sub * 8192;
            const bool same = isw ? (lsrc & 3) == (rank & 3) : (lsrc >> 2) == (rank >> 2);
            soff = same ? doff : par + (isw ? REL_WO + (unsigned)((rank & 3) * 8 + st) * CHUNK : REL_XO + (unsigned)st * CHUNK) + sub * 8192;
        }
        Deadline dl(tmo, call);
        u64 f = 0; bool bad = false;
        for (;;) {
            if (lane < 7) f = __hip_atomic_load((const u64*)(wsl + foff), __ATOMIC_RELAXED, __HIP_MEMORY_SCOPE_SYSTEM);
            const u64 anylate = __builtin_amdgcn_ballot_w64(lane < 7 && (unsigned)(f >> 32) < call);
            WAIT_V(0);
            publish();
            if (anylate == 0ull) break;
            if (dl.expired()) { bad = true; break; }
            __builtin_amdgcn_s_sleep(8);
        }
        if (bad || __builtin_amdgcn_ballot_w64(lane < 7 && (unsigned)f > 4096u) != 0ull) continue;
        const unsigned lbytes = isrec ? 512u + (unsigned)f * 4u : 8192u;
        __amdgpu_buffer_rsrc_t rp[7], rl[7];
        unsigned maxb = 0;
#pragma unroll
        for (int k = 0; k < 7; ++k) {
            const unsigned bytes = __builtin_amdgcn_readlane(lbytes, k);
            rp[k] = mk_rsrc(P.peer[k] + (unsigned)__builtin_amdgcn_readlane(soff, k), bytes);
            rl[k] = mk_rsrc(wsl + (unsigned)__builtin_amdgcn_readlane(doff, k), bytes);
            maxb = max(maxb, bytes);
        }
        for (unsigned o = o0; o < maxb; o += ostep) {
            u32x4 d[7];
#pragma unroll
            for (int k = 0; k < 7; ++k) d[k] = __builtin_amdgcn_raw_buffer_load_b128(rp[k], lo16 + o, 0, AUX_SYS);
#pragma unroll
            for (int k = 0; k < 7; ++k) __builtin_amdgcn_raw_buffer_store_b128(d[k], rl[k], lo16 + o, 0, AUX_SYS);
        }
        prev = id;
    }
    WAIT_V(0);
    publish();
}

__device__ __forceinline__ void wait_inputs(const Params& P, LAS unsigned char* lds, int st, int mt, int nt, unsigned call, int lane) {
    char* wsl = P.wsl;
    const int bx = st * 16 + mt, bw = 256 + (st & 7) * 8 + nt;
    const unsigned off = lane < 32 ? OFF_CVX + (unsigned)(bx * 32 + lane) * 8u : lane < 40 ? OFF_CVW + (unsigned)((bw - 256) * 8 + (lane & 7)) * 8u
                                   : OFF_PULLED + (unsigned)(lane == 40 ? bx : bw) * 64u;
    const unsigned need = lane == 40 ? 4u * call : 8u * call;
    Deadline dl((unsigned*)(wsl + OFF_TMO), call);
    u64 f = 0;
    for (;;) {
        if (lane < 42) f = __hip_atomic_load((const u64*)(wsl + off), __ATOMIC_RELAXED, __HIP_MEMORY_SCOPE_SYSTEM);
        const bool late = lane < 40 ? (unsigned)(f >> 32) < call : lane < 42 && (unsigned)f < need;
        if (__builtin_amdgcn_ballot_w64(late) == 0ull) break;
        if (dl.expired()) break;
        if (st == 8) __builtin_amdgcn_s_sleep(127); else __builtin_amdgcn_s_sleep(32);
    }
    __builtin_amdgcn_fence(__ATOMIC_ACQUIRE, "");
    LAS int* tab = (LAS int*)(lds + LDS_MISC + 1280);
    const int cw = __shfl((int)((unsigned)f & 0xffu), 32 + ((lane >> 2) & 7));
    if (lane < 32) tab[(st & 7) * 32 + lane] = (int)((unsigned)f & 0xffu) + cw;
    WAIT_V(0); WAIT_L(0);
}

__device__ __forceinline__ void gemm_role(const Params& P, LAS unsigned char* lds, unsigned call, int wid, int lane) {
    const int gi = blockIdx.x - NL, xcd = gi & 7, ix = (gi & 127) >> 3;
    const int wr = wid >> 2, wc = wid & 3, fr = lane & 15, fq = lane >> 4;
    const unsigned par = OFF_DATA + (call & 1u) * PAR_BYTES;
    char* wsl = P.wsl;
    const int nt = (xcd & 1) * 4 + (ix & 3);
    const unsigned aoff = (unsigned)(wr * 8192 + lane * 16), boff = (unsigned)(16384 + wc * 8192 + lane * 16);
    const LAS int* tab = (const LAS int*)(lds + LDS_MISC + 1280) + wr;
    const int ntile = gi < 32 ? 2 : 1;
    for (int ti = 0; ti < ntile; ++ti) {
        const int mt = gi >= NG ? 16 + (xcd >> 1) * 3 + (ix >> 2) : ti == 0 ? (xcd >> 1) * 4 + (ix >> 2) : 28 + (xcd >> 1);
        const int job = mt >> 4;
        const char* ga = wsl + par + (size_t)job * (64 * CHUNK) + (mt & 15) * 32768 + wid * 2048 + lane * 16;
        const char* gb = wsl + par + REL_B + nt * 65536 + wid * 4096 + lane * 16;
#define ISSUE(kt_, sb_) do { const char* _a = ga + (((kt_) >> 1) * CHUNK + ((kt_) & 1) * 16384); const char* _b = gb + (((kt_) >> 1) * CHUNK + ((kt_) & 1) * 32768); \
        LAS unsigned char* _la = lds + (sb_) + wid * 2048; LAS unsigned char* _lb = lds + (sb_) + 16384 + wid * 4096; \
        __builtin_amdgcn_global_load_lds((const unsigned*)_a, (LAS unsigned*)_la, 16, 0, 0); \
        __builtin_amdgcn_global_load_lds((const unsigned*)_a, (LAS unsigned*)_la, 16, 1024, 0); \
        __builtin_amdgcn_global_load_lds((const unsigned*)_b, (LAS unsigned*)_lb, 16, 0, 0); \
        __builtin_amdgcn_global_load_lds((const unsigned*)_b, (LAS unsigned*)_lb, 16, 1024, 0); \
        __builtin_amdgcn_global_load_lds((const unsigned*)_b, (LAS unsigned*)_lb, 16, 2048, 0); \
        __builtin_amdgcn_global_load_lds((const unsigned*)_b, (LAS unsigned*)_lb, 16, 3072, 0); } while (0)
#define LDF(A_, B_, sb_, kf_) do { _Pragma("unroll") for (int i = 0; i < 4; ++i) { \
        B_[i] = *(const LAS h16x8*)(lds + (sb_) + boff + i * 2048 + (kf_) * 1024); \
        A_[i] = *(const LAS h16x8*)(lds + (sb_) + aoff + i * 2048 + (kf_) * 1024); } } while (0)
#define MMA(A_, B_) do { __builtin_amdgcn_s_setprio(1); _Pragma("unroll") for (int m = 0; m < 4; ++m) _Pragma("unroll") for (int c = 0; c < 4; ++c) \
        acc[m][c] = __builtin_amdgcn_mfma_f32_16x16x32_f16(B_[c], A_[m], acc[m][c], 0, 0, 0); __builtin_amdgcn_s_setprio(0); } while (0)
        f32x4 acc[4][4], tot[4][4];
#pragma unroll
        for (int m = 0; m < 4; ++m)
#pragma unroll
            for (int c = 0; c < 4; ++c) acc[m][c] = tot[m][c] = (f32x4){0.f, 0.f, 0.f, 0.f};
        h16x8 A0[4], B0[4], A1[4], B1[4];
        if (wid == 0) wait_inputs(P, lds, job * 8, mt & 15, nt, call, lane);
        BAR();
        ISSUE(0, 0); ISSUE(1, STAGE); ISSUE(2, 2 * STAGE);
        WAIT_V(12); BAR();
        LDF(A0, B0, 0u, 0);
        int tnext = tab[0];
        int tcur = __builtin_amdgcn_readfirstlane(tnext);
        int tref = tcur;
        unsigned sb = 0;
        for (int kt = 0; kt < 128; ++kt) {
            const unsigned sn = sb == 2 * STAGE ? 0u : sb + STAGE;
            __builtin_amdgcn_s_waitcnt(0xC07F);
            LDF(A1, B1, sb, 1);
            const int tn = __builtin_amdgcn_readfirstlane(tnext);
            if (tn != tcur) {
                const bool nearby = abs(tn - tref) <= 30;
                const int ekeep = nearby ? tcur - tn : -2000, ebank = nearby ? -2000 : tcur - 2 * EXP_TARGET;
#pragma unroll
                for (int m = 0; m < 4; ++m)
#pragma unroll
                    for (int c = 0; c < 4; ++c)
#pragma unroll
                        for (int r = 0; r < 4; ++r) {
                            tot[m][c][r] += __builtin_ldexpf(acc[m][c][r], ebank);
                            acc[m][c][r] = __builtin_ldexpf(acc[m][c][r], ekeep);
                        }
                tref = nearby ? tref : tn;
                tcur = tn;
            }
            MMA(A0, B0);
            if (((kt + 3) & 15) == 0 && kt + 3 < 128 && wid == 0) wait_inputs(P, lds, job * 8 + ((kt + 3) >> 4), mt & 15, nt, call, lane);
            WAIT_V(6); __builtin_amdgcn_s_waitcnt(0xC07F); BAR();
            ISSUE((kt + 3) & 127, sb);
            LDF(A0, B0, sn, 0);
            tnext = tab[min(kt + 1, 127) * 2];
            MMA(A1, B1);
            sb = sn;
        }
        WAIT_V(0); WAIT_L(0); BAR();
        unsigned* tmo = (unsigned*)(wsl + OFF_TMO);
        const bool bad = (unsigned)__builtin_amdgcn_readfirstlane(__hip_atomic_load(tmo, __ATOMIC_RELAXED, __HIP_MEMORY_SCOPE_AGENT)) == call;
        LAS float* cv = (LAS float*)(lds + wid * 16384);
        unsigned mlo = 0, mhi = 0, off = 0;
        const int unsc = tcur - 2 * EXP_TARGET;
#pragma unroll
        for (int m = 0; m < 4; ++m)
#pragma unroll
            for (int c = 0; c < 4; ++c)
#pragma unroll
                for (int r = 0; r < 4; ++r) {
                    float v = tot[m][c][r] + __builtin_ldexpf(acc[m][c][r], unsc);
                    v = v <= 0.f ? 0.f : v;
                    if (bad) v = __builtin_nanf("");
                    acc[m][c][r] = v;
                    const bool nz = __float_as_uint(v) != 0u;
                    const u64 mk = __builtin_amdgcn_ballot_w64(nz);
                    const unsigned lo = (unsigned)mk, hi = (unsigned)(mk >> 32);
                    const unsigned pos = off + __builtin_amdgcn_mbcnt_hi(hi, __builtin_amdgcn_mbcnt_lo(lo, 0u));
                    if (nz) cv[pos] = v;
                    const int q = (m * 4 + c) * 4 + r;
                    mlo = lane == q ? lo : mlo; mhi = lane == q ? hi : mhi;
                    off += (unsigned)__builtin_popcountll(mk);
                }
        const int rec = (mt * 8 + nt) * 8 + wid;
        const __amdgpu_buffer_rsrc_t rl = mk_rsrc(wsl, WS_USED);
        const unsigned ro = par + REL_O + (unsigned)(P.rank * NREC + rec) * REC_BYTES;
        __builtin_amdgcn_raw_buffer_store_b64((u32x2){mlo, mhi}, rl, lane * 8, ro, AUX_SYS);
        WAIT_L(0);
        for (unsigned i = 0; i * 256 < off; ++i) {
            const u32x4 d = *(const LAS u32x4*)((LAS unsigned char*)cv + i * 1024 + lane * 16);
            __builtin_amdgcn_raw_buffer_store_b128(d, rl, lane * 16 + ro + 512 + i * 1024, 0, AUX_SYS);
        }
        LAS unsigned* nz8 = (LAS unsigned*)(lds + LDS_MISC + 1024);
        if (lane == 0) nz8[wid] = off;
        WAIT_V(0); WAIT_L(0); BAR();
        if (wid == 0 && (lane >> 3) != P.rank)
            __hip_atomic_store((u64*)(P.ws[lane >> 3] + OFF_RECFLAG) + (unsigned)(P.rank * NREC + rec + (lane & 7)), ((u64)call << 32) | nz8[lane & 7], __ATOMIC_RELAXED, __HIP_MEMORY_SCOPE_SYSTEM);
        store_dense(P.out, (P.rank >> 2) * 4096 + mt * 128 + wr * 64, (P.rank & 3) * 2048 + nt * 256 + wc * 64, fr, fq, acc);
        WAIT_L(0); BAR();
#undef ISSUE
#undef LDF
#undef MMA
    }
}

__device__ __forceinline__ void expand_role(const Params& P, LAS unsigned char* lds, unsigned call, int wid, int lane) {
    const int fr = lane & 15, fq = lane >> 4;
    constexpr unsigned NJOB = 7 * NSTG, E = NE * EW;
    const int gi = blockIdx.x - NL;
    const bool direct = gi < 32 || gi >= NG;
    const unsigned didx = (unsigned)(wid * 128 + (gi < 32 ? gi : gi - 96));
    if (direct ? didx >= 7u * DIRR : wid >= EW) return;
    const unsigned par = OFF_DATA + (call & 1u) * PAR_BYTES;
    char* wsl = P.wsl;
    unsigned* tmo = (unsigned*)(wsl + OFF_TMO);
    LAS unsigned char* buf = lds + wid * 16384;
    unsigned* q = (unsigned*)(wsl + OFF_EXPQ);
    const unsigned qbase = (call - 1u) * (NJOB + E);
    unsigned tk = qbase + NJOB + didx;
    if (!direct) { if (lane == 0) tk = __hip_atomic_fetch_add(q, 1u, __ATOMIC_RELAXED, __HIP_MEMORY_SCOPE_AGENT); }
    else {
        Deadline dl(tmo, call);
        while ((unsigned)__builtin_amdgcn_readfirstlane(__hip_atomic_load((unsigned*)(wsl + OFF_PULLED + (unsigned)(320 + NSTG - TAILR - 32) * 64u), __ATOMIC_RELAXED, __HIP_MEMORY_SCOPE_AGENT)) < call) {
            if (dl.expired()) break;
            __builtin_amdgcn_s_sleep(127); __builtin_amdgcn_s_sleep(127);
        }
    }
    int it = direct ? 1 : 0;
    unsigned vz = 0;
    asm volatile("" : "+v"(vz));
    for (;; ++it) {
        const unsigned ku = (unsigned)__builtin_amdgcn_readfirstlane(tk) - qbase;
        if (ku >= NJOB && !direct) break;
        const int k = (int)ku;
        const int rec = k / 7, src = (P.rank + 1 + (k - rec * 7)) & 7;
        const u64* flag = (const u64*)(wsl + (lane == 0 ? OFF_PULLED + (unsigned)(320 + (direct ? rec - DIRR : rec)) * 64u : OFF_RECFLAG + (unsigned)(src * NREC + rec) * 8u));
        const unsigned need = rec >= NSTG - TAILR ? TAILP * call : call;
        Deadline dl(tmo, call);
        unsigned nnz = 0; bool bad = false;
        for (;;) {
            u64 f = 0;
            if (lane < 2) f = __hip_atomic_load(flag, __ATOMIC_RELAXED, __HIP_MEMORY_SCOPE_SYSTEM);
            nnz = __builtin_amdgcn_readlane((unsigned)f, 1);
            if ((unsigned)__builtin_amdgcn_readlane((unsigned)f, 0) >= need && (unsigned)__builtin_amdgcn_readlane((unsigned)(f >> 32), 1) >= call) break;
            if (dl.expired()) { bad = true; break; }
            if (it == 0) __builtin_amdgcn_s_sleep(100); else __builtin_amdgcn_s_sleep(4);
        }
        if (nnz > 4096u) { nnz = 0; bad = true; }
        f32x4 v[4][4];
        if (!bad) {
            const char* rbase = (direct ? P.ws[src] : wsl) + par + REL_O + (size_t)(src * NREC + rec) * REC_BYTES;
            const __amdgpu_buffer_rsrc_t rr = mk_rsrc(rbase, REC_BYTES);
            const u32x2 hdr = __builtin_amdgcn_raw_buffer_load_b64(rr, lane * 8, 0, AUX_SYS);
            const char* gsrc = rbase + 512 + lane * 16;
            const unsigned bytes = nnz * 4;
            for (unsigned i = 0; i < bytes; i += 1024)
                if (i + lane * 16 < bytes) __builtin_amdgcn_global_load_lds((const unsigned*)(gsrc + i), (LAS unsigned*)(buf + i), 16, 0, AUX_SYS);
            WAIT_V(0);
            unsigned off = 0;
#pragma unroll
            for (int m = 0; m < 4; ++m)
#pragma unroll
                for (int c = 0; c < 4; ++c)
#pragma unroll
                    for (int r = 0; r < 4; ++r) {
                        const int qq = (m * 4 + c) * 4 + r;
                        const unsigned lo = __builtin_amdgcn_readlane(hdr[0], qq), hi = __builtin_amdgcn_readlane(hdr[1], qq);
                        const u64 mk = ((u64)hi << 32) | lo;
                        const unsigned pos = (off + __builtin_amdgcn_mbcnt_hi(hi, __builtin_amdgcn_mbcnt_lo(lo, 0u))) & 4095u;
                        const float t = *(const LAS float*)(buf + pos * 4);
                        v[m][c][r] = __builtin_amdgcn_inverse_ballot_w64(mk) ? t : 0.f;
                        off += (unsigned)__builtin_popcountll(mk);
                    }
        } else {
#pragma unroll
            for (int m = 0; m < 4; ++m)
#pragma unroll
                for (int c = 0; c < 4; ++c) { const float qn = __builtin_nanf(""); v[m][c] = (f32x4){qn, qn, qn, qn}; }
        }
        if (lane == 0 && !direct) tk = __hip_atomic_fetch_add(q + vz, 1u, __ATOMIC_RELAXED, __HIP_MEMORY_SCOPE_AGENT);
        const int w8 = rec & 7, nt = (rec >> 3) & 7, mt = rec >> 6;
        const int row0 = (src >> 2) * 4096 + mt * 128 + (w8 >> 2) * 64, col0 = (src & 3) * 2048 + nt * 256 + (w8 & 3) * 64;
        store_dense(P.out, row0, col0, fr, fq, v);
        WAIT_L(0);
        if (direct) break;
    }
}

__global__ __launch_bounds__(512, 2) void fused_kernel(Params P) {
    extern __shared__ __attribute__((aligned(16))) unsigned char shm[];
    LAS unsigned char* lds = (LAS unsigned char*)shm;
    const int tid = threadIdx.x, lane = tid & 63, wid = __builtin_amdgcn_readfirstlane(tid >> 6);
    LAS unsigned* misc = (LAS unsigned*)(lds + LDS_MISC);
    unsigned ticket = 0, vz = 0;
    asm volatile("" : "+v"(vz));
    if (tid == 0) ticket = __hip_atomic_fetch_add((unsigned*)(P.wsl + OFF_CTR + (blockIdx.x & 7) * 128 + vz), 1u, __ATOMIC_RELAXED, __HIP_MEMORY_SCOPE_AGENT);
    if (tid >= 64 && tid < 128) misc[tid] = 0u;
    auto ready = [&]() -> unsigned {
        asm volatile("" : "+v"(ticket));
        if (tid == 0) misc[0] = ticket / (unsigned)(GRID / 8) + 1u;
        WAIT_L(0); BAR();
        return __builtin_amdgcn_readfirstlane(misc[0]);
    };
    if (blockIdx.x < NL) link_role(P, ready(), wid, lane);
    else {
        unsigned call;
        if (blockIdx.x < NL + NG) call = convert_first(P, lds, ready, wid, lane);
        else { call = ready(); convert_role(P, lds, call, wid, lane); }
        gemm_role(P, lds, call, wid, lane);
        expand_role(P, lds, call, wid, lane);
    }
}

}

extern "C" void kernel_launch(int rank, int nranks, ncclComm_t comm,
                              hipStream_t stream,
                              void* const* d_in, const int* in_sizes, int n_in,
                              void* d_out, int out_size,
                              const int* in_dtypes, int out_dtype,
                              void* const* d_out_all,
                              void* const* d_ws_all, size_t ws_bytes) {
    using namespace gar;
    if (nranks != NRANK || (unsigned)rank >= (unsigned)NRANK || ws_bytes < (size_t)WS_USED || n_in < 2 ||
        in_sizes[0] != GM * KS || in_sizes[1] != KS * GN || out_size != GM * GN) {
        fprintf(stderr, "kernel_launch: unexpected problem shape\n");
        return;
    }
    int dev = 0, ncu = 0;
    HIP_CHECK(hipGetDevice(&dev));
    HIP_CHECK(hipDeviceGetAttribute(&ncu, hipDeviceAttributeMultiprocessorCount, dev));
    if (ncu < GRID) { fprintf(stderr, "kernel_launch: needs %d CUs, found %d\n", GRID, ncu); return; }
    Params P;
    P.x = (const float*)d_in[0]; P.w = (const float*)d_in[1]; P.out = (float*)d_out; P.rank = rank;
    for (int p = 0; p < NRANK; ++p) P.ws[p] = (char*)d_ws_all[p];
    P.wsl = P.ws[rank]; P.pad = 0;
    for (int k = 0; k < 7; ++k) P.peer[k] = P.ws[(rank + 1 + k) & 7];
    HIP_CHECK(hipFuncSetAttribute((const void*)fused_kernel, hipFuncAttributeMaxDynamicSharedMemorySize, LDS_BYTES));
    hipLaunchKernelGGL(fused_kernel, dim3(GRID), dim3(512), LDS_BYTES, stream, P);
    HIP_CHECK(hipGetLastError());
}
```

```cpp
#include <hip/hip_runtime.h>
#include <rccl/rccl.h>
#include <cstdio>

#define HIP_CHECK(expr) do { hipError_t _e = (expr); if (_e != hipSuccess) { \
    fprintf(stderr, "HIP error %s at %s:%d\n", hipGetErrorString(_e), __FILE__, __LINE__); \
    return; } } while (0)

namespace gar {

#define LAS __attribute__((address_space(3)))
typedef _Float16 h16x8 __attribute__((ext_vector_type(8)));
typedef float f32x4 __attribute__((ext_vector_type(4)));
typedef unsigned u32x4 __attribute__((ext_vector_type(4)));
typedef unsigned u32x2 __attribute__((ext_vector_type(2)));
typedef unsigned long long u64;

constexpr int NRANK = 8;
constexpr int GM = 8192, GN = 8192, KS = 1024;
constexpr int PW = 4;
constexpr int EW = 4;
constexpr int TAILR = 192, TAILP = 4;
constexpr int DIRR = 32;
constexpr int NL = 32;
constexpr int NG = 128;
constexpr int NE = 96;
constexpr int GRID = NL + NG + NE;
constexpr int NREC = 2048;
constexpr int NSTG = NREC - DIRR;
constexpr unsigned REC_BYTES = 17408;
static_assert(GRID == 256 && DIRR <= TAILR && 7 * DIRR <= 8 * 128 && PW <= 8 && EW <= 8, "geometry");
constexpr unsigned NITEM_IN = 24 * 64;

constexpr unsigned OFF_TMO = 0, OFF_EXPQ = 128, OFF_LNKQ = 256, OFF_CTR = 1024;
constexpr unsigned OFF_CVW = 4096;
constexpr unsigned OFF_CVX = 0x180000;
constexpr unsigned OFF_PULLED = 65536;
constexpr unsigned OFF_RECFLAG = 0x140000;
constexpr unsigned OFF_DATA = 2u << 20;
constexpr unsigned CHUNK = 512u << 10;
constexpr unsigned A_BYTES = 128 * CHUNK, B_BYTES = 64 * CHUNK, XO_BYTES = 16 * CHUNK, WO_BYTES = 32 * CHUNK, O_BYTES = NRANK * NREC * REC_BYTES;
constexpr unsigned REL_B = A_BYTES, REL_XO = REL_B + B_BYTES, REL_WO = REL_XO + XO_BYTES, REL_O = REL_WO + WO_BYTES;
constexpr unsigned PAR_BYTES = REL_O + O_BYTES;
constexpr unsigned WS_USED = OFF_DATA + 2 * PAR_BYTES;
constexpr unsigned STAGE = 49152, LDS_MISC = 3 * STAGE, LDS_BYTES = LDS_MISC + 2304;
constexpr u64 DEADLINE = 150000000ull;
constexpr int AUX_SYS = 17;
constexpr int EXP_TARGET = 142;

struct Params {
    const float* x; char* wsl; int rank; int pad;
    const float* w; float* out; char* ws[NRANK];
    char* peer[7];
};

#define WAIT_V(n) asm volatile("s_waitcnt vmcnt(" #n ")" ::: "memory")
#define WAIT_L(n) asm volatile("s_waitcnt lgkmcnt(" #n ")" ::: "memory")
#define BAR() do { asm volatile("" ::: "memory"); __builtin_amdgcn_s_barrier(); asm volatile("" ::: "memory"); } while (0)

__device__ __forceinline__ __amdgpu_buffer_rsrc_t mk_rsrc(const void* p, unsigned bytes) {
    return __builtin_amdgcn_make_buffer_rsrc((void*)p, 0, bytes, 0x00020000);
}
__device__ __forceinline__ void st16_sys(char* p, u32x4 v) {
    __builtin_amdgcn_global_store_b128((__attribute__((address_space(1))) u32x4*)p, v, "");
}
__device__ __forceinline__ u32x4 cvt8(f32x4 a, f32x4 b, int sh) {
    h16x8 h;
#pragma unroll
    for (int j = 0; j < 4; ++j) { h[j] = (_Float16)__builtin_ldexpf(a[j], sh); h[4 + j] = (_Float16)__builtin_ldexpf(b[j], sh); }
    return __builtin_bit_cast(u32x4, h);
}
__device__ __forceinline__ unsigned rec_groups(unsigned nnz, bool direct) { return direct ? 0u : nnz >> 6; }
__device__ __forceinline__ unsigned rec_hdr_off(unsigned nnz) { return (nnz * 4u + 7u) & ~7u; }
struct Deadline {
    unsigned* tmo; unsigned call; u64 t0; unsigned polls;
    __device__ __forceinline__ Deadline(unsigned* tmo_, unsigned call_) : tmo(tmo_), call(call_), t0(__builtin_amdgcn_s_memrealtime()), polls(1) {}
    __device__ __forceinline__ bool expired() {
        if ((polls++ & 63u) == 0u &&
            (unsigned)__builtin_amdgcn_readfirstlane(__hip_atomic_load(tmo, __ATOMIC_RELAXED, __HIP_MEMORY_SCOPE_AGENT)) == call) return true;
        if (__builtin_amdgcn_s_memrealtime() - t0 > DEADLINE) {
            __hip_atomic_store(tmo, call, __ATOMIC_RELAXED, __HIP_MEMORY_SCOPE_AGENT);
            return true;
        }
        return false;
    }
};
__device__ __forceinline__ unsigned wave_max8(unsigned v) {
    unsigned r = 0;
#pragma unroll
    for (int b = 7; b >= 0; --b) { const unsigned c = r | (1u << b); r = __builtin_amdgcn_ballot_w64(v >= c) != 0ull ? c : r; }
    return r;
}
__device__ __forceinline__ unsigned exp1_max4(unsigned m, u32x4 v) {
#pragma unroll
    for (int i = 0; i < 4; ++i) m = max(m, (v[i] + 0x00800000u) & 0x7f800000u);
    return m;
}

__device__ __forceinline__ void store_dense(float* out, int row0, int col0, int fr, int fq, const f32x4 (&v)[4][4]) {
    float* p = out + (size_t)(row0 + fr) * GN + col0 + 4 * fq;
#pragma unroll
    for (int m = 0; m < 4; ++m)
#pragma unroll
        for (int r = 0; r < 4; ++r) {
            f32x4 o = {v[m][0][r], v[m][1][r], v[m][2][r], v[m][3][r]};
            st16_sys((char*)(p + (size_t)m * 16 * GN + r * 16), __builtin_bit_cast(u32x4, o));
        }
}

__device__ __forceinline__ unsigned block_code(LAS unsigned* word, unsigned m, int lane) {
    m = wave_max8(m >> 23);
    if (lane == 0) __hip_atomic_fetch_max(word, m, __ATOMIC_RELAXED, __HIP_MEMORY_SCOPE_WORKGROUP);
    WAIT_L(0); BAR();
    return min(((unsigned)__builtin_amdgcn_readfirstlane(*word) + 3u) & ~3u, 255u);
}
__device__ __forceinline__ void load_xblock(const Params& P, f32x4 (&v)[16], int half, int st, int mt, int wid, int fr, int fq) {
    const float* s = P.x + (size_t)(half * 4096 + (st >> 3) * 2048 + mt * 128 + wid * 16 + fr) * KS + (st & 7) * 128 + fq * 8;
#pragma unroll
    for (int i = 0; i < 8; ++i) v[i] = *(const f32x4*)(s + (i >> 1) * 32 + (i & 1) * 4);
}

__device__ __forceinline__ void convert_role(const Params& P, LAS unsigned char* lds, unsigned call, int wid, int lane) {
    const int fr = lane & 15, fq = lane >> 4, rank = P.rank;
    const unsigned par = OFF_DATA + (call & 1u) * PAR_BYTES;
    LAS unsigned* red = (LAS unsigned*)(lds + LDS_MISC + 256);
    const __amdgpu_buffer_rsrc_t rl = mk_rsrc(P.wsl, WS_USED);
    int n = 0;
    __builtin_amdgcn_s_sleep(64);
    for (int i = blockIdx.x - NL - NG; i < 23 * 32; i += NE, ++n) {
        const int u = (i >> 5) + 1, sub = i & 31;
        const bool isw = u < 16 && (u & 1);
        const int st = u < 16 ? u >> 1 : u - 8;
        f32x4 v[16];
        unsigned m = 0, foff; int dst;
        if (isw) {
            const int b = sub >> 3, nt = sub & 7;
            const float* s = P.w + (size_t)(st * 128 + (wid >> 2) * 64 + fq * 8) * GN + b * 2048 + nt * 256 + (wid & 3) * 64 + 4 * (4 * (fr & 3) + (fr >> 2));
#pragma unroll
            for (int e = 0; e < 16; ++e) v[e] = *(const f32x4*)(s + (size_t)((e >> 3) * 32 + (e & 7)) * GN);
#pragma unroll
            for (int e = 0; e < 16; ++e) m = exp1_max4(m, __builtin_bit_cast(u32x4, v[e]));
            m = block_code(red + n, m, lane);
            const int sh = EXP_TARGET - (int)m;
            const unsigned off = par + (b == (rank & 3) ? REL_B + (st * 8 + rank) * CHUNK : REL_WO + (b * 8 + st) * CHUNK) + nt * 65536 + wid * 8192;
#pragma unroll
            for (int c = 0; c < 4; ++c)
#pragma unroll
                for (int kf = 0; kf < 2; ++kf) {
                    const f32x4* q = v + 8 * kf;
                    const u32x4 h = cvt8((f32x4){q[0][c], q[1][c], q[2][c], q[3][c]}, (f32x4){q[4][c], q[5][c], q[6][c], q[7][c]}, sh);
                    __builtin_amdgcn_raw_buffer_store_b128(h, rl, lane * 16 + off + c * 2048 + kf * 1024, 0, AUX_SYS);
                }
            foff = OFF_CVW + (unsigned)((st * 8 + nt) * 8 + rank) * 8u; dst = lane < 2 ? b + 4 * lane : -1;
        } else {
            const int half = sub >> 4, mt = sub & 15;
            load_xblock(P, v, half, st, mt, wid, fr, fq);
#pragma unroll
            for (int e = 0; e < 8; ++e) m = exp1_max4(m, __builtin_bit_cast(u32x4, v[e]));
            m = block_code(red + n, m, lane);
            const int sh = EXP_TARGET - (int)m;
            const unsigned off = par + (half == (rank >> 2) ? (st * 8 + rank) * CHUNK : REL_XO + st * CHUNK) + mt * 32768 + wid * 2048;
#pragma unroll
            for (int kq = 0; kq < 4; ++kq)
                __builtin_amdgcn_raw_buffer_store_b128(cvt8(v[2 * kq], v[2 * kq + 1], sh), rl, lane * 16 + off + (kq >> 1) * 16384 + (kq & 1) * 1024, 0, AUX_SYS);
            foff = OFF_CVX + (unsigned)(((st * 16 + mt) * 8 + rank) * 4 + (lane & 3)) * 8u; dst = lane < 16 ? half * 4 + (lane >> 2) : -1;
        }
        WAIT_V(0); BAR();
        if (wid == 0 && dst >= 0)
            __hip_atomic_store((u64*)(P.ws[dst & 7] + foff), ((u64)call << 32) | m, __ATOMIC_RELAXED, __HIP_MEMORY_SCOPE_SYSTEM);
    }
}

template <class F>
__device__ __forceinline__ unsigned convert_first(const Params& P, LAS unsigned char* lds, F&& ready, int wid, int lane) {
    const int gi = blockIdx.x - NL, half = gi >> 6, mt = (gi >> 2) & 15, sub = gi & 3, rank = P.rank;
    const float* s = P.x + (size_t)(half * 4096 + mt * 128 + (sub & 1) * 64 + (wid >> 1) * 16 + (lane & 15)) * KS + (sub >> 1) * 64 + (wid & 1) * 32 + (lane >> 4) * 8;
    const f32x4 v0 = *(const f32x4*)s, v1 = *(const f32x4*)(s + 4);
    char* dst = P.ws[half * 4 + (lane & 3)];
    const unsigned call = ready();
    asm volatile("" : "+v"(dst));
    LAS unsigned* red = (LAS unsigned*)(lds + LDS_MISC + 256);
    const unsigned m = block_code(red, exp1_max4(exp1_max4(0u, __builtin_bit_cast(u32x4, v0)), __builtin_bit_cast(u32x4, v1)), lane);
    const unsigned off = OFF_DATA + (call & 1u) * PAR_BYTES + (half == (rank >> 2) ? rank * CHUNK : REL_XO) + mt * 32768 + sub * 8192 + wid * 1024 + lane * 16;
    __builtin_amdgcn_raw_buffer_store_b128(cvt8(v0, v1, EXP_TARGET - (int)m), mk_rsrc(P.wsl, WS_USED), off, 0, AUX_SYS);
    WAIT_V(0); BAR();
    if (wid == 0 && lane < 4)
        __hip_atomic_store((u64*)(dst + OFF_CVX) + (mt * 8 + rank) * 4 + sub, ((u64)call << 32) | m, __ATOMIC_RELAXED, __HIP_MEMORY_SCOPE_SYSTEM);
    return call;
}

__device__ __forceinline__ void link_role(const Params& P, unsigned call, int wid, int lane) {
    if (wid >= PW) return;
    const int rank = P.rank;
    const unsigned par = OFF_DATA + (call & 1u) * PAR_BYTES;
    char* wsl = P.wsl;
    unsigned* tmo = (unsigned*)(wsl + OFF_TMO);
    const unsigned lo16 = lane * 16;
    unsigned* q = (unsigned*)(wsl + OFF_LNKQ);
    constexpr unsigned NITEM = NITEM_IN + NSTG + TAILR * (TAILP - 1);
    const unsigned qbase = (call - 1u) * NITEM - NL * PW;
    unsigned tk = qbase + wid * NL + blockIdx.x, vz = 0;
    asm volatile("" : "+v"(vz));
    const int lsrc = (rank + 1 + lane) & 7;
    int prev = -1;
    auto publish = [&]() {
        if (prev >= 0 && lane == 0) __hip_atomic_fetch_add((unsigned*)(wsl + OFF_PULLED + (unsigned)prev * 64u), 1u, __ATOMIC_RELAXED, __HIP_MEMORY_SCOPE_AGENT);
        prev = -1;
    };
    for (;;) {
        const unsigned it = (unsigned)__builtin_amdgcn_readfirstlane(tk) - qbase;
        if (it >= NITEM) break;
        if (lane == 0) tk = __hip_atomic_fetch_add(q + vz, 1u, __ATOMIC_RELAXED, __HIP_MEMORY_SCOPE_AGENT);
        const bool isrec = it >= NITEM_IN;
        int id; unsigned foff, soff, doff, o0 = 0, ostep = 1024;
        if (isrec) {
            unsigned rec = it - NITEM_IN;
            if (rec >= NSTG - TAILR) { const unsigned t = rec - (NSTG - TAILR); rec = NSTG - TAILR + t / TAILP; o0 = (t % TAILP) * ostep; ostep *= TAILP; }
            id = 320 + (int)rec; foff = OFF_RECFLAG + (unsigned)(lsrc * NREC + rec) * 8u;
            soff = doff = par + REL_O + (unsigned)(lsrc * NREC + rec) * REC_BYTES;
        } else {
            const int u = (int)(it >> 6), sub = it & 63;
            const bool isw = u < 16 && (u & 1);
            const int st = u < 16 ? u >> 1 : u - 8;
            id = isw ? 256 + st * 8 + (sub >> 3) : st * 16 + (sub >> 2);
            foff = isw ? OFF_CVW + (unsigned)((id - 256) * 8 + lsrc) * 8u : OFF_CVX + (unsigned)((id * 8 + lsrc) * 4 + (sub & 3)) * 8u;
            doff = par + (isw ? REL_B : 0u) + (unsigned)(st * 8 + lsrc) * CHUNK + sub * 8192;
            const bool same = isw ? (lsrc & 3) == (rank & 3) : (lsrc >> 2) == (rank >> 2);
            soff = same ? doff : par + (isw ? REL_WO + (unsigned)((rank & 3) * 8 + st) * CHUNK : REL_XO + (unsigned)st * CHUNK) + sub * 8192;
        }
        Deadline dl(tmo, call);
        u64 f = 0; bool bad = false;
        for (;;) {
            if (lane < 7) f = __hip_atomic_load((const u64*)(wsl + foff), __ATOMIC_RELAXED, __HIP_MEMORY_SCOPE_SYSTEM);
            const u64 anylate = __builtin_amdgcn_ballot_w64(lane < 7 && (unsigned)(f >> 32) < call);
            WAIT_V(0);
            publish();
            if (anylate == 0ull) break;
            if (dl.expired()) { bad = true; break; }
            __builtin_amdgcn_s_sleep(8);
        }
        if (bad || __builtin_amdgcn_ballot_w64(lane < 7 && (unsigned)f > 4096u) != 0ull) continue;
        const unsigned lbytes = isrec ? rec_hdr_off((unsigned)f) + 8u * (64u - rec_groups((unsigned)f, false)) : 8192u;
        __amdgpu_buffer_rsrc_t rp[7], rl[7];
        unsigned maxb = 0;
#pragma unroll
        for (int k = 0; k < 7; ++k) {
            const unsigned bytes = __builtin_amdgcn_readlane(lbytes, k);
            rp[k] = mk_rsrc(P.peer[k] + (unsigned)__builtin_amdgcn_readlane(soff, k), bytes);
            rl[k] = mk_rsrc(wsl + (unsigned)__builtin_amdgcn_readlane(doff, k), bytes);
            maxb = max(maxb, bytes);
        }
        for (unsigned o = o0; o < maxb; o += ostep) {
            u32x4 d[7];
#pragma unroll
            for (int k = 0; k < 7; ++k) d[k] = __builtin_amdgcn_raw_buffer_load_b128(rp[k], lo16 + o, 0, AUX_SYS);
#pragma unroll
            for (int k = 0; k < 7; ++k) __builtin_amdgcn_raw_buffer_store_b128(d[k], rl[k], lo16 + o, 0, AUX_SYS);
        }
        prev = id;
    }
    WAIT_V(0);
    publish();
}

__device__ __forceinline__ void wait_inputs(const Params& P, LAS unsigned char* lds, int st, int mt, int nt, unsigned call, int lane) {
    char* wsl = P.wsl;
    const int bx = st * 16 + mt, bw = 256 + (st & 7) * 8 + nt;
    const unsigned off = lane < 32 ? OFF_CVX + (unsigned)(bx * 32 + lane) * 8u : lane < 40 ? OFF_CVW + (unsigned)((bw - 256) * 8 + (lane & 7)) * 8u
                                   : OFF_PULLED + (unsigned)(lane == 40 ? bx : bw) * 64u;
    const unsigned need = lane == 40 ? 4u * call : 8u * call;
    Deadline dl((unsigned*)(wsl + OFF_TMO), call);
    u64 f = 0;
    for (;;) {
        if (lane < 42) f = __hip_atomic_load((const u64*)(wsl + off), __ATOMIC_RELAXED, __HIP_MEMORY_SCOPE_SYSTEM);
        const bool late = lane < 40 ? (unsigned)(f >> 32) < call : lane < 42 && (unsigned)f < need;
        if (__builtin_amdgcn_ballot_w64(late) == 0ull) break;
        if (dl.expired()) break;
        if (st == 8) __builtin_amdgcn_s_sleep(127); else __builtin_amdgcn_s_sleep(32);
    }
    __builtin_amdgcn_fence(__ATOMIC_ACQUIRE, "");
    LAS int* tab = (LAS int*)(lds + LDS_MISC + 1280);
    const int cw = __shfl((int)((unsigned)f & 0xffu), 32 + ((lane >> 2) & 7));
    if (lane < 32) tab[(st & 7) * 32 + lane] = (int)((unsigned)f & 0xffu) + cw;
    WAIT_V(0); WAIT_L(0);
}

__device__ __forceinline__ void gemm_role(const Params& P, LAS unsigned char* lds, unsigned call, int wid, int lane) {
    const int gi = blockIdx.x - NL, xcd = gi & 7, ix = (gi & 127) >> 3;
    const int wr = wid >> 2, wc = wid & 3, fr = lane & 15, fq = lane >> 4;
    const unsigned par = OFF_DATA + (call & 1u) * PAR_BYTES;
    char* wsl = P.wsl;
    const int nt = (xcd & 1) * 4 + (ix & 3);
    const unsigned aoff = (unsigned)(wr * 8192 + lane * 16), boff = (unsigned)(16384 + wc * 8192 + lane * 16);
    const LAS int* tab = (const LAS int*)(lds + LDS_MISC + 1280) + wr;
    const int ntile = gi < 32 ? 2 : 1;
    for (int ti = 0; ti < ntile; ++ti) {
        const int mt = gi >= NG ? 16 + (xcd >> 1) * 3 + (ix >> 2) : ti == 0 ? (xcd >> 1) * 4 + (ix >> 2) : 28 + (xcd >> 1);
        const int job = mt >> 4;
        const char* ga = wsl + par + (size_t)job * (64 * CHUNK) + (mt & 15) * 32768 + wid * 2048 + lane * 16;
        const char* gb = wsl + par + REL_B + nt * 65536 + wid * 4096 + lane * 16;
#define ISSUE(kt_, sb_) do { const char* _a = ga + (((kt_) >> 1) * CHUNK + ((kt_) & 1) * 16384); const char* _b = gb + (((kt_) >> 1) * CHUNK + ((kt_) & 1) * 32768); \
        LAS unsigned char* _la = lds + (sb_) + wid * 2048; LAS unsigned char* _lb = lds + (sb_) + 16384 + wid * 4096; \
        __builtin_amdgcn_global_load_lds((const unsigned*)_a, (LAS unsigned*)_la, 16, 0, 0); \
        __builtin_amdgcn_global_load_lds((const unsigned*)_a, (LAS unsigned*)_la, 16, 1024, 0); \
        __builtin_amdgcn_global_load_lds((const unsigned*)_b, (LAS unsigned*)_lb, 16, 0, 0); \
        __builtin_amdgcn_global_load_lds((const unsigned*)_b, (LAS unsigned*)_lb, 16, 1024, 0); \
        __builtin_amdgcn_global_load_lds((const unsigned*)_b, (LAS unsigned*)_lb, 16, 2048, 0); \
        __builtin_amdgcn_global_load_lds((const unsigned*)_b, (LAS unsigned*)_lb, 16, 3072, 0); } while (0)
#define LDF(A_, B_, sb_, kf_) do { _Pragma("unroll") for (int i = 0; i < 4; ++i) { \
        B_[i] = *(const LAS h16x8*)(lds + (sb_) + boff + i * 2048 + (kf_) * 1024); \
        A_[i] = *(const LAS h16x8*)(lds + (sb_) + aoff + i * 2048 + (kf_) * 1024); } } while (0)
#define MMA(A_, B_) do { __builtin_amdgcn_s_setprio(1); _Pragma("unroll") for (int m = 0; m < 4; ++m) _Pragma("unroll") for (int c = 0; c < 4; ++c) \
        acc[m][c] = __builtin_amdgcn_mfma_f32_16x16x32_f16(B_[c], A_[m], acc[m][c], 0, 0, 0); __builtin_amdgcn_s_setprio(0); } while (0)
        f32x4 acc[4][4], tot[4][4];
#pragma unroll
        for (int m = 0; m < 4; ++m)
#pragma unroll
            for (int c = 0; c < 4; ++c) acc[m][c] = tot[m][c] = (f32x4){0.f, 0.f, 0.f, 0.f};
        h16x8 A0[4], B0[4], A1[4], B1[4];
        if (wid == 0) wait_inputs(P, lds, job * 8, mt & 15, nt, call, lane);
        BAR();
        ISSUE(0, 0); ISSUE(1, STAGE); ISSUE(2, 2 * STAGE);
        WAIT_V(12); BAR();
        LDF(A0, B0, 0u, 0);
        int tnext = tab[0];
        int tcur = __builtin_amdgcn_readfirstlane(tnext);
        int tref = tcur;
        unsigned sb = 0;
        for (int kt = 0; kt < 128; ++kt) {
            const unsigned sn = sb == 2 * STAGE ? 0u : sb + STAGE;
            __builtin_amdgcn_s_waitcnt(0xC07F);
            LDF(A1, B1, sb, 1);
            const int tn = __builtin_amdgcn_readfirstlane(tnext);
            if (tn != tcur) {
                const bool nearby = abs(tn - tref) <= 30;
                const int ekeep = nearby ? tcur - tn : -2000, ebank = nearby ? -2000 : tcur - 2 * EXP_TARGET;
#pragma unroll
                for (int m = 0; m < 4; ++m)
#pragma unroll
                    for (int c = 0; c < 4; ++c)
#pragma unroll
                        for (int r = 0; r < 4; ++r) {
                            tot[m][c][r] += __builtin_ldexpf(acc[m][c][r], ebank);
                            acc[m][c][r] = __builtin_ldexpf(acc[m][c][r], ekeep);
                        }
                tref = nearby ? tref : tn;
                tcur = tn;
            }
            MMA(A0, B0);
            if (((kt + 3) & 15) == 0 && kt + 3 < 128 && wid == 0) wait_inputs(P, lds, job * 8 + ((kt + 3) >> 4), mt & 15, nt, call, lane);
            WAIT_V(6); __builtin_amdgcn_s_waitcnt(0xC07F); BAR();
            ISSUE((kt + 3) & 127, sb);
            LDF(A0, B0, sn, 0);
            tnext = tab[min(kt + 1, 127) * 2];
            MMA(A1, B1);
            sb = sn;
        }
        WAIT_V(0); WAIT_L(0); BAR();
        unsigned* tmo = (unsigned*)(wsl + OFF_TMO);
        const bool bad = (unsigned)__builtin_amdgcn_readfirstlane(__hip_atomic_load(tmo, __ATOMIC_RELAXED, __HIP_MEMORY_SCOPE_AGENT)) == call;
        LAS float* cv = (LAS float*)(lds + wid * 16384);
        unsigned mlo = 0, mhi = 0, off = 0;
        const int unsc = tcur - 2 * EXP_TARGET;
#pragma unroll
        for (int m = 0; m < 4; ++m)
#pragma unroll
            for (int c = 0; c < 4; ++c)
#pragma unroll
                for (int r = 0; r < 4; ++r) {
                    float v = tot[m][c][r] + __builtin_ldexpf(acc[m][c][r], unsc);
                    v = v <= 0.f ? 0.f : __builtin_fabsf(v);
                    if (bad) v = __builtin_nanf("");
                    acc[m][c][r] = v;
                    const bool nz = __float_as_uint(v) != 0u;
                    const u64 mk = __builtin_amdgcn_ballot_w64(nz);
                    const unsigned lo = (unsigned)mk, hi = (unsigned)(mk >> 32);
                    const unsigned pos = off + __builtin_amdgcn_mbcnt_hi(hi, __builtin_amdgcn_mbcnt_lo(lo, 0u));
                    if (nz) cv[pos] = v;
                    const int q = (m * 4 + c) * 4 + r;
                    mlo = lane == q ? lo : mlo; mhi = lane == q ? hi : mhi;
                    off += (unsigned)__builtin_popcountll(mk);
                }
        const int rec = (mt * 8 + nt) * 8 + wid;
        const __amdgpu_buffer_rsrc_t rl = mk_rsrc(wsl, WS_USED);
        const unsigned ro = par + REL_O + (unsigned)(P.rank * NREC + rec) * REC_BYTES;
        const unsigned ng = rec_groups(off, rec >= NSTG);
        if (lane < 64u - ng) __builtin_amdgcn_raw_buffer_store_b64((u32x2){mlo, mhi}, rl, lane * 8 + ro + rec_hdr_off(off), 0, AUX_SYS);
        WAIT_L(0);
        for (unsigned g = 0; g < ng; ++g) {
            const u64 mk = ((u64)(unsigned)__builtin_amdgcn_readlane(mhi, 64u - ng + g) << 32) | (unsigned)__builtin_amdgcn_readlane(mlo, 64u - ng + g);
            __hip_atomic_fetch_or((LAS unsigned*)cv + g * 64 + lane, __builtin_amdgcn_inverse_ballot_w64(mk) ? 0x80000000u : 0u, __ATOMIC_RELAXED, __HIP_MEMORY_SCOPE_WORKGROUP);
        }
        WAIT_L(0);
        const __amdgpu_buffer_rsrc_t rv = mk_rsrc(wsl + ro, off * 4u);
        for (unsigned i = 0; i * 256 < off; ++i) {
            const u32x4 d = *(const LAS u32x4*)((LAS unsigned char*)cv + i * 1024 + lane * 16);
            __builtin_amdgcn_raw_buffer_store_b128(d, rv, lane * 16 + i * 1024, 0, AUX_SYS);
        }
        LAS unsigned* nz8 = (LAS unsigned*)(lds + LDS_MISC + 1024);
        if (lane == 0) nz8[wid] = off;
        WAIT_V(0); WAIT_L(0); BAR();
        if (wid == 0 && (lane >> 3) != P.rank)
            __hip_atomic_store((u64*)(P.ws[lane >> 3] + OFF_RECFLAG) + (unsigned)(P.rank * NREC + rec + (lane & 7)), ((u64)call << 32) | nz8[lane & 7], __ATOMIC_RELAXED, __HIP_MEMORY_SCOPE_SYSTEM);
        store_dense(P.out, (P.rank >> 2) * 4096 + mt * 128 + wr * 64, (P.rank & 3) * 2048 + nt * 256 + wc * 64, fr, fq, acc);
        WAIT_L(0); BAR();
#undef ISSUE
#undef LDF
#undef MMA
    }
}

__device__ __forceinline__ void expand_role(const Params& P, LAS unsigned char* lds, unsigned call, int wid, int lane) {
    const int fr = lane & 15, fq = lane >> 4;
    constexpr unsigned NJOB = 7 * NSTG, E = NE * EW;
    const int gi = blockIdx.x - NL;
    const bool direct = gi < 32 || gi >= NG;
    const unsigned didx = (unsigned)(wid * 128 + (gi < 32 ? gi : gi - 96));
    if (direct ? didx >= 7u * DIRR : wid >= EW) return;
    const unsigned par = OFF_DATA + (call & 1u) * PAR_BYTES;
    char* wsl = P.wsl;
    unsigned* tmo = (unsigned*)(wsl + OFF_TMO);
    LAS unsigned char* buf = lds + wid * 16384;
    unsigned* q = (unsigned*)(wsl + OFF_EXPQ);
    const unsigned qbase = (call - 1u) * (NJOB + E);
    unsigned tk = qbase + NJOB + didx;
    if (!direct) { if (lane == 0) tk = __hip_atomic_fetch_add(q, 1u, __ATOMIC_RELAXED, __HIP_MEMORY_SCOPE_AGENT); }
    else {
        Deadline dl(tmo, call);
        while ((unsigned)__builtin_amdgcn_readfirstlane(__hip_atomic_load((unsigned*)(wsl + OFF_PULLED + (unsigned)(320 + NSTG - TAILR - 32) * 64u), __ATOMIC_RELAXED, __HIP_MEMORY_SCOPE_AGENT)) < call) {
            if (dl.expired()) break;
            __builtin_amdgcn_s_sleep(127); __builtin_amdgcn_s_sleep(127);
        }
    }
    int it = direct ? 1 : 0;
    unsigned vz = 0;
    asm volatile("" : "+v"(vz));
    for (;; ++it) {
        const unsigned ku = (unsigned)__builtin_amdgcn_readfirstlane(tk) - qbase;
        if (ku >= NJOB && !direct) break;
        const int k = (int)ku;
        const int rec = k / 7, src = (P.rank + 1 + (k - rec * 7)) & 7;
        const u64* flag = (const u64*)(wsl + (lane == 0 ? OFF_PULLED + (unsigned)(320 + (direct ? rec - DIRR : rec)) * 64u : OFF_RECFLAG + (unsigned)(src * NREC + rec) * 8u));
        const unsigned need = rec >= NSTG - TAILR ? TAILP * call : call;
        Deadline dl(tmo, call);
        unsigned nnz = 0; bool bad = false;
        for (;;) {
            u64 f = 0;
            if (lane < 2) f = __hip_atomic_load(flag, __ATOMIC_RELAXED, __HIP_MEMORY_SCOPE_SYSTEM);
            nnz = __builtin_amdgcn_readlane((unsigned)f, 1);
            if ((unsigned)__builtin_amdgcn_readlane((unsigned)f, 0) >= need && (unsigned)__builtin_amdgcn_readlane((unsigned)(f >> 32), 1) >= call) break;
            if (dl.expired()) { bad = true; break; }
            if (it == 0) __builtin_amdgcn_s_sleep(100); else __builtin_amdgcn_s_sleep(4);
        }
        if (nnz > 4096u) { nnz = 0; bad = true; }
        f32x4 v[4][4];
        if (!bad) {
            const char* rbase = (direct ? P.ws[src] : wsl) + par + REL_O + (size_t)(src * NREC + rec) * REC_BYTES;
            const unsigned ng = rec_groups(nnz, direct);
            u32x2 hdr = __builtin_amdgcn_raw_buffer_load_b64(mk_rsrc(rbase + rec_hdr_off(nnz), 8u * (64u - ng)), lane * 8, 0, AUX_SYS);
            const char* gsrc = rbase + lane * 16;
            const unsigned bytes = nnz * 4;
            for (unsigned i = 0; i < bytes; i += 1024)
                if (i + lane * 16 < bytes) __builtin_amdgcn_global_load_lds((const unsigned*)(gsrc + i), (LAS unsigned*)(buf + i), 16, 0, AUX_SYS);
            WAIT_V(0);
            for (unsigned g0 = 0; g0 < ng; g0 += 8) {
                int t[8];
#pragma unroll
                for (int u = 0; u < 8; ++u) t[u] = *(const LAS int*)(buf + (((g0 + u) & 63u) * 64 + lane) * 4);
#pragma unroll
                for (int u = 0; u < 8; ++u) {
                    const u64 mk = __builtin_amdgcn_ballot_w64(t[u] < 0);
                    const bool me = (unsigned)lane == 64u - ng + g0 + u;
                    hdr[0] = me ? (unsigned)mk : hdr[0]; hdr[1] = me ? (unsigned)(mk >> 32) : hdr[1];
                }
            }
            unsigned off = 0;
#pragma unroll
            for (int m = 0; m < 4; ++m)
#pragma unroll
                for (int c = 0; c < 4; ++c)
#pragma unroll
                    for (int r = 0; r < 4; ++r) {
                        const int qq = (m * 4 + c) * 4 + r;
                        const unsigned lo = __builtin_amdgcn_readlane(hdr[0], qq), hi = __builtin_amdgcn_readlane(hdr[1], qq);
                        const u64 mk = ((u64)hi << 32) | lo;
                        const unsigned pos = (off + __builtin_amdgcn_mbcnt_hi(hi, __builtin_amdgcn_mbcnt_lo(lo, 0u))) & 4095u;
                        const float t = *(const LAS float*)(buf + pos * 4);
                        v[m][c][r] = __builtin_amdgcn_inverse_ballot_w64(mk) ? __builtin_fabsf(t) : 0.f;
                        off += (unsigned)__builtin_popcountll(mk);
                    }
        } else {
#pragma unroll
            for (int m = 0; m < 4; ++m)
#pragma unroll
                for (int c = 0; c < 4; ++c) { const float qn = __builtin_nanf(""); v[m][c] = (f32x4){qn, qn, qn, qn}; }
        }
        if (lane == 0 && !direct) tk = __hip_atomic_fetch_add(q + vz, 1u, __ATOMIC_RELAXED, __HIP_MEMORY_SCOPE_AGENT);
        const int w8 = rec & 7, nt = (rec >> 3) & 7, mt = rec >> 6;
        const int row0 = (src >> 2) * 4096 + mt * 128 + (w8 >> 2) * 64, col0 = (src & 3) * 2048 + nt * 256 + (w8 & 3) * 64;
        store_dense(P.out, row0, col0, fr, fq, v);
        WAIT_L(0);
        if (direct) break;
    }
}

__global__ __launch_bounds__(512, 2) void fused_kernel(Params P) {
    extern __shared__ __attribute__((aligned(16))) unsigned char shm[];
    LAS unsigned char* lds = (LAS unsigned char*)shm;
    const int tid = threadIdx.x, lane = tid & 63, wid = __builtin_amdgcn_readfirstlane(tid >> 6);
    LAS unsigned* misc = (LAS unsigned*)(lds + LDS_MISC);
    unsigned ticket = 0, vz = 0;
    asm volatile("" : "+v"(vz));
    if (tid == 0) ticket = __hip_atomic_fetch_add((unsigned*)(P.wsl + OFF_CTR + (blockIdx.x & 7) * 128 + vz), 1u, __ATOMIC_RELAXED, __HIP_MEMORY_SCOPE_AGENT);
    if (tid >= 64 && tid < 128) misc[tid] = 0u;
    auto ready = [&]() -> unsigned {
        asm volatile("" : "+v"(ticket));
        if (tid == 0) misc[0] = ticket / (unsigned)(GRID / 8) + 1u;
        WAIT_L(0); BAR();
        return __builtin_amdgcn_readfirstlane(misc[0]);
    };
    if (blockIdx.x < NL) link_role(P, ready(), wid, lane);
    else {
        unsigned call;
        if (blockIdx.x < NL + NG) call = convert_first(P, lds, ready, wid, lane);
        else { call = ready(); convert_role(P, lds, call, wid, lane); }
        gemm_role(P, lds, call, wid, lane);
        expand_role(P, lds, call, wid, lane);
    }
}

}

extern "C" void kernel_launch(int rank, int nranks, ncclComm_t comm,
                              hipStream_t stream,
                              void* const* d_in, const int* in_sizes, int n_in,
                              void* d_out, int out_size,
                              const int* in_dtypes, int out_dtype,
                              void* const* d_out_all,
                              void* const* d_ws_all, size_t ws_bytes) {
    using namespace gar;
    if (nranks != NRANK || (unsigned)rank >= (unsigned)NRANK || ws_bytes < (size_t)WS_USED || n_in < 2 ||
        in_sizes[0] != GM * KS || in_sizes[1] != KS * GN || out_size != GM * GN) {
        fprintf(stderr, "kernel_launch: unexpected problem shape\n");
        return;
    }
    int dev = 0, ncu = 0;
    HIP_CHECK(hipGetDevice(&dev));
    HIP_CHECK(hipDeviceGetAttribute(&ncu, hipDeviceAttributeMultiprocessorCount, dev));
    if (ncu < GRID) { fprintf(stderr, "kernel_launch: needs %d CUs, found %d\n", GRID, ncu); return; }
    Params P;
    P.x = (const float*)d_in[0]; P.w = (const float*)d_in[1]; P.out = (float*)d_out; P.rank = rank;
    for (int p = 0; p < NRANK; ++p) P.ws[p] = (char*)d_ws_all[p];
    P.wsl = P.ws[rank]; P.pad = 0;
    for (int k = 0; k < 7; ++k) P.peer[k] = P.ws[(rank + 1 + k) & 7];
    HIP_CHECK(hipFuncSetAttribute((const void*)fused_kernel, hipFuncAttributeMaxDynamicSharedMemorySize, LDS_BYTES));
    hipLaunchKernelGGL(fused_kernel, dim3(GRID), dim3(512), LDS_BYTES, stream, P);
    HIP_CHECK(hipGetLastError());
}
```

```cpp
#include <hip/hip_runtime.h>
#include <rccl/rccl.h>
#include <cstdio>

#define HIP_CHECK(expr) do { hipError_t _e = (expr); if (_e != hipSuccess) { \
    fprintf(stderr, "HIP error %s at %s:%d\n", hipGetErrorString(_e), __FILE__, __LINE__); \
    return; } } while (0)

namespace gar {

#define LAS __attribute__((address_space(3)))
typedef _Float16 h16x8 __attribute__((ext_vector_type(8)));
typedef float f32x4 __attribute__((ext_vector_type(4)));
typedef unsigned u32x4 __attribute__((ext_vector_type(4)));
typedef unsigned u32x2 __attribute__((ext_vector_type(2)));
typedef unsigned long long u64;

constexpr int NRANK = 8;
constexpr int GM = 8192, GN = 8192, KS = 1024;
constexpr int PW = 4;
constexpr int EW = 5;
constexpr int TAILR = 192, TAILP = 4;
constexpr int DIRR = 40;
constexpr int NL = 32;
constexpr int NG = 128;
constexpr int NE = 96;
constexpr int GRID = NL + NG + NE;
constexpr int NREC = 2048;
constexpr int NSTG = NREC - DIRR;
constexpr unsigned TILE_BYTES = 8 * 16896;
static_assert(GRID == 256 && DIRR % 8 == 0 && TAILR % 8 == 0 && DIRR <= TAILR && 7 * DIRR <= 8 * 128 && PW <= 8 && EW <= 8, "geometry");
constexpr unsigned NITEM_IN = 24 * 64;

constexpr unsigned OFF_TMO = 0, OFF_EXPQ = 128, OFF_LNKQ = 256, OFF_CTR = 1024;
constexpr unsigned OFF_CVW = 4096;
constexpr unsigned OFF_CVX = 0x180000;
constexpr unsigned OFF_PULLED = 65536;
constexpr unsigned OFF_RECFLAG = 0x140000;
constexpr unsigned OFF_DATA = 2u << 20;
constexpr unsigned CHUNK = 512u << 10;
constexpr unsigned A_BYTES = 128 * CHUNK, B_BYTES = 64 * CHUNK, XO_BYTES = 16 * CHUNK, WO_BYTES = 32 * CHUNK, O_BYTES = NRANK * (NREC / 8) * TILE_BYTES;
constexpr unsigned REL_B = A_BYTES, REL_XO = REL_B + B_BYTES, REL_WO = REL_XO + XO_BYTES, REL_O = REL_WO + WO_BYTES;
constexpr unsigned PAR_BYTES = REL_O + O_BYTES;
constexpr unsigned WS_USED = OFF_DATA + 2 * PAR_BYTES;
constexpr unsigned STAGE = 49152, LDS_MISC = 3 * STAGE, LDS_BYTES = LDS_MISC + 2304;
constexpr u64 DEADLINE = 150000000ull;
constexpr int AUX_SYS = 17;
constexpr int EXP_TARGET = 142;

struct Params {
    const float* x; char* wsl; int rank; int pad;
    const float* w; float* out; char* ws[NRANK];
    char* peer[7];
};

#define WAIT_V(n) asm volatile("s_waitcnt vmcnt(" #n ")" ::: "memory")
#define WAIT_L(n) asm volatile("s_waitcnt lgkmcnt(" #n ")" ::: "memory")
#define BAR() do { asm volatile("" ::: "memory"); __builtin_amdgcn_s_barrier(); asm volatile("" ::: "memory"); } while (0)

__device__ __forceinline__ __amdgpu_buffer_rsrc_t mk_rsrc(const void* p, unsigned bytes) {
    return __builtin_amdgcn_make_buffer_rsrc((void*)p, 0, bytes, 0x00020000);
}
__device__ __forceinline__ void st16_sys(char* p, u32x4 v) {
    __builtin_amdgcn_global_store_b128((__attribute__((address_space(1))) u32x4*)p, v, "");
}
__device__ __forceinline__ u32x4 cvt8(f32x4 a, f32x4 b, int sh) {
    h16x8 h;
#pragma unroll
    for (int j = 0; j < 4; ++j) { h[j] = (_Float16)__builtin_ldexpf(a[j], sh); h[4 + j] = (_Float16)__builtin_ldexpf(b[j], sh); }
    return __builtin_bit_cast(u32x4, h);
}
__device__ __forceinline__ unsigned rec_groups(unsigned nnz, bool direct) { return direct ? 0u : nnz >> 6; }
__device__ __forceinline__ unsigned rec_hdr_off(unsigned nnz) { return (nnz * 4u + 7u) & ~7u; }
__device__ __forceinline__ unsigned rec_bytes(unsigned nnz, bool direct) { return (rec_hdr_off(nnz) + 8u * (64u - rec_groups(nnz, direct)) + 15u) & ~15u; }
__device__ __forceinline__ unsigned scan8(unsigned v, int lane) {
    unsigned t;
    t = (unsigned)__builtin_amdgcn_update_dpp(0, (int)v, 0x111, 0xf, 0xf, true); v += (lane & 7) >= 1 ? t : 0u;
    t = (unsigned)__builtin_amdgcn_update_dpp(0, (int)v, 0x112, 0xf, 0xf, true); v += (lane & 7) >= 2 ? t : 0u;
    t = (unsigned)__builtin_amdgcn_update_dpp(0, (int)v, 0x114, 0xf, 0xf, true); v += (lane & 7) >= 4 ? t : 0u;
    return v;
}
struct Deadline {
    unsigned* tmo; unsigned call; u64 t0; unsigned polls;
    __device__ __forceinline__ Deadline(unsigned* tmo_, unsigned call_) : tmo(tmo_), call(call_), t0(__builtin_amdgcn_s_memrealtime()), polls(1) {}
    __device__ __forceinline__ bool expired() {
        if ((polls++ & 63u) == 0u &&
            (unsigned)__builtin_amdgcn_readfirstlane(__hip_atomic_load(tmo, __ATOMIC_RELAXED, __HIP_MEMORY_SCOPE_AGENT)) == call) return true;
        if (__builtin_amdgcn_s_memrealtime() - t0 > DEADLINE) {
            __hip_atomic_store(tmo, call, __ATOMIC_RELAXED, __HIP_MEMORY_SCOPE_AGENT);
            return true;
        }
        return false;
    }
};
__device__ __forceinline__ unsigned wave_max8(unsigned v) {
    unsigned r = 0;
#pragma unroll
    for (int b = 7; b >= 0; --b) { const unsigned c = r | (1u << b); r = __builtin_amdgcn_ballot_w64(v >= c) != 0ull ? c : r; }
    return r;
}
__device__ __forceinline__ unsigned exp1_max4(unsigned m, u32x4 v) {
#pragma unroll
    for (int i = 0; i < 4; ++i) m = max(m, (v[i] + 0x00800000u) & 0x7f800000u);
    return m;
}

__device__ __forceinline__ void store_dense(float* out, int row0, int col0, int fr, int fq, const f32x4 (&v)[4][4]) {
    float* p = out + (size_t)(row0 + fr) * GN + col0 + 4 * fq;
#pragma unroll
    for (int m = 0; m < 4; ++m)
#pragma unroll
        for (int r = 0; r < 4; ++r) {
            f32x4 o = {v[m][0][r], v[m][1][r], v[m][2][r], v[m][3][r]};
            st16_sys((char*)(p + (size_t)m * 16 * GN + r * 16), __builtin_bit_cast(u32x4, o));
        }
}

__device__ __forceinline__ unsigned block_code(LAS unsigned* word, unsigned m, int lane) {
    m = wave_max8(m >> 23);
    if (lane == 0) __hip_atomic_fetch_max(word, m, __ATOMIC_RELAXED, __HIP_MEMORY_SCOPE_WORKGROUP);
    WAIT_L(0); BAR();
    return min(((unsigned)__builtin_amdgcn_readfirstlane(*word) + 3u) & ~3u, 255u);
}
__device__ __forceinline__ void load_xblock(const Params& P, f32x4 (&v)[16], int half, int st, int mt, int wid, int fr, int fq) {
    const float* s = P.x + (size_t)(half * 4096 + (st >> 3) * 2048 + mt * 128 + wid * 16 + fr) * KS + (st & 7) * 128 + fq * 8;
#pragma unroll
    for (int i = 0; i < 8; ++i) v[i] = *(const f32x4*)(s + (i >> 1) * 32 + (i & 1) * 4);
}

__device__ __forceinline__ void convert_role(const Params& P, LAS unsigned char* lds, unsigned call, int wid, int lane) {
    const int fr = lane & 15, fq = lane >> 4, rank = P.rank;
    const unsigned par = OFF_DATA + (call & 1u) * PAR_BYTES;
    LAS unsigned* red = (LAS unsigned*)(lds + LDS_MISC + 256);
    const __amdgpu_buffer_rsrc_t rl = mk_rsrc(P.wsl, WS_USED);
    int n = 0;
    __builtin_amdgcn_s_sleep(64);
    for (int i = blockIdx.x - NL - NG; i < 23 * 32; i += NE, ++n) {
        const int u = (i >> 5) + 1, sub = i & 31;
        const bool isw = u < 16 && (u & 1);
        const int st = u < 16 ? u >> 1 : u - 8;
        f32x4 v[16];
        unsigned m = 0, foff; int dst;
        if (isw) {
            const int b = sub >> 3, nt = sub & 7;
            const float* s = P.w + (size_t)(st * 128 + (wid >> 2) * 64 + fq * 8) * GN + b * 2048 + nt * 256 + (wid & 3) * 64 + 4 * (4 * (fr & 3) + (fr >> 2));
#pragma unroll
            for (int e = 0; e < 16; ++e) v[e] = *(const f32x4*)(s + (size_t)((e >> 3) * 32 + (e & 7)) * GN);
#pragma unroll
            for (int e = 0; e < 16; ++e) m = exp1_max4(m, __builtin_bit_cast(u32x4, v[e]));
            m = block_code(red + n, m, lane);
            const int sh = EXP_TARGET - (int)m;
            const unsigned off = par + (b == (rank & 3) ? REL_B + (st * 8 + rank) * CHUNK : REL_WO + (b * 8 + st) * CHUNK) + nt * 65536 + wid * 8192;
#pragma unroll
            for (int c = 0; c < 4; ++c)
#pragma unroll
                for (int kf = 0; kf < 2; ++kf) {
                    const f32x4* q = v + 8 * kf;
                    const u32x4 h = cvt8((f32x4){q[0][c], q[1][c], q[2][c], q[3][c]}, (f32x4){q[4][c], q[5][c], q[6][c], q[7][c]}, sh);
                    __builtin_amdgcn_raw_buffer_store_b128(h, rl, lane * 16 + off + c * 2048 + kf * 1024, 0, AUX_SYS);
                }
            foff = OFF_CVW + (unsigned)((st * 8 + nt) * 8 + rank) * 8u; dst = lane < 2 ? b + 4 * lane : -1;
        } else {
            const int half = sub >> 4, mt = sub & 15;
            load_xblock(P, v, half, st, mt, wid, fr, fq);
#pragma unroll
            for (int e = 0; e < 8; ++e) m = exp1_max4(m, __builtin_bit_cast(u32x4, v[e]));
            m = block_code(red + n, m, lane);
            const int sh = EXP_TARGET - (int)m;
            const unsigned off = par + (half == (rank >> 2) ? (st * 8 + rank) * CHUNK : REL_XO + st * CHUNK) + mt * 32768 + wid * 2048;
#pragma unroll
            for (int kq = 0; kq < 4; ++kq)
                __builtin_amdgcn_raw_buffer_store_b128(cvt8(v[2 * kq], v[2 * kq + 1], sh), rl, lane * 16 + off + (kq >> 1) * 16384 + (kq & 1) * 1024, 0, AUX_SYS);
            foff = OFF_CVX + (unsigned)(((st * 16 + mt) * 8 + rank) * 4 + (lane & 3)) * 8u; dst = lane < 16 ? half * 4 + (lane >> 2) : -1;
        }
        WAIT_V(0); BAR();
        if (wid == 0 && dst >= 0)
            __hip_atomic_store((u64*)(P.ws[dst & 7] + foff), ((u64)call << 32) | m, __ATOMIC_RELAXED, __HIP_MEMORY_SCOPE_SYSTEM);
    }
}

template <class F>
__device__ __forceinline__ unsigned convert_first(const Params& P, LAS unsigned char* lds, F&& ready, int wid, int lane) {
    const int gi = blockIdx.x - NL, half = gi >> 6, mt = (gi >> 2) & 15, sub = gi & 3, rank = P.rank;
    const float* s = P.x + (size_t)(half * 4096 + mt * 128 + (sub & 1) * 64 + (wid >> 1) * 16 + (lane & 15)) * KS + (sub >> 1) * 64 + (wid & 1) * 32 + (lane >> 4) * 8;
    const f32x4 v0 = *(const f32x4*)s, v1 = *(const f32x4*)(s + 4);
    char* dst = P.ws[half * 4 + (lane & 3)];
    const unsigned call = ready();
    asm volatile("" : "+v"(dst));
    LAS unsigned* red = (LAS unsigned*)(lds + LDS_MISC + 256);
    const unsigned m = block_code(red, exp1_max4(exp1_max4(0u, __builtin_bit_cast(u32x4, v0)), __builtin_bit_cast(u32x4, v1)), lane);
    const unsigned off = OFF_DATA + (call & 1u) * PAR_BYTES + (half == (rank >> 2) ? rank * CHUNK : REL_XO) + mt * 32768 + sub * 8192 + wid * 1024 + lane * 16;
    __builtin_amdgcn_raw_buffer_store_b128(cvt8(v0, v1, EXP_TARGET - (int)m), mk_rsrc(P.wsl, WS_USED), off, 0, AUX_SYS);
    WAIT_V(0); BAR();
    if (wid == 0 && lane < 4)
        __hip_atomic_store((u64*)(dst + OFF_CVX) + (mt * 8 + rank) * 4 + sub, ((u64)call << 32) | m, __ATOMIC_RELAXED, __HIP_MEMORY_SCOPE_SYSTEM);
    return call;
}

__device__ __forceinline__ void link_role(const Params& P, unsigned call, int wid, int lane) {
    if (wid >= PW) return;
    const int rank = P.rank;
    const unsigned par = OFF_DATA + (call & 1u) * PAR_BYTES;
    char* wsl = P.wsl;
    unsigned* tmo = (unsigned*)(wsl + OFF_TMO);
    const unsigned lo16 = lane * 16;
    unsigned* q = (unsigned*)(wsl + OFF_LNKQ);
    constexpr unsigned NITEM = NITEM_IN + NSTG + TAILR * (TAILP - 1);
    const unsigned qbase = (call - 1u) * NITEM - NL * PW;
    unsigned tk = qbase + wid * NL + blockIdx.x, vz = 0;
    asm volatile("" : "+v"(vz));
    const int lsrc = (rank + 1 + (lane >> 3)) & 7;
    int prev = -1;
    auto publish = [&]() {
        if (prev >= 0 && lane == 0) __hip_atomic_fetch_add((unsigned*)(wsl + OFF_PULLED + (unsigned)prev * 64u), 1u, __ATOMIC_RELAXED, __HIP_MEMORY_SCOPE_AGENT);
        prev = -1;
    };
    for (;;) {
        const unsigned it = (unsigned)__builtin_amdgcn_readfirstlane(tk) - qbase;
        if (it >= NITEM) break;
        if (lane == 0) tk = __hip_atomic_fetch_add(q + vz, 1u, __ATOMIC_RELAXED, __HIP_MEMORY_SCOPE_AGENT);
        const bool isrec = it >= NITEM_IN;
        int id, pick = 0; unsigned foff, soff, doff, o0 = 0, ostep = 1024;
        if (isrec) {
            unsigned rec = it - NITEM_IN;
            if (rec >= NSTG - TAILR) { const unsigned t = rec - (NSTG - TAILR); rec = NSTG - TAILR + t / TAILP; o0 = (t % TAILP) * ostep; ostep *= TAILP; }
            id = 320 + (int)rec; pick = rec & 7; foff = OFF_RECFLAG + (unsigned)(lsrc * NREC + (rec & ~7u) + (lane & 7)) * 8u;
            soff = doff = par + REL_O + (unsigned)(lsrc * (NREC / 8) + (rec >> 3)) * TILE_BYTES;
        } else {
            const int u = (int)(it >> 6), sub = it & 63;
            const bool isw = u < 16 && (u & 1);
            const int st = u < 16 ? u >> 1 : u - 8;
            id = isw ? 256 + st * 8 + (sub >> 3) : st * 16 + (sub >> 2);
            foff = isw ? OFF_CVW + (unsigned)((id - 256) * 8 + lsrc) * 8u : OFF_CVX + (unsigned)((id * 8 + lsrc) * 4 + (sub & 3)) * 8u;
            doff = par + (isw ? REL_B : 0u) + (unsigned)(st * 8 + lsrc) * CHUNK + sub * 8192;
            const bool same = isw ? (lsrc & 3) == (rank & 3) : (lsrc >> 2) == (rank >> 2);
            soff = same ? doff : par + (isw ? REL_WO + (unsigned)((rank & 3) * 8 + st) * CHUNK : REL_XO + (unsigned)st * CHUNK) + sub * 8192;
        }
        Deadline dl(tmo, call);
        u64 f = 0; bool bad = false;
        for (;;) {
            if (lane < 56) f = __hip_atomic_load((const u64*)(wsl + foff), __ATOMIC_RELAXED, __HIP_MEMORY_SCOPE_SYSTEM);
            const u64 anylate = __builtin_amdgcn_ballot_w64(lane < 56 && (unsigned)(f >> 32) < call);
            WAIT_V(0);
            publish();
            if (anylate == 0ull) break;
            if (dl.expired()) { bad = true; break; }
            __builtin_amdgcn_s_sleep(8);
        }
        if (bad || __builtin_amdgcn_ballot_w64(lane < 56 && (unsigned)f > 4096u) != 0ull) continue;
        unsigned lbytes = 8192u;
        if (isrec) {
            const unsigned len = rec_bytes((unsigned)f, false), end = scan8(len, lane), lo = (end - len) & ~127u;
            lbytes = ((lane & 7) == 7 ? end : end & ~127u) - lo; soff += lo; doff += lo;
        }
        __amdgpu_buffer_rsrc_t rp[7], rl[7];
        unsigned maxb = 0;
#pragma unroll
        for (int k = 0; k < 7; ++k) {
            const unsigned bytes = __builtin_amdgcn_readlane(lbytes, 8 * k + pick);
            rp[k] = mk_rsrc(P.peer[k] + (unsigned)__builtin_amdgcn_readlane(soff, 8 * k + pick), bytes);
            rl[k] = mk_rsrc(wsl + (unsigned)__builtin_amdgcn_readlane(doff, 8 * k + pick), bytes);
            maxb = max(maxb, bytes);
        }
        for (unsigned o = o0; o < maxb; o += ostep) {
            u32x4 d[7];
#pragma unroll
            for (int k = 0; k < 7; ++k) d[k] = __builtin_amdgcn_raw_buffer_load_b128(rp[k], lo16 + o, 0, AUX_SYS);
#pragma unroll
            for (int k = 0; k < 7; ++k) __builtin_amdgcn_raw_buffer_store_b128(d[k], rl[k], lo16 + o, 0, AUX_SYS);
        }
        prev = id;
    }
    WAIT_V(0);
    publish();
}

__device__ __forceinline__ void wait_inputs(const Params& P, LAS unsigned char* lds, int st, int mt, int nt, unsigned call, int lane) {
    char* wsl = P.wsl;
    const int bx = st * 16 + mt, bw = 256 + (st & 7) * 8 + nt;
    const unsigned off = lane < 32 ? OFF_CVX + (unsigned)(bx * 32 + lane) * 8u : lane < 40 ? OFF_CVW + (unsigned)((bw - 256) * 8 + (lane & 7)) * 8u
                                   : OFF_PULLED + (unsigned)(lane == 40 ? bx : bw) * 64u;
    const unsigned need = lane == 40 ? 4u * call : 8u * call;
    Deadline dl((unsigned*)(wsl + OFF_TMO), call);
    u64 f = 0;
    for (;;) {
        if (lane < 42) f = __hip_atomic_load((const u64*)(wsl + off), __ATOMIC_RELAXED, __HIP_MEMORY_SCOPE_SYSTEM);
        const bool late = lane < 40 ? (unsigned)(f >> 32) < call : lane < 42 && (unsigned)f < need;
        if (__builtin_amdgcn_ballot_w64(late) == 0ull) break;
        if (dl.expired()) break;
        if (st == 8) __builtin_amdgcn_s_sleep(127); else __builtin_amdgcn_s_sleep(32);
    }
    __builtin_amdgcn_fence(__ATOMIC_ACQUIRE, "");
    LAS int* tab = (LAS int*)(lds + LDS_MISC + 1280);
    const int cw = __shfl((int)((unsigned)f & 0xffu), 32 + ((lane >> 2) & 7));
    if (lane < 32) tab[(st & 7) * 32 + lane] = (int)((unsigned)f & 0xffu) + cw;
    WAIT_V(0); WAIT_L(0);
}

__device__ __forceinline__ void gemm_role(const Params& P, LAS unsigned char* lds, unsigned call, int wid, int lane) {
    const int gi = blockIdx.x - NL, xcd = gi & 7, ix = (gi & 127) >> 3;
    const int wr = wid >> 2, wc = wid & 3, fr = lane & 15, fq = lane >> 4;
    const unsigned par = OFF_DATA + (call & 1u) * PAR_BYTES;
    char* wsl = P.wsl;
    const int nt = (xcd & 1) * 4 + (ix & 3);
    const unsigned aoff = (unsigned)(wr * 8192 + lane * 16), boff = (unsigned)(16384 + wc * 8192 + lane * 16);
    const LAS int* tab = (const LAS int*)(lds + LDS_MISC + 1280) + wr;
    const int ntile = gi < 32 ? 2 : 1;
    for (int ti = 0; ti < ntile; ++ti) {
        const int mt = gi >= NG ? 16 + (xcd >> 1) * 3 + (ix >> 2) : ti == 0 ? (xcd >> 1) * 4 + (ix >> 2) : 28 + (xcd >> 1);
        const int job = mt >> 4;
        const char* ga = wsl + par + (size_t)job * (64 * CHUNK) + (mt & 15) * 32768 + wid * 2048 + lane * 16;
        const char* gb = wsl + par + REL_B + nt * 65536 + wid * 4096 + lane * 16;
#define ISSUE(kt_, sb_) do { const char* _a = ga + (((kt_) >> 1) * CHUNK + ((kt_) & 1) * 16384); const char* _b = gb + (((kt_) >> 1) * CHUNK + ((kt_) & 1) * 32768); \
        LAS unsigned char* _la = lds + (sb_) + wid * 2048; LAS unsigned char* _lb = lds + (sb_) + 16384 + wid * 4096; \
        __builtin_amdgcn_global_load_lds((const unsigned*)_a, (LAS unsigned*)_la, 16, 0, 0); \
        __builtin_amdgcn_global_load_lds((const unsigned*)_a, (LAS unsigned*)_la, 16, 1024, 0); \
        __builtin_amdgcn_global_load_lds((const unsigned*)_b, (LAS unsigned*)_lb, 16, 0, 0); \
        __builtin_amdgcn_global_load_lds((const unsigned*)_b, (LAS unsigned*)_lb, 16, 1024, 0); \
        __builtin_amdgcn_global_load_lds((const unsigned*)_b, (LAS unsigned*)_lb, 16, 2048, 0); \
        __builtin_amdgcn_global_load_lds((const unsigned*)_b, (LAS unsigned*)_lb, 16, 3072, 0); } while (0)
#define LDF(A_, B_, sb_, kf_) do { _Pragma("unroll") for (int i = 0; i < 4; ++i) { \
        B_[i] = *(const LAS h16x8*)(lds + (sb_) + boff + i * 2048 + (kf_) * 1024); \
        A_[i] = *(const LAS h16x8*)(lds + (sb_) + aoff + i * 2048 + (kf_) * 1024); } } while (0)
#define MMA(A_, B_) do { __builtin_amdgcn_s_setprio(1); _Pragma("unroll") for (int m = 0; m < 4; ++m) _Pragma("unroll") for (int c = 0; c < 4; ++c) \
        acc[m][c] = __builtin_amdgcn_mfma_f32_16x16x32_f16(B_[c], A_[m], acc[m][c], 0, 0, 0); __builtin_amdgcn_s_setprio(0); } while (0)
        f32x4 acc[4][4], tot[4][4];
#pragma unroll
        for (int m = 0; m < 4; ++m)
#pragma unroll
            for (int c = 0; c < 4; ++c) acc[m][c] = tot[m][c] = (f32x4){0.f, 0.f, 0.f, 0.f};
        h16x8 A0[4], B0[4], A1[4], B1[4];
        if (wid == 0) wait_inputs(P, lds, job * 8, mt & 15, nt, call, lane);
        BAR();
        ISSUE(0, 0); ISSUE(1, STAGE); ISSUE(2, 2 * STAGE);
        WAIT_V(12); BAR();
        LDF(A0, B0, 0u, 0);
        int tnext = tab[0];
        int tcur = __builtin_amdgcn_readfirstlane(tnext);
        int tref = tcur;
        unsigned sb = 0;
        for (int kt = 0; kt < 128; ++kt) {
            const unsigned sn = sb == 2 * STAGE ? 0u : sb + STAGE;
            __builtin_amdgcn_s_waitcnt(0xC07F);
            LDF(A1, B1, sb, 1);
            const int tn = __builtin_amdgcn_readfirstlane(tnext);
            if (tn != tcur) {
                const bool nearby = abs(tn - tref) <= 30;
                const int ekeep = nearby ? tcur - tn : -2000, ebank = nearby ? -2000 : tcur - 2 * EXP_TARGET;
#pragma unroll
                for (int m = 0; m < 4; ++m)
#pragma unroll
                    for (int c = 0; c < 4; ++c)
#pragma unroll
                        for (int r = 0; r < 4; ++r) {
                            tot[m][c][r] += __builtin_ldexpf(acc[m][c][r], ebank);
                            acc[m][c][r] = __builtin_ldexpf(acc[m][c][r], ekeep);
                        }
                tref = nearby ? tref : tn;
                tcur = tn;
            }
            MMA(A0, B0);
            if (((kt + 3) & 15) == 0 && kt + 3 < 128 && wid == 0) wait_inputs(P, lds, job * 8 + ((kt + 3) >> 4), mt & 15, nt, call, lane);
            WAIT_V(6); __builtin_amdgcn_s_waitcnt(0xC07F); BAR();
            ISSUE((kt + 3) & 127, sb);
            LDF(A0, B0, sn, 0);
            tnext = tab[min(kt + 1, 127) * 2];
            MMA(A1, B1);
            sb = sn;
        }
        WAIT_V(0); WAIT_L(0); BAR();
        unsigned* tmo = (unsigned*)(wsl + OFF_TMO);
        const bool bad = (unsigned)__builtin_amdgcn_readfirstlane(__hip_atomic_load(tmo, __ATOMIC_RELAXED, __HIP_MEMORY_SCOPE_AGENT)) == call;
        LAS float* cv = (LAS float*)(lds + wid * 16384);
        unsigned mlo = 0, mhi = 0, off = 0;
        const int unsc = tcur - 2 * EXP_TARGET;
#pragma unroll
        for (int m = 0; m < 4; ++m)
#pragma unroll
            for (int c = 0; c < 4; ++c)
#pragma unroll
                for (int r = 0; r < 4; ++r) {
                    float v = tot[m][c][r] + __builtin_ldexpf(acc[m][c][r], unsc);
                    v = v <= 0.f ? 0.f : __builtin_fabsf(v);
                    if (bad) v = __builtin_nanf("");
                    acc[m][c][r] = v;
                    const bool nz = __float_as_uint(v) != 0u;
                    const u64 mk = __builtin_amdgcn_ballot_w64(nz);
                    const unsigned lo = (unsigned)mk, hi = (unsigned)(mk >> 32);
                    const unsigned pos = off + __builtin_amdgcn_mbcnt_hi(hi, __builtin_amdgcn_mbcnt_lo(lo, 0u));
                    if (nz) cv[pos] = v;
                    const int q = (m * 4 + c) * 4 + r;
                    mlo = lane == q ? lo : mlo; mhi = lane == q ? hi : mhi;
                    off += (unsigned)__builtin_popcountll(mk);
                }
        const int rec = (mt * 8 + nt) * 8 + wid;
        LAS unsigned* nz8 = (LAS unsigned*)(lds + LDS_MISC + 1024);
        if (lane == 0) nz8[wid] = off;
        WAIT_L(0); BAR();
        const unsigned nzl = nz8[lane & 7], lenl = rec_bytes(nzl, rec >= NSTG);
        const __amdgpu_buffer_rsrc_t rl = mk_rsrc(wsl, WS_USED);
        const unsigned ro = par + REL_O + (unsigned)(P.rank * (NREC / 8) + (rec >> 3)) * TILE_BYTES + (unsigned)__builtin_amdgcn_readlane(scan8(lenl, lane) - lenl, wid);
        const unsigned ng = rec_groups(off, rec >= NSTG);
        if (lane < 64u - ng) __builtin_amdgcn_raw_buffer_store_b64((u32x2){mlo, mhi}, rl, lane * 8 + ro + rec_hdr_off(off), 0, AUX_SYS);
        WAIT_L(0);
        for (unsigned g = 0; g < ng; ++g) {
            const u64 mk = ((u64)(unsigned)__builtin_amdgcn_readlane(mhi, 64u - ng + g) << 32) | (unsigned)__builtin_amdgcn_readlane(mlo, 64u - ng + g);
            __hip_atomic_fetch_or((LAS unsigned*)cv + g * 64 + lane, __builtin_amdgcn_inverse_ballot_w64(mk) ? 0x80000000u : 0u, __ATOMIC_RELAXED, __HIP_MEMORY_SCOPE_WORKGROUP);
        }
        WAIT_L(0);
        const __amdgpu_buffer_rsrc_t rv = mk_rsrc(wsl + ro, off * 4u);
        for (unsigned i = 0; i * 256 < off; ++i) {
            const u32x4 d = *(const LAS u32x4*)((LAS unsigned char*)cv + i * 1024 + lane * 16);
            __builtin_amdgcn_raw_buffer_store_b128(d, rv, lane * 16 + i * 1024, 0, AUX_SYS);
        }
        WAIT_V(0); WAIT_L(0); BAR();
        if (wid == 0 && (lane >> 3) != P.rank)
            __hip_atomic_store((u64*)(P.ws[lane >> 3] + OFF_RECFLAG) + (unsigned)(P.rank * NREC + rec + (lane & 7)), ((u64)call << 32) | nzl, __ATOMIC_RELAXED, __HIP_MEMORY_SCOPE_SYSTEM);
        store_dense(P.out, (P.rank >> 2) * 4096 + mt * 128 + wr * 64, (P.rank & 3) * 2048 + nt * 256 + wc * 64, fr, fq, acc);
        WAIT_L(0); BAR();
#undef ISSUE
#undef LDF
#undef MMA
    }
}

__device__ __forceinline__ void expand_role(const Params& P, LAS unsigned char* lds, unsigned call, int wid, int lane) {
    const int fr = lane & 15, fq = lane >> 4;
    constexpr unsigned NJOB = 7 * NSTG, E = NE * EW;
    const int gi = blockIdx.x - NL;
    const bool direct = gi < 32 || gi >= NG;
    const unsigned didx = (unsigned)(wid * 128 + (gi < 32 ? gi : gi - 96));
    if (direct ? didx >= 7u * DIRR : wid >= EW) return;
    const unsigned par = OFF_DATA + (call & 1u) * PAR_BYTES;
    char* wsl = P.wsl;
    unsigned* tmo = (unsigned*)(wsl + OFF_TMO);
    LAS unsigned char* buf = lds + wid * 16384;
    unsigned* q = (unsigned*)(wsl + OFF_EXPQ);
    const unsigned qbase = (call - 1u) * (NJOB + E);
    unsigned tk = qbase + NJOB + didx;
    if (!direct) { if (lane == 0) tk = __hip_atomic_fetch_add(q, 1u, __ATOMIC_RELAXED, __HIP_MEMORY_SCOPE_AGENT); }
    else {
        Deadline dl(tmo, call);
        while ((unsigned)__builtin_amdgcn_readfirstlane(__hip_atomic_load((unsigned*)(wsl + OFF_PULLED + (unsigned)(320 + NSTG - TAILR - 32) * 64u), __ATOMIC_RELAXED, __HIP_MEMORY_SCOPE_AGENT)) < call) {
            if (dl.expired()) break;
            __builtin_amdgcn_s_sleep(127); __builtin_amdgcn_s_sleep(127);
        }
    }
    int it = direct ? 1 : 0;
    unsigned vz = 0;
    asm volatile("" : "+v"(vz));
    for (;; ++it) {
        const unsigned ku = (unsigned)__builtin_amdgcn_readfirstlane(tk) - qbase;
        if (ku >= NJOB && !direct) break;
        const int k = (int)ku;
        const int rec = k / 7, src = (P.rank + 1 + (k - rec * 7)) & 7;
        const int crec = direct ? rec - DIRR : lane == 9 && (rec & 7) != 7 ? rec + 1 : rec;
        const u64* flag = (const u64*)(wsl + (lane < 8 ? OFF_RECFLAG + (unsigned)(src * NREC + (rec & ~7) + lane) * 8u : OFF_PULLED + (unsigned)(320 + crec) * 64u));
        const unsigned need = rec >= NSTG - TAILR ? TAILP * call : call;
        Deadline dl(tmo, call);
        u64 f = 0; bool bad = false;
        for (;;) {
            if (lane < 10) f = __hip_atomic_load(flag, __ATOMIC_RELAXED, __HIP_MEMORY_SCOPE_SYSTEM);
            if (__builtin_amdgcn_ballot_w64(lane < 8 ? (unsigned)(f >> 32) < call : lane < 10 && (unsigned)f < need) == 0ull) break;
            if (dl.expired()) { bad = true; break; }
            if (it == 0) __builtin_amdgcn_s_sleep(100); else __builtin_amdgcn_s_sleep(4);
        }
        if (__builtin_amdgcn_ballot_w64(lane < 8 && (unsigned)f > 4096u) != 0ull) bad = true;
        f32x4 v[4][4];
        if (!bad) {
            const unsigned nnz = __builtin_amdgcn_readlane((unsigned)f, rec & 7), lenl = rec_bytes((unsigned)f, direct);
            const char* rbase = (direct ? P.ws[src] : wsl) + par + REL_O + (size_t)(src * (NREC / 8) + (rec >> 3)) * TILE_BYTES
                                + (unsigned)__builtin_amdgcn_readlane(scan8(lenl, lane) - lenl, rec & 7);
            const unsigned ng = rec_groups(nnz, direct);
            u32x2 hdr = __builtin_amdgcn_raw_buffer_load_b64(mk_rsrc(rbase + rec_hdr_off(nnz), 8u * (64u - ng)), lane * 8, 0, AUX_SYS);
            const char* gsrc = rbase + lane * 16;
            const unsigned bytes = nnz * 4;
            for (unsigned i = 0; i < bytes; i += 1024)
                if (i + lane * 16 < bytes) __builtin_amdgcn_global_load_lds((const unsigned*)(gsrc + i), (LAS unsigned*)(buf + i), 16, 0, AUX_SYS);
            WAIT_V(0);
            for (unsigned g0 = 0; g0 < ng; g0 += 8) {
                int t[8];
#pragma unroll
                for (int u = 0; u < 8; ++u) t[u] = *(const LAS int*)(buf + (((g0 + u) & 63u) * 64 + lane) * 4);
#pragma unroll
                for (int u = 0; u < 8; ++u) {
                    const u64 mk = __builtin_amdgcn_ballot_w64(t[u] < 0);
                    const bool me = (unsigned)lane == 64u - ng + g0 + u;
                    hdr[0] = me ? (unsigned)mk : hdr[0]; hdr[1] = me ? (unsigned)(mk >> 32) : hdr[1];
                }
            }
            unsigned off = 0;
#pragma unroll
            for (int m = 0; m < 4; ++m)
#pragma unroll
                for (int c = 0; c < 4; ++c)
#pragma unroll
                    for (int r = 0; r < 4; ++r) {
                        const int qq = (m * 4 + c) * 4 + r;
                        const unsigned lo = __builtin_amdgcn_readlane(hdr[0], qq), hi = __builtin_amdgcn_readlane(hdr[1], qq);
                        const u64 mk = ((u64)hi << 32) | lo;
                        const unsigned pos = (off + __builtin_amdgcn_mbcnt_hi(hi, __builtin_amdgcn_mbcnt_lo(lo, 0u))) & 4095u;
                        const float t = *(const LAS float*)(buf + pos * 4);
                        v[m][c][r] = __builtin_amdgcn_inverse_ballot_w64(mk) ? __builtin_fabsf(t) : 0.f;
                        off += (unsigned)__builtin_popcountll(mk);
                    }
        } else {
#pragma unroll
            for (int m = 0; m < 4; ++m)
#pragma unroll
                for (int c = 0; c < 4; ++c) { const float qn = __builtin_nanf(""); v[m][c] = (f32x4){qn, qn, qn, qn}; }
        }
        if (lane == 0 && !direct) tk = __hip_atomic_fetch_add(q + vz, 1u, __ATOMIC_RELAXED, __HIP_MEMORY_SCOPE_AGENT);
        const int w8 = rec & 7, nt = (rec >> 3) & 7, mt = rec >> 6;
        const int row0 = (src >> 2) * 4096 + mt * 128 + (w8 >> 2) * 64, col0 = (src & 3) * 2048 + nt * 256 + (w8 & 3) * 64;
        store_dense(P.out, row0, col0, fr, fq, v);
        WAIT_L(0);
        if (direct) break;
    }
}

__global__ __launch_bounds__(512, 2) void fused_kernel(Params P) {
    extern __shared__ __attribute__((aligned(16))) unsigned char shm[];
    LAS unsigned char* lds = (LAS unsigned char*)shm;
    const int tid = threadIdx.x, lane = tid & 63, wid = __builtin_amdgcn_readfirstlane(tid >> 6);
    LAS unsigned* misc = (LAS unsigned*)(lds + LDS_MISC);
    unsigned ticket = 0, vz = 0;
    asm volatile("" : "+v"(vz));
    if (tid == 0) ticket = __hip_atomic_fetch_add((unsigned*)(P.wsl + OFF_CTR + (blockIdx.x & 7) * 128 + vz), 1u, __ATOMIC_RELAXED, __HIP_MEMORY_SCOPE_AGENT);
    if (tid >= 64 && tid < 128) misc[tid] = 0u;
    auto ready = [&]() -> unsigned {
        asm volatile("" : "+v"(ticket));
        if (tid == 0) misc[0] = ticket / (unsigned)(GRID / 8) + 1u;
        WAIT_L(0); BAR();
        return __builtin_amdgcn_readfirstlane(misc[0]);
    };
    if (blockIdx.x < NL) link_role(P, ready(), wid, lane);
    else {
        unsigned call;
        if (blockIdx.x < NL + NG) call = convert_first(P, lds, ready, wid, lane);
        else { call = ready(); convert_role(P, lds, call, wid, lane); }
        gemm_role(P, lds, call, wid, lane);
        expand_role(P, lds, call, wid, lane);
    }
}

}

extern "C" void kernel_launch(int rank, int nranks, ncclComm_t comm,
                              hipStream_t stream,
                              void* const* d_in, const int* in_sizes, int n_in,
                              void* d_out, int out_size,
                              const int* in_dtypes, int out_dtype,
                              void* const* d_out_all,
                              void* const* d_ws_all, size_t ws_bytes) {
    using namespace gar;
    if (nranks != NRANK || (unsigned)rank >= (unsigned)NRANK || ws_bytes < (size_t)WS_USED || n_in < 2 ||
        in_sizes[0] != GM * KS || in_sizes[1] != KS * GN || out_size != GM * GN) {
        fprintf(stderr, "kernel_launch: unexpected problem shape\n");
        return;
    }
    int dev = 0, ncu = 0;
    HIP_CHECK(hipGetDevice(&dev));
    HIP_CHECK(hipDeviceGetAttribute(&ncu, hipDeviceAttributeMultiprocessorCount, dev));
    if (ncu < GRID) { fprintf(stderr, "kernel_launch: needs %d CUs, found %d\n", GRID, ncu); return; }
    Params P;
    P.x = (const float*)d_in[0]; P.w = (const float*)d_in[1]; P.out = (float*)d_out; P.rank = rank;
    for (int p = 0; p < NRANK; ++p) P.ws[p] = (char*)d_ws_all[p];
    P.wsl = P.ws[rank]; P.pad = 0;
    for (int k = 0; k < 7; ++k) P.peer[k] = P.ws[(rank + 1 + k) & 7];
    HIP_CHECK(hipFuncSetAttribute((const void*)fused_kernel, hipFuncAttributeMaxDynamicSharedMemorySize, LDS_BYTES));
    hipLaunchKernelGGL(fused_kernel, dim3(GRID), dim3(512), LDS_BYTES, stream, P);
    HIP_CHECK(hipGetLastError());
}
```

```cpp
#include <hip/hip_runtime.h>
#include <rccl/rccl.h>
#include <cstdio>

#define HIP_CHECK(expr) do { hipError_t _e = (expr); if (_e != hipSuccess) { \
    fprintf(stderr, "HIP error %s at %s:%d\n", hipGetErrorString(_e), __FILE__, __LINE__); \
    return; } } while (0)

namespace gar {

#define LAS __attribute__((address_space(3)))
#define GAS __attribute__((address_space(1)))
typedef _Float16 h16x8 __attribute__((ext_vector_type(8)));
typedef float f32x4 __attribute__((ext_vector_type(4)));
typedef unsigned u32x4 __attribute__((ext_vector_type(4)));
typedef unsigned u32x2 __attribute__((ext_vector_type(2)));
typedef unsigned long long u64;

constexpr int NRANK = 8;
constexpr int GM = 8192, GN = 8192, KS = 1024;
constexpr int PW = 4;
constexpr int EW = 5;
constexpr int TAILR = 192, TAILP = 4;
constexpr int DIRR = 40;
constexpr int NL = 32;
constexpr int NG = 128;
constexpr int NE = 96;
constexpr int GRID = NL + NG + NE;
constexpr int NREC = 2048;
constexpr int NSTG = NREC - DIRR;
constexpr unsigned TILE_BYTES = 8 * 16896;
static_assert(GRID == 256 && DIRR % 8 == 0 && TAILR % 8 == 0 && DIRR <= TAILR && 7 * DIRR <= 8 * 128 && PW <= 8 && EW <= 8, "geometry");
constexpr unsigned NITEM_IN = 24 * 64;

constexpr unsigned OFF_TMO = 0, OFF_EXPQ = 128, OFF_LNKQ = 256, OFF_CTR = 1024;
constexpr unsigned OFF_CVW = 4096;
constexpr unsigned OFF_CVX = 0x180000;
constexpr unsigned OFF_PULLED = 65536;
constexpr unsigned OFF_RECFLAG = 0x140000;
constexpr unsigned OFF_DATA = 2u << 20;
constexpr unsigned CHUNK = 512u << 10;
constexpr unsigned A_BYTES = 128 * CHUNK, B_BYTES = 64 * CHUNK, XO_BYTES = 16 * CHUNK, WO_BYTES = 32 * CHUNK, O_BYTES = NRANK * (NREC / 8) * TILE_BYTES;
constexpr unsigned REL_B = A_BYTES, REL_XO = REL_B + B_BYTES, REL_WO = REL_XO + XO_BYTES, REL_O = REL_WO + WO_BYTES;
constexpr unsigned PAR_BYTES = REL_O + O_BYTES;
constexpr unsigned WS_USED = OFF_DATA + 2 * PAR_BYTES;
constexpr unsigned STAGE = 49152, LDS_MISC = 3 * STAGE, LDS_BYTES = LDS_MISC + 2304;
constexpr u64 DEADLINE = 150000000ull;
constexpr int AUX_SYS = 17;
constexpr int EXP_TARGET = 142;

struct Params {
    const float* x; char* wsl; int rank; int pad;
    const float* w; float* out; char* ws[NRANK];
    char* peer[7];
};

#define WAIT_V(n) asm volatile("s_waitcnt vmcnt(" #n ")" ::: "memory")
#define WAIT_L(n) asm volatile("s_waitcnt lgkmcnt(" #n ")" ::: "memory")
#define BAR() do { asm volatile("" ::: "memory"); __builtin_amdgcn_s_barrier(); asm volatile("" ::: "memory"); } while (0)

__device__ __forceinline__ __amdgpu_buffer_rsrc_t mk_rsrc(const void* p, unsigned bytes) {
    return __builtin_amdgcn_make_buffer_rsrc((void*)p, 0, bytes, 0x00020000);
}
__device__ __forceinline__ void st16_sys(char* p, u32x4 v) {
    __builtin_amdgcn_global_store_b128((GAS u32x4*)p, v, "");
}
__device__ __forceinline__ u32x4 cvt8(f32x4 a, f32x4 b, int sh) {
    h16x8 h;
#pragma unroll
    for (int j = 0; j < 4; ++j) { h[j] = (_Float16)__builtin_ldexpf(a[j], sh); h[4 + j] = (_Float16)__builtin_ldexpf(b[j], sh); }
    return __builtin_bit_cast(u32x4, h);
}
__device__ __forceinline__ unsigned rec_groups(unsigned nnz, bool direct) { return direct ? 0u : nnz >> 6; }
__device__ __forceinline__ unsigned rec_hdr_off(unsigned nnz) { return (nnz * 4u + 7u) & ~7u; }
__device__ __forceinline__ unsigned rec_bytes(unsigned nnz, bool direct) { return (rec_hdr_off(nnz) + 8u * (64u - rec_groups(nnz, direct)) + 15u) & ~15u; }
__device__ __forceinline__ unsigned scan8(unsigned v, int lane) {
    unsigned t;
    t = (unsigned)__builtin_amdgcn_update_dpp(0, (int)v, 0x111, 0xf, 0xf, true); v += (lane & 7) >= 1 ? t : 0u;
    t = (unsigned)__builtin_amdgcn_update_dpp(0, (int)v, 0x112, 0xf, 0xf, true); v += (lane & 7) >= 2 ? t : 0u;
    t = (unsigned)__builtin_amdgcn_update_dpp(0, (int)v, 0x114, 0xf, 0xf, true); v += (lane & 7) >= 4 ? t : 0u;
    return v;
}
struct Deadline {
    unsigned* tmo; unsigned call; u64 t0; unsigned polls;
    __device__ __forceinline__ Deadline(unsigned* tmo_, unsigned call_) : tmo(tmo_), call(call_), t0(__builtin_amdgcn_s_memrealtime()), polls(1) {}
    __device__ __forceinline__ bool expired() {
        if ((polls++ & 63u) == 0u &&
            (unsigned)__builtin_amdgcn_readfirstlane(__hip_atomic_load(tmo, __ATOMIC_RELAXED, __HIP_MEMORY_SCOPE_AGENT)) == call) return true;
        if (__builtin_amdgcn_s_memrealtime() - t0 > DEADLINE) {
            __hip_atomic_store(tmo, call, __ATOMIC_RELAXED, __HIP_MEMORY_SCOPE_AGENT);
            return true;
        }
        return false;
    }
};
__device__ __forceinline__ unsigned wave_max8(unsigned v) {
    unsigned r = 0;
#pragma unroll
    for (int b = 7; b >= 0; --b) { const unsigned c = r | (1u << b); r = __builtin_amdgcn_ballot_w64(v >= c) != 0ull ? c : r; }
    return r;
}
__device__ __forceinline__ unsigned exp1_max4(unsigned m, u32x4 v) {
#pragma unroll
    for (int i = 0; i < 4; ++i) m = max(m, (v[i] + 0x00800000u) & 0x7f800000u);
    return m;
}

__device__ __forceinline__ void store_dense(float* out, int row0, int col0, int fr, int fq, const f32x4 (&v)[4][4]) {
    float* p = out + (size_t)(row0 + fr) * GN + col0 + 4 * fq;
#pragma unroll
    for (int m = 0; m < 4; ++m)
#pragma unroll
        for (int r = 0; r < 4; ++r) {
            f32x4 o = {v[m][0][r], v[m][1][r], v[m][2][r], v[m][3][r]};
            st16_sys((char*)(p + (size_t)m * 16 * GN + r * 16), __builtin_bit_cast(u32x4, o));
        }
}

__device__ __forceinline__ unsigned block_code(LAS unsigned* word, unsigned m, int lane) {
    m = wave_max8(m >> 23);
    if (lane == 0) __hip_atomic_fetch_max(word, m, __ATOMIC_RELAXED, __HIP_MEMORY_SCOPE_WORKGROUP);
    WAIT_L(0); BAR();
    return min(((unsigned)__builtin_amdgcn_readfirstlane(*word) + 3u) & ~3u, 255u);
}
__device__ __forceinline__ void load_xblock(const Params& P, f32x4 (&v)[16], int half, int st, int mt, int wid, int fr, int fq) {
    const float* s = P.x + (size_t)(half * 4096 + (st >> 3) * 2048 + mt * 128 + wid * 16 + fr) * KS + (st & 7) * 128 + fq * 8;
#pragma unroll
    for (int i = 0; i < 8; ++i) v[i] = *(const f32x4*)(s + (i >> 1) * 32 + (i & 1) * 4);
}

__device__ __forceinline__ void convert_role(const Params& P, LAS unsigned char* lds, unsigned call, int wid, int lane) {
    const int fr = lane & 15, fq = lane >> 4, rank = P.rank;
    const unsigned par = OFF_DATA + (call & 1u) * PAR_BYTES;
    LAS unsigned* red = (LAS unsigned*)(lds + LDS_MISC + 256);
    const __amdgpu_buffer_rsrc_t rl = mk_rsrc(P.wsl, WS_USED);
    int n = 0;
    __builtin_amdgcn_s_sleep(64);
    for (int i = blockIdx.x - NL - NG; i < 23 * 32; i += NE, ++n) {
        const int u = (i >> 5) + 1, sub = i & 31;
        const bool isw = u < 16 && (u & 1);
        const int st = u < 16 ? u >> 1 : u - 8;
        f32x4 v[16];
        unsigned m = 0, foff; int dst;
        if (isw) {
            const int b = sub >> 3, nt = sub & 7;
            const float* s = P.w + (size_t)(st * 128 + (wid >> 2) * 64 + fq * 8) * GN + b * 2048 + nt * 256 + (wid & 3) * 64 + 4 * (4 * (fr & 3) + (fr >> 2));
#pragma unroll
            for (int e = 0; e < 16; ++e) v[e] = *(const f32x4*)(s + (size_t)((e >> 3) * 32 + (e & 7)) * GN);
#pragma unroll
            for (int e = 0; e < 16; ++e) m = exp1_max4(m, __builtin_bit_cast(u32x4, v[e]));
            m = block_code(red + n, m, lane);
            const int sh = EXP_TARGET - (int)m;
            const unsigned off = par + (b == (rank & 3) ? REL_B + (st * 8 + rank) * CHUNK : REL_WO + (b * 8 + st) * CHUNK) + nt * 65536 + wid * 8192;
#pragma unroll
            for (int c = 0; c < 4; ++c)
#pragma unroll
                for (int kf = 0; kf < 2; ++kf) {
                    const f32x4* q = v + 8 * kf;
                    const u32x4 h = cvt8((f32x4){q[0][c], q[1][c], q[2][c], q[3][c]}, (f32x4){q[4][c], q[5][c], q[6][c], q[7][c]}, sh);
                    __builtin_amdgcn_raw_buffer_store_b128(h, rl, lane * 16 + off + c * 2048 + kf * 1024, 0, AUX_SYS);
                }
            foff = OFF_CVW + (unsigned)((st * 8 + nt) * 8 + rank) * 8u; dst = lane < 2 ? b + 4 * lane : -1;
        } else {
            const int half = sub >> 4, mt = sub & 15;
            load_xblock(P, v, half, st, mt, wid, fr, fq);
#pragma unroll
            for (int e = 0; e < 8; ++e) m = exp1_max4(m, __builtin_bit_cast(u32x4, v[e]));
            m = block_code(red + n, m, lane);
            const int sh = EXP_TARGET - (int)m;
            const unsigned off = par + (half == (rank >> 2) ? (st * 8 + rank) * CHUNK : REL_XO + st * CHUNK) + mt * 32768 + wid * 2048;
#pragma unroll
            for (int kq = 0; kq < 4; ++kq)
                __builtin_amdgcn_raw_buffer_store_b128(cvt8(v[2 * kq], v[2 * kq + 1], sh), rl, lane * 16 + off + (kq >> 1) * 16384 + (kq & 1) * 1024, 0, AUX_SYS);
            foff = OFF_CVX + (unsigned)(((st * 16 + mt) * 8 + rank) * 4 + (lane & 3)) * 8u; dst = lane < 16 ? half * 4 + (lane >> 2) : -1;
        }
        WAIT_V(0); BAR();
        if (wid == 0 && dst >= 0)
            __hip_atomic_store((u64*)(P.ws[dst & 7] + foff), ((u64)call << 32) | m, __ATOMIC_RELAXED, __HIP_MEMORY_SCOPE_SYSTEM);
    }
}

template <class F>
__device__ __forceinline__ unsigned convert_first(const Params& P, LAS unsigned char* lds, F&& ready, int wid, int lane) {
    const int gi = blockIdx.x - NL, half = gi >> 6, mt = (gi >> 2) & 15, sub = gi & 3, rank = P.rank;
    const float* s = P.x + (size_t)(half * 4096 + mt * 128 + (sub & 1) * 64 + (wid >> 1) * 16 + (lane & 15)) * KS + (sub >> 1) * 64 + (wid & 1) * 32 + (lane >> 4) * 8;
    const f32x4 v0 = *(const f32x4*)s, v1 = *(const f32x4*)(s + 4);
    GAS u64* dst = (GAS u64*)(P.ws[half * 4 + (lane & 3)] + OFF_CVX);
    const unsigned call = ready();
    asm volatile("" : "+v"(dst));
    LAS unsigned* red = (LAS unsigned*)(lds + LDS_MISC + 256);
    const unsigned m = block_code(red, exp1_max4(exp1_max4(0u, __builtin_bit_cast(u32x4, v0)), __builtin_bit_cast(u32x4, v1)), lane);
    const unsigned off = OFF_DATA + (call & 1u) * PAR_BYTES + (half == (rank >> 2) ? rank * CHUNK : REL_XO) + mt * 32768 + sub * 8192 + wid * 1024 + lane * 16;
    __builtin_amdgcn_raw_buffer_store_b128(cvt8(v0, v1, EXP_TARGET - (int)m), mk_rsrc(P.wsl, WS_USED), off, 0, AUX_SYS);
    WAIT_V(0); BAR();
    if (wid == 0 && lane < 4)
        __hip_atomic_store(dst + (mt * 8 + rank) * 4 + sub, ((u64)call << 32) | m, __ATOMIC_RELAXED, __HIP_MEMORY_SCOPE_SYSTEM);
    return call;
}

__device__ __forceinline__ void link_role(const Params& P, unsigned call, int wid, int lane) {
    if (wid >= PW) return;
    const int rank = P.rank;
    const unsigned par = OFF_DATA + (call & 1u) * PAR_BYTES;
    char* wsl = P.wsl;
    unsigned* tmo = (unsigned*)(wsl + OFF_TMO);
    const unsigned lo16 = lane * 16;
    unsigned* q = (unsigned*)(wsl + OFF_LNKQ);
    constexpr unsigned NITEM = NITEM_IN + NSTG + TAILR * (TAILP - 1);
    const unsigned qbase = (call - 1u) * NITEM - NL * PW;
    unsigned tk = qbase + wid * NL + blockIdx.x, vz = 0;
    asm volatile("" : "+v"(vz));
    const int lsrc = (rank + 1 + (lane >> 3)) & 7;
    int prev = -1;
    auto publish = [&]() {
        if (prev >= 0 && lane == 0) __hip_atomic_fetch_add((unsigned*)(wsl + OFF_PULLED + (unsigned)prev * 64u), 1u, __ATOMIC_RELAXED, __HIP_MEMORY_SCOPE_AGENT);
        prev = -1;
    };
    for (;;) {
        const unsigned it = (unsigned)__builtin_amdgcn_readfirstlane(tk) - qbase;
        if (it >= NITEM) break;
        if (lane == 0) tk = __hip_atomic_fetch_add(q + vz, 1u, __ATOMIC_RELAXED, __HIP_MEMORY_SCOPE_AGENT);
        const bool isrec = it >= NITEM_IN;
        int id, pick = 0; unsigned foff, soff, doff, o0 = 0, ostep = 1024;
        if (isrec) {
            unsigned rec = it - NITEM_IN;
            if (rec >= NSTG - TAILR) { const unsigned t = rec - (NSTG - TAILR); rec = NSTG - TAILR + t / TAILP; o0 = (t % TAILP) * ostep; ostep *= TAILP; }
            id = 320 + (int)rec; pick = rec & 7; foff = OFF_RECFLAG + (unsigned)(lsrc * NREC + (rec & ~7u) + (lane & 7)) * 8u;
            soff = doff = par + REL_O + (unsigned)(lsrc * (NREC / 8) + (rec >> 3)) * TILE_BYTES;
        } else {
            const int u = (int)(it >> 6), sub = it & 63;
            const bool isw = u < 16 && (u & 1);
            const int st = u < 16 ? u >> 1 : u - 8;
            id = isw ? 256 + st * 8 + (sub >> 3) : st * 16 + (sub >> 2);
            foff = isw ? OFF_CVW + (unsigned)((id - 256) * 8 + lsrc) * 8u : OFF_CVX + (unsigned)((id * 8 + lsrc) * 4 + (sub & 3)) * 8u;
            doff = par + (isw ? REL_B : 0u) + (unsigned)(st * 8 + lsrc) * CHUNK + sub * 8192;
            const bool same = isw ? (lsrc & 3) == (rank & 3) : (lsrc >> 2) == (rank >> 2);
            soff = same ? doff : par + (isw ? REL_WO + (unsigned)((rank & 3) * 8 + st) * CHUNK : REL_XO + (unsigned)st * CHUNK) + sub * 8192;
        }
        Deadline dl(tmo, call);
        u64 f = 0; bool bad = false;
        for (;;) {
            if (lane < 56) f = __hip_atomic_load((const u64*)(wsl + foff), __ATOMIC_RELAXED, __HIP_MEMORY_SCOPE_SYSTEM);
            const u64 anylate = __builtin_amdgcn_ballot_w64(lane < 56 && (unsigned)(f >> 32) < call);
            WAIT_V(0);
            publish();
            if (anylate == 0ull) break;
            if (dl.expired()) { bad = true; break; }
            __builtin_amdgcn_s_sleep(8);
        }
        if (bad || __builtin_amdgcn_ballot_w64(lane < 56 && (unsigned)f > 4096u) != 0ull) continue;
        unsigned lbytes = 8192u;
        if (isrec) {
            const unsigned len = rec_bytes((unsigned)f, false), end = scan8(len, lane), lo = (end - len) & ~127u;
            lbytes = ((lane & 7) == 7 ? end : end & ~127u) - lo; soff += lo; doff += lo;
        }
        __amdgpu_buffer_rsrc_t rp[7], rl[7];
        unsigned maxb = 0;
#pragma unroll
        for (int k = 0; k < 7; ++k) {
            const unsigned bytes = __builtin_amdgcn_readlane(lbytes, 8 * k + pick);
            rp[k] = mk_rsrc(P.peer[k] + (unsigned)__builtin_amdgcn_readlane(soff, 8 * k + pick), bytes);
            rl[k] = mk_rsrc(wsl + (unsigned)__builtin_amdgcn_readlane(doff, 8 * k + pick), bytes);
            maxb = max(maxb, bytes);
        }
        for (unsigned o = o0; o < maxb; o += ostep) {
            u32x4 d[7];
#pragma unroll
            for (int k = 0; k < 7; ++k) d[k] = __builtin_amdgcn_raw_buffer_load_b128(rp[k], lo16 + o, 0, AUX_SYS);
#pragma unroll
            for (int k = 0; k < 7; ++k) __builtin_amdgcn_raw_buffer_store_b128(d[k], rl[k], lo16 + o, 0, AUX_SYS);
        }
        prev = id;
    }
    WAIT_V(0);
    publish();
}

__device__ __forceinline__ void wait_inputs(const Params& P, LAS unsigned char* lds, int st, int mt, int nt, unsigned call, int lane) {
    char* wsl = P.wsl;
    const int bx = st * 16 + mt, bw = 256 + (st & 7) * 8 + nt;
    const unsigned off = lane < 32 ? OFF_CVX + (unsigned)(bx * 32 + lane) * 8u : lane < 40 ? OFF_CVW + (unsigned)((bw - 256) * 8 + (lane & 7)) * 8u
                                   : OFF_PULLED + (unsigned)(lane == 40 ? bx : bw) * 64u;
    const unsigned need = lane == 40 ? 4u * call : 8u * call;
    Deadline dl((unsigned*)(wsl + OFF_TMO), call);
    u64 f = 0;
    for (;;) {
        if (lane < 42) f = __hip_atomic_load((const u64*)(wsl + off), __ATOMIC_RELAXED, __HIP_MEMORY_SCOPE_SYSTEM);
        const bool late = lane < 40 ? (unsigned)(f >> 32) < call : lane < 42 && (unsigned)f < need;
        if (__builtin_amdgcn_ballot_w64(late) == 0ull) break;
        if (dl.expired()) break;
        if (st == 8) __builtin_amdgcn_s_sleep(127); else __builtin_amdgcn_s_sleep(32);
    }
    __builtin_amdgcn_fence(__ATOMIC_ACQUIRE, "");
    LAS int* tab = (LAS int*)(lds + LDS_MISC + 1280);
    const int cw = __shfl((int)((unsigned)f & 0xffu), 32 + ((lane >> 2) & 7));
    if (lane < 32) tab[(st & 7) * 32 + lane] = (int)((unsigned)f & 0xffu) + cw;
    WAIT_V(0); WAIT_L(0);
}

__device__ __forceinline__ void gemm_role(const Params& P, LAS unsigned char* lds, unsigned call, int wid, int lane) {
    const int gi = blockIdx.x - NL, xcd = gi & 7, ix = (gi & 127) >> 3;
    const int wr = wid >> 2, wc = wid & 3, fr = lane & 15, fq = lane >> 4;
    const unsigned par = OFF_DATA + (call & 1u) * PAR_BYTES;
    char* wsl = P.wsl;
    const int nt = (xcd & 1) * 4 + (ix & 3);
    const unsigned aoff = (unsigned)(wr * 8192 + lane * 16), boff = (unsigned)(16384 + wc * 8192 + lane * 16);
    const LAS int* tab = (const LAS int*)(lds + LDS_MISC + 1280) + wr;
    const int ntile = gi < 32 ? 2 : 1;
    for (int ti = 0; ti < ntile; ++ti) {
        const int mt = gi >= NG ? 16 + (xcd >> 1) * 3 + (ix >> 2) : ti == 0 ? (xcd >> 1) * 4 + (ix >> 2) : 28 + (xcd >> 1);
        const int job = mt >> 4;
        const char* ga = wsl + par + (size_t)job * (64 * CHUNK) + (mt & 15) * 32768 + wid * 2048 + lane * 16;
        const char* gb = wsl + par + REL_B + nt * 65536 + wid * 4096 + lane * 16;
#define ISSUE(kt_, sb_) do { const char* _a = ga + (((kt_) >> 1) * CHUNK + ((kt_) & 1) * 16384); const char* _b = gb + (((kt_) >> 1) * CHUNK + ((kt_) & 1) * 32768); \
        LAS unsigned char* _la = lds + (sb_) + wid * 2048; LAS unsigned char* _lb = lds + (sb_) + 16384 + wid * 4096; \
        __builtin_amdgcn_global_load_lds((const unsigned*)_a, (LAS unsigned*)_la, 16, 0, 0); \
        __builtin_amdgcn_global_load_lds((const unsigned*)_a, (LAS unsigned*)_la, 16, 1024, 0); \
        __builtin_amdgcn_global_load_lds((const unsigned*)_b, (LAS unsigned*)_lb, 16, 0, 0); \
        __builtin_amdgcn_global_load_lds((const unsigned*)_b, (LAS unsigned*)_lb, 16, 1024, 0); \
        __builtin_amdgcn_global_load_lds((const unsigned*)_b, (LAS unsigned*)_lb, 16, 2048, 0); \
        __builtin_amdgcn_global_load_lds((const unsigned*)_b, (LAS unsigned*)_lb, 16, 3072, 0); } while (0)
#define LDF(A_, B_, sb_, kf_) do { _Pragma("unroll") for (int i = 0; i < 4; ++i) { \
        B_[i] = *(const LAS h16x8*)(lds + (sb_) + boff + i * 2048 + (kf_) * 1024); \
        A_[i] = *(const LAS h16x8*)(lds + (sb_) + aoff + i * 2048 + (kf_) * 1024); } } while (0)
#define MMA(A_, B_) do { __builtin_amdgcn_s_setprio(1); _Pragma("unroll") for (int m = 0; m < 4; ++m) _Pragma("unroll") for (int c = 0; c < 4; ++c) \
        acc[m][c] = __builtin_amdgcn_mfma_f32_16x16x32_f16(B_[c], A_[m], acc[m][c], 0, 0, 0); __builtin_amdgcn_s_setprio(0); } while (0)
        f32x4 acc[4][4], tot[4][4];
#pragma unroll
        for (int m = 0; m < 4; ++m)
#pragma unroll
            for (int c = 0; c < 4; ++c) acc[m][c] = tot[m][c] = (f32x4){0.f, 0.f, 0.f, 0.f};
        h16x8 A0[4], B0[4], A1[4], B1[4];
        if (wid == 0) wait_inputs(P, lds, job * 8, mt & 15, nt, call, lane);
        BAR();
        ISSUE(0, 0); ISSUE(1, STAGE); ISSUE(2, 2 * STAGE);
        WAIT_V(12); BAR();
        LDF(A0, B0, 0u, 0);
        int tnext = tab[0];
        int tcur = __builtin_amdgcn_readfirstlane(tnext);
        int tref = tcur;
        unsigned sb = 0;
        for (int kt = 0; kt < 128; ++kt) {
            const unsigned sn = sb == 2 * STAGE ? 0u : sb + STAGE;
            __builtin_amdgcn_s_waitcnt(0xC07F);
            LDF(A1, B1, sb, 1);
            const int tn = __builtin_amdgcn_readfirstlane(tnext);
            if (tn != tcur) {
                const bool nearby = abs(tn - tref) <= 30;
                const int ekeep = nearby ? tcur - tn : -2000, ebank = nearby ? -2000 : tcur - 2 * EXP_TARGET;
#pragma unroll
                for (int m = 0; m < 4; ++m)
#pragma unroll
                    for (int c = 0; c < 4; ++c)
#pragma unroll
                        for (int r = 0; r < 4; ++r) {
                            tot[m][c][r] += __builtin_ldexpf(acc[m][c][r], ebank);
                            acc[m][c][r] = __builtin_ldexpf(acc[m][c][r], ekeep);
                        }
                tref = nearby ? tref : tn;
                tcur = tn;
            }
            MMA(A0, B0);
            if (((kt + 3) & 15) == 0 && kt + 3 < 128 && wid == 0) wait_inputs(P, lds, job * 8 + ((kt + 3) >> 4), mt & 15, nt, call, lane);
            WAIT_V(6); __builtin_amdgcn_s_waitcnt(0xC07F); BAR();
            ISSUE((kt + 3) & 127, sb);
            LDF(A0, B0, sn, 0);
            tnext = tab[min(kt + 1, 127) * 2];
            MMA(A1, B1);
            sb = sn;
        }
        WAIT_V(0); WAIT_L(0); BAR();
        unsigned* tmo = (unsigned*)(wsl + OFF_TMO);
        const bool bad = (unsigned)__builtin_amdgcn_readfirstlane(__hip_atomic_load(tmo, __ATOMIC_RELAXED, __HIP_MEMORY_SCOPE_AGENT)) == call;
        LAS float* cv = (LAS float*)(lds + wid * 16384);
        unsigned mlo = 0, mhi = 0, off = 0;
        const int unsc = tcur - 2 * EXP_TARGET;
#pragma unroll
        for (int m = 0; m < 4; ++m)
#pragma unroll
            for (int c = 0; c < 4; ++c)
#pragma unroll
                for (int r = 0; r < 4; ++r) {
                    float v = tot[m][c][r] + __builtin_ldexpf(acc[m][c][r], unsc);
                    v = v <= 0.f ? 0.f : __builtin_fabsf(v);
                    if (bad) v = __builtin_nanf("");
                    acc[m][c][r] = v;
                    const bool nz = __float_as_uint(v) != 0u;
                    const u64 mk = __builtin_amdgcn_ballot_w64(nz);
                    const unsigned lo = (unsigned)mk, hi = (unsigned)(mk >> 32);
                    const unsigned pos = off + __builtin_amdgcn_mbcnt_hi(hi, __builtin_amdgcn_mbcnt_lo(lo, 0u));
                    if (nz) cv[pos] = v;
                    const int q = (m * 4 + c) * 4 + r;
                    mlo = lane == q ? lo : mlo; mhi = lane == q ? hi : mhi;
                    off += (unsigned)__builtin_popcountll(mk);
                }
        const int rec = (mt * 8 + nt) * 8 + wid;
        LAS unsigned* nz8 = (LAS unsigned*)(lds + LDS_MISC + 1024);
        if (lane == 0) nz8[wid] = off;
        WAIT_L(0); BAR();
        const unsigned nzl = nz8[lane & 7], lenl = rec_bytes(nzl, rec >= NSTG);
        const __amdgpu_buffer_rsrc_t rl = mk_rsrc(wsl, WS_USED);
        const unsigned ro = par + REL_O + (unsigned)(P.rank * (NREC / 8) + (rec >> 3)) * TILE_BYTES + (unsigned)__builtin_amdgcn_readlane(scan8(lenl, lane) - lenl, wid);
        const unsigned ng = rec_groups(off, rec >= NSTG);
        if (lane < 64u - ng) __builtin_amdgcn_raw_buffer_store_b64((u32x2){mlo, mhi}, rl, lane * 8 + ro + rec_hdr_off(off), 0, AUX_SYS);
        WAIT_L(0);
        for (unsigned g = 0; g < ng; ++g) {
            const u64 mk = ((u64)(unsigned)__builtin_amdgcn_readlane(mhi, 64u - ng + g) << 32) | (unsigned)__builtin_amdgcn_readlane(mlo, 64u - ng + g);
            __hip_atomic_fetch_or((LAS unsigned*)cv + g * 64 + lane, __builtin_amdgcn_inverse_ballot_w64(mk) ? 0x80000000u : 0u, __ATOMIC_RELAXED, __HIP_MEMORY_SCOPE_WORKGROUP);
        }
        WAIT_L(0);
        const __amdgpu_buffer_rsrc_t rv = mk_rsrc(wsl + ro, off * 4u);
        for (unsigned i = 0; i * 256 < off; ++i) {
            const u32x4 d = *(const LAS u32x4*)((LAS unsigned char*)cv + i * 1024 + lane * 16);
            __builtin_amdgcn_raw_buffer_store_b128(d, rv, lane * 16 + i * 1024, 0, AUX_SYS);
        }
        WAIT_V(0); WAIT_L(0); BAR();
        if (wid == 0 && (lane >> 3) != P.rank)
            __hip_atomic_store((u64*)(P.ws[lane >> 3] + OFF_RECFLAG) + (unsigned)(P.rank * NREC + rec + (lane & 7)), ((u64)call << 32) | nzl, __ATOMIC_RELAXED, __HIP_MEMORY_SCOPE_SYSTEM);
        store_dense(P.out, (P.rank >> 2) * 4096 + mt * 128 + wr * 64, (P.rank & 3) * 2048 + nt * 256 + wc * 64, fr, fq, acc);
        WAIT_L(0); BAR();
#undef ISSUE
#undef LDF
#undef MMA
    }
}

__device__ __forceinline__ void expand_role(const Params& P, LAS unsigned char* lds, unsigned call, int wid, int lane) {
    const int fr = lane & 15, fq = lane >> 4;
    constexpr unsigned NJOB = 7 * NSTG, E = NE * EW;
    const int gi = blockIdx.x - NL;
    const bool direct = gi < 32 || gi >= NG;
    const unsigned didx = (unsigned)(wid * 128 + (gi < 32 ? gi : gi - 96));
    if (direct ? didx >= 7u * DIRR : wid >= EW) return;
    const unsigned par = OFF_DATA + (call & 1u) * PAR_BYTES;
    char* wsl = P.wsl;
    unsigned* tmo = (unsigned*)(wsl + OFF_TMO);
    LAS unsigned char* buf = lds + wid * 16384;
    unsigned* q = (unsigned*)(wsl + OFF_EXPQ);
    const unsigned qbase = (call - 1u) * (NJOB + E);
    unsigned tk = qbase + NJOB + didx;
    if (!direct) { if (lane == 0) tk = __hip_atomic_fetch_add(q, 1u, __ATOMIC_RELAXED, __HIP_MEMORY_SCOPE_AGENT); }
    else {
        Deadline dl(tmo, call);
        while ((unsigned)__builtin_amdgcn_readfirstlane(__hip_atomic_load((unsigned*)(wsl + OFF_PULLED + (unsigned)(320 + NSTG - TAILR - 32) * 64u), __ATOMIC_RELAXED, __HIP_MEMORY_SCOPE_AGENT)) < call) {
            if (dl.expired()) break;
            __builtin_amdgcn_s_sleep(127); __builtin_amdgcn_s_sleep(127);
        }
    }
    int it = direct ? 1 : 0;
    unsigned vz = 0;
    asm volatile("" : "+v"(vz));
    for (;; ++it) {
        const unsigned ku = (unsigned)__builtin_amdgcn_readfirstlane(tk) - qbase;
        if (ku >= NJOB && !direct) break;
        const int k = (int)ku;
        const int rec = k / 7, src = (P.rank + 1 + (k - rec * 7)) & 7;
        const int crec = direct ? rec - DIRR : lane == 9 && (rec & 7) != 7 ? rec + 1 : rec;
        const u64* flag = (const u64*)(wsl + (lane < 8 ? OFF_RECFLAG + (unsigned)(src * NREC + (rec & ~7) + lane) * 8u : OFF_PULLED + (unsigned)(320 + crec) * 64u));
        const unsigned need = rec >= NSTG - TAILR ? TAILP * call : call;
        Deadline dl(tmo, call);
        u64 f = 0; bool bad = false;
        for (;;) {
            if (lane < 10) f = __hip_atomic_load(flag, __ATOMIC_RELAXED, __HIP_MEMORY_SCOPE_SYSTEM);
            if (__builtin_amdgcn_ballot_w64(lane < 8 ? (unsigned)(f >> 32) < call : lane < 10 && (unsigned)f < need) == 0ull) break;
            if (dl.expired()) { bad = true; break; }
            if (it == 0) __builtin_amdgcn_s_sleep(100); else __builtin_amdgcn_s_sleep(4);
        }
        if (__builtin_amdgcn_ballot_w64(lane < 8 && (unsigned)f > 4096u) != 0ull) bad = true;
        f32x4 v[4][4];
        if (!bad) {
            const unsigned nnz = __builtin_amdgcn_readlane((unsigned)f, rec & 7), lenl = rec_bytes((unsigned)f, direct);
            const char* rbase = (direct ? P.ws[src] : wsl) + par + REL_O + (size_t)(src * (NREC / 8) + (rec >> 3)) * TILE_BYTES
                                + (unsigned)__builtin_amdgcn_readlane(scan8(lenl, lane) - lenl, rec & 7);
            const unsigned ng = rec_groups(nnz, direct);
            u32x2 hdr = __builtin_amdgcn_raw_buffer_load_b64(mk_rsrc(rbase + rec_hdr_off(nnz), 8u * (64u - ng)), lane * 8, 0, AUX_SYS);
            const char* gsrc = rbase + lane * 16;
            const unsigned bytes = nnz * 4;
            for (unsigned i = 0; i < bytes; i += 1024)
                if (i + lane * 16 < bytes) __builtin_amdgcn_global_load_lds((const unsigned*)(gsrc + i), (LAS unsigned*)(buf + i), 16, 0, AUX_SYS);
            WAIT_V(0);
            for (unsigned g0 = 0; g0 < ng; g0 += 8) {
                int t[8];
#pragma unroll
                for (int u = 0; u < 8; ++u) t[u] = *(const LAS int*)(buf + (((g0 + u) & 63u) * 64 + lane) * 4);
#pragma unroll
                for (int u = 0; u < 8; ++u) {
                    const u64 mk = __builtin_amdgcn_ballot_w64(t[u] < 0);
                    const bool me = (unsigned)lane == 64u - ng + g0 + u;
                    hdr[0] = me ? (unsigned)mk : hdr[0]; hdr[1] = me ? (unsigned)(mk >> 32) : hdr[1];
                }
            }
            unsigned off = 0;
#pragma unroll
            for (int m = 0; m < 4; ++m)
#pragma unroll
                for (int c = 0; c < 4; ++c)
#pragma unroll
                    for (int r = 0; r < 4; ++r) {
                        const int qq = (m * 4 + c) * 4 + r;
                        const unsigned lo = __builtin_amdgcn_readlane(hdr[0], qq), hi = __builtin_amdgcn_readlane(hdr[1], qq);
                        const u64 mk = ((u64)hi << 32) | lo;
                        const unsigned pos = (off + __builtin_amdgcn_mbcnt_hi(hi, __builtin_amdgcn_mbcnt_lo(lo, 0u))) & 4095u;
                        const float t = *(const LAS float*)(buf + pos * 4);
                        v[m][c][r] = __builtin_amdgcn_inverse_ballot_w64(mk) ? __builtin_fabsf(t) : 0.f;
                        off += (unsigned)__builtin_popcountll(mk);
                    }
        } else {
#pragma unroll
            for (int m = 0; m < 4; ++m)
#pragma unroll
                for (int c = 0; c < 4; ++c) { const float qn = __builtin_nanf(""); v[m][c] = (f32x4){qn, qn, qn, qn}; }
        }
        if (lane == 0 && !direct) tk = __hip_atomic_fetch_add(q + vz, 1u, __ATOMIC_RELAXED, __HIP_MEMORY_SCOPE_AGENT);
        const int w8 = rec & 7, nt = (rec >> 3) & 7, mt = rec >> 6;
        const int row0 = (src >> 2) * 4096 + mt * 128 + (w8 >> 2) * 64, col0 = (src & 3) * 2048 + nt * 256 + (w8 & 3) * 64;
        store_dense(P.out, row0, col0, fr, fq, v);
        WAIT_L(0);
        if (direct) break;
    }
}

__global__ __launch_bounds__(512, 2) void fused_kernel(Params P) {
    extern __shared__ __attribute__((aligned(16))) unsigned char shm[];
    LAS unsigned char* lds = (LAS unsigned char*)shm;
    const int tid = threadIdx.x, lane = tid & 63, wid = __builtin_amdgcn_readfirstlane(tid >> 6);
    LAS unsigned* misc = (LAS unsigned*)(lds + LDS_MISC);
    unsigned ticket = 0, vz = 0;
    asm volatile("" : "+v"(vz));
    if (tid == 0) ticket = __hip_atomic_fetch_add((unsigned*)(P.wsl + OFF_CTR + (blockIdx.x & 7) * 128 + vz), 1u, __ATOMIC_RELAXED, __HIP_MEMORY_SCOPE_AGENT);
    if (tid >= 64 && tid < 128) misc[tid] = 0u;
    auto ready = [&]() -> unsigned {
        asm volatile("" : "+v"(ticket));
        if (tid == 0) misc[0] = ticket / (unsigned)(GRID / 8) + 1u;
        WAIT_L(0); BAR();
        return __builtin_amdgcn_readfirstlane(misc[0]);
    };
    if (blockIdx.x < NL) link_role(P, ready(), wid, lane);
    else {
        unsigned call;
        if (blockIdx.x < NL + NG) call = convert_first(P, lds, ready, wid, lane);
        else { call = ready(); convert_role(P, lds, call, wid, lane); }
        gemm_role(P, lds, call, wid, lane);
        expand_role(P, lds, call, wid, lane);
    }
}

}

extern "C" void kernel_launch(int rank, int nranks, ncclComm_t comm,
                              hipStream_t stream,
                              void* const* d_in, const int* in_sizes, int n_in,
                              void* d_out, int out_size,
                              const int* in_dtypes, int out_dtype,
                              void* const* d_out_all,
                              void* const* d_ws_all, size_t ws_bytes) {
    using namespace gar;
    if (nranks != NRANK || (unsigned)rank >= (unsigned)NRANK || ws_bytes < (size_t)WS_USED || n_in < 2 ||
        in_sizes[0] != GM * KS || in_sizes[1] != KS * GN || out_size != GM * GN) {
        fprintf(stderr, "kernel_launch: unexpected problem shape\n");
        return;
    }
    int dev = 0, ncu = 0;
    HIP_CHECK(hipGetDevice(&dev));
    HIP_CHECK(hipDeviceGetAttribute(&ncu, hipDeviceAttributeMultiprocessorCount, dev));
    if (ncu < GRID) { fprintf(stderr, "kernel_launch: needs %d CUs, found %d\n", GRID, ncu); return; }
    Params P;
    P.x = (const float*)d_in[0]; P.w = (const float*)d_in[1]; P.out = (float*)d_out; P.rank = rank;
    for (int p = 0; p < NRANK; ++p) P.ws[p] = (char*)d_ws_all[p];
    P.wsl = P.ws[rank]; P.pad = 0;
    for (int k = 0; k < 7; ++k) P.peer[k] = P.ws[(rank + 1 + k) & 7];
    HIP_CHECK(hipFuncSetAttribute((const void*)fused_kernel, hipFuncAttributeMaxDynamicSharedMemorySize, LDS_BYTES));
    hipLaunchKernelGGL(fused_kernel, dim3(GRID), dim3(512), LDS_BYTES, stream, P);
    HIP_CHECK(hipGetLastError());
}
```
